```python
import numpy as np

kernel_name = "gpu_gemm_a2a_m4096_k4096_n16384_bf16_reluq8"


ABSMAX_RTOL = 0.02

COMPUTE_DTYPE = "bfloat16"
EPILOGUE = "relu_quant_int8"
IN_QUANT = None
OUT_QUANT = "int8"
IN_DTYPES = {"x": "bfloat16", "w_mat": "bfloat16"}
WORLD_SIZE = 8

def _to_bf16(x):
    u = np.ascontiguousarray(x, dtype=np.float32).view(np.uint32)
    return ((u + 0x7FFF + ((u >> 16) & 1)) & 0xFFFF0000).view(np.float32)

def setup_inputs():

    x = _to_bf16(np.random.randn(4096, 4096).astype(np.float32))
    w_mat = _to_bf16((np.random.randn(4096, 16384) / np.sqrt(4096)).astype(np.float32))
    return {"x": x, "w_mat": w_mat}

def _quant_dequant(y, amax):
    scale = amax / 127.0
    q = np.clip(np.round(y / scale), -127.0, 127.0)
    return (q * scale).astype(np.float32)
def _epilogue(y):
    y = np.maximum(y, 0.0)
    return _quant_dequant(y, np.abs(y).max())

def reference(x, w_mat):
    y = (x @ w_mat).astype(np.float32)
    return _epilogue(y)
```

```cpp
#include <hip/hip_runtime.h>
#include <rccl/rccl.h>
#include <cstdio>

#define HIP_CHECK(expr) do { hipError_t _e = (expr); if (_e != hipSuccess) { \
    fprintf(stderr, "HIP error %s at %s:%d\n", hipGetErrorString(_e), __FILE__, __LINE__); \
    return; } } while (0)

#define LAS __attribute__((address_space(3)))
#define GAS __attribute__((address_space(1)))
typedef short s16x4 __attribute__((ext_vector_type(4)));
typedef __bf16 bf16x8 __attribute__((ext_vector_type(8)));
typedef float f32x4 __attribute__((ext_vector_type(4)));
typedef unsigned u32x4 __attribute__((ext_vector_type(4)));
typedef unsigned long long u64;
typedef unsigned short u16;

constexpr int NR = 8;
constexpr int MS = 512, NG = 16384, KD = 4096;
constexpr int NS = NG / NR;
constexpr int BM = 256, BN = 128, BK = 64, NT = KD / BK;
constexpr int NXS = 3, NWS = 4;
constexpr int XSLOT = BM * BK * 2, WSLOT = BK * BN * 2;
constexpr int XBASE = 0, WBASE = NXS * XSLOT;
constexpr int LDS_BYTES = NXS * XSLOT + NWS * WSLOT;
constexpr int EP_PITCH = BN * 4 + 16;
constexpr int EP_MISC = BM * EP_PITCH;
static_assert(EP_MISC + 256 <= LDS_BYTES, "lds");
constexpr int NWG = 256;

constexpr size_t WS_CTR = 0;
constexpr size_t WS_TMO = 16384;
constexpr size_t WS_LG = 20480;
constexpr int NREP = 8;
constexpr size_t WS_GG = 24576;
constexpr size_t WS_DF = 32768;
constexpr size_t WS_DIAG = 65536;
constexpr size_t WS_END = 65536 + 256 * 128;
constexpr u64 DEADLINE = 50000000ull;

struct Params {
    const u16* x; const u16* w;
    char* out[NR]; char* ws[NR];
    int rank;
};

#define WAIT_V(n) asm volatile("s_waitcnt vmcnt(" #n ")" ::: "memory")
#define BARRIER() asm volatile("s_barrier" ::: "memory")
#define RLX __ATOMIC_RELAXED
#define SC_AGENT __HIP_MEMORY_SCOPE_AGENT
#define SC_SYS __HIP_MEMORY_SCOPE_SYSTEM

__device__ __forceinline__ void dma16(__amdgpu_buffer_rsrc_t rsrc, LAS char* dst, unsigned voff, unsigned soff) {
    __builtin_amdgcn_raw_ptr_buffer_load_lds(rsrc, (LAS void*)dst, 16, voff, soff, 0, 0);
}
__device__ __forceinline__ unsigned q4(f32x4 v, float inv) {
    unsigned r = 0;
    r = __builtin_amdgcn_cvt_pk_u8_f32(__builtin_rintf(v[0] * inv), 0, r);
    r = __builtin_amdgcn_cvt_pk_u8_f32(__builtin_rintf(v[1] * inv), 1, r);
    r = __builtin_amdgcn_cvt_pk_u8_f32(__builtin_rintf(v[2] * inv), 2, r);
    r = __builtin_amdgcn_cvt_pk_u8_f32(__builtin_rintf(v[3] * inv), 3, r);
    return r;
}
#define STAMP(i) do { } while (0)

__global__ __launch_bounds__(512, 2) void fused_kernel(const Params p) {
    extern __shared__ __attribute__((aligned(16))) char smem[];
    LAS char* lds = (LAS char*)smem;
    const int tid = threadIdx.x, lane = tid & 63;
    const int wid = __builtin_amdgcn_readfirstlane(tid >> 6);
    const int b = blockIdx.x;
    const int dst = b & 7, idx = b >> 3;
    const int cbl = idx >> 1, mh = idx & 1;
    const int m0 = mh * BM, n0 = (dst * 16 + cbl) * BN;
    char* const myws = p.ws[p.rank];
    unsigned call = 0;
    float wmax = 0.f;

    if (wid >= 4) {
        const int lw = wid - 4;
        __amdgpu_buffer_rsrc_t rx = __builtin_amdgcn_make_buffer_rsrc((void*)p.x, 0, MS * KD * 2, 0x00020000);
        __amdgpu_buffer_rsrc_t rw = __builtin_amdgcn_make_buffer_rsrc((void*)p.w, 0, (unsigned)((size_t)KD * NG * 2), 0x00020000);
        const unsigned xrow = m0 + lw * 64 + (lane >> 3);
        const unsigned xv0 = xrow * (KD * 2) + (((lane & 7) ^ (lane >> 4)) << 4);
        const unsigned xv1 = xrow * (KD * 2) + (((lane & 7) ^ (4 + (lane >> 4))) << 4);
        const unsigned wrow = lw * 16 + (lane >> 4);
        const unsigned wv0 = wrow * (NG * 2) + n0 * 2 + (((lane & 15) ^ ((lane >> 4) << 1)) << 4);
        const unsigned wv1 = wrow * (NG * 2) + n0 * 2 + (((lane & 15) ^ (((lane >> 4) | 4) << 1)) << 4);
        auto issueX = [&](int t, int slot) {
            LAS char* d = lds + XBASE + slot * XSLOT + lw * 64 * 128;
#pragma unroll
            for (int i = 0; i < 8; ++i) dma16(rx, d + i * 1024, (i & 1) ? xv1 : xv0, (unsigned)(t * 128 + i * 8 * KD * 2));
        };
        auto issueW = [&](int t, int slot) {
            LAS char* d = lds + WBASE + slot * WSLOT + lw * 16 * 256;
#pragma unroll
            for (int i = 0; i < 4; ++i) dma16(rw, d + i * 1024, (i & 2) ? wv1 : wv0, (unsigned)((t * 64 + i * 4) * NG * 2));
        };
        issueW(0, 0); issueX(0, 0);
        issueW(1, 1); issueX(1, 1);
        issueW(2, 2); WAIT_V(16);
        BARRIER();
        int xs = 2, ws = NWS - 1;
        for (int t = 0; t < NT - 3; ++t) {
            issueX(t + 2, xs); issueW(t + 3, ws);
            xs = (xs == NXS - 1) ? 0 : xs + 1; ws = (ws == NWS - 1) ? 0 : ws + 1;
            WAIT_V(16);
            BARRIER();
        }
        issueX(NT - 1, xs);
        WAIT_V(12); BARRIER();
        WAIT_V(0); BARRIER();
        if (wid == 7) call = __hip_atomic_load((unsigned*)(myws + WS_CTR + b * 64), RLX, SC_AGENT) + 1u;
        BARRIER();
        BARRIER();
        BARRIER();
    } else {
        const int wm = wid >> 1, wn = wid & 1;
        if (wid == 0) STAMP(0);
        const int fr = lane & 15, g = lane >> 4;
        const unsigned xa0 = XBASE + wm * 16384 + fr * 128 + ((g ^ (fr >> 1)) << 4);
        const int q = fr >> 2, pp = lane & 3;
        const int f = q | ((g & 1) << 2);
        const unsigned wa0 = WBASE + (8 * g + q) * 256 + ((((wn * 8) + (pp >> 1)) ^ (f << 1)) << 4) + (pp & 1) * 8;
        f32x4 acc[8][4];
#pragma unroll
        for (int i = 0; i < 8; ++i)
#pragma unroll
            for (int j = 0; j < 4; ++j) acc[i][j] = (f32x4){0.f, 0.f, 0.f, 0.f};
        bf16x8 XL[4], XH[4], WA[4], WB[4];
        auto ldX = [&](bf16x8 (&d)[4], unsigned base, int mf0) {
#pragma unroll
            for (int i = 0; i < 4; ++i) d[i] = *(LAS bf16x8*)(lds + base + (mf0 + i) * 2048);
        };
        auto ldW = [&](bf16x8 (&d)[4], unsigned base, int ks) {
#pragma unroll
            for (int j = 0; j < 4; ++j) {
                unsigned a = base ^ (j << 5);
                s16x4 lo = __builtin_amdgcn_ds_read_tr16_b64_v4i16((LAS s16x4*)(lds + a + ks * 8192));
                s16x4 hi = __builtin_amdgcn_ds_read_tr16_b64_v4i16((LAS s16x4*)(lds + a + ks * 8192 + 1024));
                d[j] = __builtin_bit_cast(bf16x8, __builtin_shufflevector(lo, hi, 0, 1, 2, 3, 4, 5, 6, 7));
            }
        };
        auto mma = [&](int mf0, bf16x8 (&X)[4], bf16x8 (&W)[4]) {
#pragma unroll
            for (int i = 0; i < 4; ++i)
#pragma unroll
                for (int j = 0; j < 4; ++j)
                    acc[mf0 + i][j] = __builtin_amdgcn_mfma_f32_16x16x32_bf16(W[j], X[i], acc[mf0 + i][j], 0, 0, 0);
        };
#define SGB(mask, n) __builtin_amdgcn_sched_group_barrier(mask, n, 0)
#define SCHED_A() do { _Pragma("unroll") for (int _i = 0; _i < 4; ++_i) { SGB(0x8, 1); SGB(0x100, 1); } SGB(0x8, 12); __builtin_amdgcn_sched_barrier(0); } while (0)
#define SCHED_B() do { _Pragma("unroll") for (int _i = 0; _i < 12; ++_i) { SGB(0x8, 1); SGB(0x100, 1); } SGB(0x8, 4); __builtin_amdgcn_sched_barrier(0); } while (0)
        BARRIER();
        ldW(WA, wa0, 0); ldX(XL, xa0, 0);
        __builtin_amdgcn_sched_barrier(0);
        int xs = 0, ws = 0;
        __builtin_amdgcn_s_waitcnt(0xC07F);
        for (int t = 0; t < NT; ++t) {
            const unsigned xb = xa0 + xs * XSLOT, wb = wa0 + ws * WSLOT;
            xs = (xs == NXS - 1) ? 0 : xs + 1; ws = (ws == NWS - 1) ? 0 : ws + 1;
            const unsigned xbn = xa0 + xs * XSLOT, wbn = wa0 + ws * WSLOT;
            ldX(XH, xb, 4); mma(0, XL, WA); SCHED_A();
            ldW(WB, wb, 1); ldX(XL, xb ^ 64, 0); mma(4, XH, WA); SCHED_B();
            ldX(XH, xb ^ 64, 4); mma(0, XL, WB); SCHED_A();
            __builtin_amdgcn_s_waitcnt(0xC07F);
            BARRIER();
            __builtin_amdgcn_sched_barrier(0);
            ldW(WA, wbn, 0); ldX(XL, xbn, 0); mma(4, XH, WB); SCHED_B();
        }
        __builtin_amdgcn_s_waitcnt(0xC07F);
        BARRIER();
        float mx = 0.f;
#pragma unroll
        for (int i = 0; i < 8; ++i)
#pragma unroll
            for (int j = 0; j < 4; ++j)
                mx = __builtin_fmaxf(__builtin_fmaxf(mx, __builtin_fmaxf(acc[i][j][0], acc[i][j][1])), __builtin_fmaxf(acc[i][j][2], acc[i][j][3]));
#pragma unroll
        for (int o = 32; o >= 1; o >>= 1) mx = __builtin_fmaxf(mx, __shfl_xor(mx, o));
        if (lane == 0) *(LAS float*)(lds + EP_MISC + wid * 4) = mx;
        __builtin_amdgcn_s_waitcnt(0xC07F);
        BARRIER();
#pragma unroll
        for (int i = 0; i < 8; ++i)
#pragma unroll
            for (int j = 0; j < 4; ++j)
                *(LAS f32x4*)(lds + (wm * 128 + i * 16 + fr) * EP_PITCH + (wn * 64 + j * 16 + g * 4) * 4) = acc[i][j];
    }
    unsigned* const tmo = (unsigned*)(myws + WS_TMO);
    if (wid == 7) {
        call = __builtin_amdgcn_readfirstlane(call);
        STAMP(1);
        const f32x4 m4 = *(LAS f32x4*)(lds + EP_MISC);
        wmax = __builtin_fmaxf(__builtin_fmaxf(m4[0], m4[1]), __builtin_fmaxf(m4[2], m4[3]));
        if (lane == 0) {
            __hip_atomic_store((u64*)(myws + WS_LG) + b, ((u64)call << 32) | __float_as_uint(wmax), RLX, SC_AGENT);
            __hip_atomic_store((unsigned*)(myws + WS_CTR + b * 64), call, RLX, SC_AGENT);
        }
        const u64 t0 = __builtin_amdgcn_s_memrealtime();
        bool dead = false;
        if (b == 0) {
            unsigned m;
            for (;;) {
                bool ok = true; m = 0;
#pragma unroll
                for (int k = 0; k < 4; ++k) {
                    const u64 v = __hip_atomic_load((u64*)(myws + WS_LG) + k * 64 + lane, RLX, SC_AGENT);
                    ok &= (unsigned)(v >> 32) == call; m = max(m, (unsigned)v);
                }
                if (__all(ok)) break;
                if (__builtin_amdgcn_s_memrealtime() - t0 > DEADLINE) { dead = true; break; }
                __builtin_amdgcn_s_sleep(1);
            }
#pragma unroll
            for (int o = 32; o >= 1; o >>= 1) m = max(m, (unsigned)__shfl_xor((int)m, o));
            if (!dead) {
                char* base = p.ws[0];
#pragma unroll
                for (int r = 1; r < NR; ++r) base = (lane & 7) == r ? p.ws[r] : base;
                __hip_atomic_store((u64*)(base + WS_GG + ((call & 1) * NREP + (lane >> 3)) * 128) + p.rank, ((u64)call << 32) | m, RLX, SC_SYS);
            }
            STAMP(2);
        }
        unsigned gm;
        for (;;) {
            const u64 v = __hip_atomic_load((u64*)(myws + WS_GG + ((call & 1) * NREP + (b & (NREP - 1))) * 128) + (lane & 7), RLX, SC_SYS);
            gm = (unsigned)v;
            if (__all((unsigned)(v >> 32) == call)) break;
            if (dead || __hip_atomic_load(tmo, RLX, SC_AGENT) == call || __builtin_amdgcn_s_memrealtime() - t0 > DEADLINE) { dead = true; break; }
            __builtin_amdgcn_s_sleep(1);
        }
#pragma unroll
        for (int o = 4; o >= 1; o >>= 1) gm = max(gm, (unsigned)__shfl_xor((int)gm, o));
        if (dead && lane == 0) __hip_atomic_store(tmo, call, RLX, SC_AGENT);
        if (lane == 0) {
            const float am = __uint_as_float(gm);
            *(LAS float*)(lds + EP_MISC + 16) = am > 0.f ? 127.0f / am : 0.f;
            *(LAS unsigned*)(lds + EP_MISC + 20) = dead ? 1u : 0u;
        }
        STAMP(3);
    }
    __builtin_amdgcn_s_waitcnt(0xC07F);
    BARRIER();
    {
        const float inv = *(LAS float*)(lds + EP_MISC + 16);
        const bool dead = *(LAS unsigned*)(lds + EP_MISC + 20) != 0u;
        char* o = p.out[dst] + (size_t)(p.rank * MS + m0 + wid * 32 + (lane >> 3)) * NS + cbl * BN + (lane & 7) * 16;
#pragma unroll
        for (int s = 0; s < 4; ++s) {
            const LAS char* src = lds + (wid * 32 + s * 8 + (lane >> 3)) * EP_PITCH + (lane & 7) * 64;
            u32x4 v;
#pragma unroll
            for (int e = 0; e < 4; ++e) v[e] = q4(*(const LAS f32x4*)(src + e * 16), inv);
            if (dead) v = (u32x4){0x80808080u, 0x80808080u, 0x80808080u, 0x80808080u};
            asm volatile("global_store_dwordx4 %0, %1, off sc0 sc1\n\ts_nop 1" :: "v"(o + (size_t)s * 8 * NS), "v"(v) : "memory");
        }
    }
    WAIT_V(0);
    BARRIER();
    if (wid == 7) {
        STAMP(4);
        if (lane == 0) __hip_atomic_store((unsigned*)(p.ws[dst] + WS_DF + (p.rank * 32 + idx) * 64), call, RLX, SC_SYS);
        const u64 t0 = __builtin_amdgcn_s_memrealtime();
        while (__hip_atomic_load((unsigned*)(myws + WS_DF + b * 64), RLX, SC_SYS) < call) {
            if (__hip_atomic_load(tmo, RLX, SC_AGENT) == call || __builtin_amdgcn_s_memrealtime() - t0 > DEADLINE) {
                if (lane == 0) { __hip_atomic_store(tmo, call, RLX, SC_AGENT); p.out[p.rank][(size_t)(p.rank * MS) * NS] = (char)0x80; }
                break;
            }
            __builtin_amdgcn_s_sleep(1);
        }
        STAMP(5);
    }
}

extern "C" void kernel_launch(int rank, int nranks, ncclComm_t comm, hipStream_t stream,
                              void* const* d_in, const int* in_sizes, int n_in,
                              void* d_out, int out_size, const int* in_dtypes, int out_dtype,
                              void* const* d_out_all, void* const* d_ws_all, size_t ws_bytes) {
    if (nranks != NR || (unsigned)rank >= (unsigned)NR || ws_bytes < WS_END) return;
    static bool init[NR];
    if (!init[rank]) {
        HIP_CHECK(hipFuncSetAttribute((const void*)fused_kernel, hipFuncAttributeMaxDynamicSharedMemorySize, LDS_BYTES));
        init[rank] = true;
    }
    Params p;
    p.x = (const u16*)d_in[0]; p.w = (const u16*)d_in[1];
    for (int r = 0; r < NR; ++r) { p.out[r] = (char*)d_out_all[r]; p.ws[r] = (char*)d_ws_all[r]; }
    p.rank = rank;
    fused_kernel<<<NWG, 512, LDS_BYTES, stream>>>(p);
    HIP_CHECK(hipGetLastError());
}
```

```cpp
#include <hip/hip_runtime.h>
#include <rccl/rccl.h>
#include <cstdio>

#define HIP_CHECK(expr) do { hipError_t _e = (expr); if (_e != hipSuccess) { \
    fprintf(stderr, "HIP error %s at %s:%d\n", hipGetErrorString(_e), __FILE__, __LINE__); \
    return; } } while (0)

#define LAS __attribute__((address_space(3)))
typedef short s16x4 __attribute__((ext_vector_type(4)));
typedef __bf16 bf16x8 __attribute__((ext_vector_type(8)));
typedef float f32x4 __attribute__((ext_vector_type(4)));
typedef unsigned u32x4 __attribute__((ext_vector_type(4)));
typedef unsigned u32x2 __attribute__((ext_vector_type(2)));
typedef unsigned long long u64;
typedef unsigned short u16;

constexpr int NR = 8;
constexpr int MS = 512, NG = 16384, KD = 4096;
constexpr int NS = NG / NR;
constexpr int BM = 256, BN = 128, BK = 64, NT = KD / BK;
constexpr int NXS = 3, NWS = 4;
constexpr int XSLOT = BM * BK * 2, WSLOT = BK * BN * 2;
constexpr int XBASE = 0, WBASE = NXS * XSLOT;
constexpr int LDS_BYTES = NXS * XSLOT + NWS * WSLOT;
constexpr int NWG = 256;
constexpr int STRM = 8192 + 2048;
constexpr int EP_STRM = 0;
constexpr int EP_LUT = 8 * STRM;
constexpr int EP_SCALE = EP_LUT + 4096;

constexpr size_t WS_CTR = 0;
constexpr size_t WS_TMO = 16384;
constexpr size_t WS_LG = 20480;
constexpr int NREP = 32;
constexpr size_t WS_GG = 32768;
constexpr size_t WS_MF = 49152;
constexpr size_t WS_DIAG = 65536;
constexpr size_t WS_MR = 1u << 20;
constexpr size_t WS_RS = 3u << 20;
constexpr size_t WS_END = WS_RS + (size_t)NWG * 4 * STRM;
constexpr u64 DEADLINE = 50000000ull;
#define WAUX 2

#define PACE 1
#define OUTWT 1
#define NAGG 4
#define ASLEEP 1
#define RSLEEP 8
#define GSLEEP 1

struct Params {
    const u16* x; const u16* w;
    char* out; char* ws[NR];
    int rank;
};

#define WAIT_V(n) asm volatile("s_waitcnt vmcnt(" #n ")" ::: "memory")
#define BARRIER() asm volatile("s_barrier" ::: "memory")
#define RLX __ATOMIC_RELAXED
#define SC_AGENT __HIP_MEMORY_SCOPE_AGENT
#define SC_SYS __HIP_MEMORY_SCOPE_SYSTEM

template <int AUX> __device__ __forceinline__ void dma16(__amdgpu_buffer_rsrc_t rsrc, LAS char* dst, unsigned voff, unsigned soff) {
    __builtin_amdgcn_raw_ptr_buffer_load_lds(rsrc, (LAS void*)dst, 16, voff, soff, 0, AUX);
}
__device__ __forceinline__ void st16_sys(char* p, u32x4 v) {
    asm volatile("global_store_dwordx4 %0, %1, off sc0 sc1\n\ts_nop 1" :: "v"(p), "v"(v) : "memory");
}
__device__ __forceinline__ void st16_out(char* p, u32x4 v) {
    asm volatile("global_store_dwordx4 %0, %1, off sc1\n\ts_nop 1" :: "v"(p), "v"(v) : "memory");
}
__device__ __forceinline__ u32x4 ld16_sys(const char* p) {
    u32x4 v; asm volatile("global_load_dwordx4 %0, %1, off sc0 sc1\n\ts_waitcnt vmcnt(0)" : "=&v"(v) : "v"(p) : "memory"); return v;
}
__device__ __forceinline__ unsigned q4(f32x4 v, float inv) {
    unsigned r = 0;
    r = __builtin_amdgcn_cvt_pk_u8_f32(__builtin_rintf(v[0] * inv), 0, r);
    r = __builtin_amdgcn_cvt_pk_u8_f32(__builtin_rintf(v[1] * inv), 1, r);
    r = __builtin_amdgcn_cvt_pk_u8_f32(__builtin_rintf(v[2] * inv), 2, r);
    r = __builtin_amdgcn_cvt_pk_u8_f32(__builtin_rintf(v[3] * inv), 3, r);
    return r;
}
template <int CTRL, int RMASK> __device__ __forceinline__ unsigned dpp0(unsigned x) { return (unsigned)__builtin_amdgcn_update_dpp(0, (int)x, CTRL, RMASK, 0xf, true); }
__device__ __forceinline__ unsigned wave_scan_add(unsigned x) {
    x += dpp0<0x111, 0xf>(x); x += dpp0<0x112, 0xf>(x); x += dpp0<0x114, 0xf>(x); x += dpp0<0x118, 0xf>(x);
    x += dpp0<0x142, 0xa>(x); x += dpp0<0x143, 0xc>(x); return x;
}
__device__ __forceinline__ unsigned wave_max_u(unsigned x) {
    x = max(x, dpp0<0x111, 0xf>(x)); x = max(x, dpp0<0x112, 0xf>(x)); x = max(x, dpp0<0x114, 0xf>(x)); x = max(x, dpp0<0x118, 0xf>(x));
    x = max(x, dpp0<0x142, 0xa>(x)); x = max(x, dpp0<0x143, 0xc>(x)); return (unsigned)__builtin_amdgcn_readlane((int)x, 63);
}
__device__ __forceinline__ void stream_offsets(const u32x4 keep, unsigned (&start)[4], unsigned (&cum)[4]) {
    const unsigned c0 = __builtin_popcount(keep[0]), c1 = __builtin_popcount(keep[1]), c2 = __builtin_popcount(keep[2]), c3 = __builtin_popcount(keep[3]);
    const unsigned a = wave_scan_add(c0 | (c1 << 16)), b = wave_scan_add(c2 | (c3 << 16));
    const unsigned ta = (unsigned)__builtin_amdgcn_readlane((int)a, 63), tb = (unsigned)__builtin_amdgcn_readlane((int)b, 63);
    cum[0] = ta & 0xffffu; cum[1] = cum[0] + (ta >> 16); cum[2] = cum[1] + (tb & 0xffffu); cum[3] = cum[2] + (tb >> 16);
    start[0] = (a & 0xffffu) - c0; start[1] = cum[0] + (a >> 16) - c1; start[2] = cum[1] + (b & 0xffffu) - c2; start[3] = cum[2] + (b >> 16) - c3;
}
__device__ __forceinline__ unsigned stream_tag(unsigned call) {
    const unsigned n = call % 15u + 1u;
    return ((n & 1u) << 7) | ((n & 2u) << 14) | ((n & 4u) << 21) | ((n & 8u) << 28);
}
__device__ __forceinline__ u32x4 xpose4(unsigned d0, unsigned d1, unsigned d2, unsigned d3) {
    auto a = __builtin_amdgcn_permlane32_swap(d0, d2, false, false); auto b = __builtin_amdgcn_permlane32_swap(d1, d3, false, false);
    auto c = __builtin_amdgcn_permlane16_swap(a[0], b[0], false, false); auto d = __builtin_amdgcn_permlane16_swap(a[1], b[1], false, false);
    return (u32x4){c[0], c[1], d[0], d[1]};
}
#define STAMP(i) do { } while (0)

__global__ __launch_bounds__(512, 2) void fused_kernel(const Params p) {
    extern __shared__ __attribute__((aligned(16))) char smem[];
    LAS char* lds = (LAS char*)smem;
    const int tid = threadIdx.x, lane = tid & 63;
    const int wid = __builtin_amdgcn_readfirstlane(tid >> 6);
    const int b = blockIdx.x;
    const int peer = b & 7, idx = b >> 3;
    const int cbl = idx >> 1, mh = idx & 1;
    const int m0 = mh * BM, n0 = (peer * 16 + cbl) * BN;
    const bool local = peer == p.rank;
    char* const myws = p.ws[p.rank];
    unsigned* const tmo = (unsigned*)(myws + WS_TMO);
    const int fr = lane & 15, g = lane >> 4;
    const int w4 = wid & 3, wm = w4 >> 1, wn = w4 & 1;

    if (wid >= 4) {
        const int lw = w4;
        __amdgpu_buffer_rsrc_t rx = __builtin_amdgcn_make_buffer_rsrc((void*)p.x, 0, MS * KD * 2, 0x00020000);
        __amdgpu_buffer_rsrc_t rw = __builtin_amdgcn_make_buffer_rsrc((void*)p.w, 0, (unsigned)((size_t)KD * NG * 2), 0x00020000);
        const unsigned xrow = m0 + lw * 64 + (lane >> 3);
        const unsigned xv0 = xrow * (KD * 2) + (((lane & 7) ^ (lane >> 4)) << 4);
        const unsigned xv1 = xrow * (KD * 2) + (((lane & 7) ^ (4 + (lane >> 4))) << 4);
        const unsigned wrow = lw * 16 + (lane >> 4);
        const unsigned wv0 = wrow * (NG * 2) + n0 * 2 + (((lane & 15) ^ ((lane >> 4) << 1)) << 4);
        const unsigned wv1 = wrow * (NG * 2) + n0 * 2 + (((lane & 15) ^ (((lane >> 4) | 4) << 1)) << 4);
        auto issueX = [&](int t, int slot) {
            LAS char* d = lds + XBASE + slot * XSLOT + lw * 64 * 128;
#pragma unroll
            for (int i = 0; i < 8; ++i) dma16<0>(rx, d + i * 1024, (i & 1) ? xv1 : xv0, (unsigned)(t * 128 + i * 8 * KD * 2));
        };
        auto issueW = [&](int t, int slot) {
            LAS char* d = lds + WBASE + slot * WSLOT + lw * 16 * 256;
#pragma unroll
            for (int i = 0; i < 4; ++i) dma16<WAUX>(rw, d + i * 1024, (i & 2) ? wv1 : wv0, (unsigned)((t * 64 + i * 4) * NG * 2));
        };
        issueW(0, 0); issueX(0, 0);
        issueW(1, 1); issueX(1, 1);
        issueW(2, 2);
        unsigned touch = 0;
        asm volatile("buffer_load_dword %0, %1, %2, 0 offen" : "+v"(touch) : "v"((unsigned)(b * 16384 + lw * 4096 + (lane & 31) * 128)), "s"(rx) : "memory");
        WAIT_V(17);
        BARRIER();
        int xs = 2, ws = NWS - 1;
        for (int t = 0; t < NT - 3; ++t) {
            issueX(t + 2, xs); issueW(t + 3, ws);
            xs = (xs == NXS - 1) ? 0 : xs + 1; ws = (ws == NWS - 1) ? 0 : ws + 1;
            WAIT_V(16);
            BARRIER();
        }
        issueX(NT - 1, xs);
        char* const rs = myws + WS_RS + (size_t)(b * 4 + w4) * STRM;
        asm volatile("global_load_dword %0, %1, off sc1" : "+v"(touch) : "v"(myws + WS_CTR + b * 64) : "memory");
        WAIT_V(13); BARRIER();
        WAIT_V(0);
        asm volatile("" : "+v"(touch));
        const unsigned call = __builtin_amdgcn_readfirstlane(touch) + 1u;
        st16_sys(rs + (call & 7u) * 1024 + lane * 16, (u32x4){0u, 0u, 0u, 0u});
        BARRIER();
        BARRIER();
        if (wid == 4) asm volatile("ds_write_b128 %0, %1\n\ts_waitcnt lgkmcnt(0)" :: "v"((unsigned)EP_SCALE), "v"((u32x4){0u, 0u, 0u, 0u}) : "memory");
        WAIT_V(0);
        BARRIER();
        const unsigned tag = stream_tag(call);
        const u64 t0 = __builtin_amdgcn_s_memrealtime();
        if (local) {
            if (idx < NAGG) {
                bool dead = false, ok = false; unsigned m = 0;
                const char* gp = myws + WS_LG + w4 * 2048 + lane * 16;
                for (;;) {
                    if (!ok) {
                        u32x4 v0, v1;
                        asm volatile("global_load_dwordx4 %0, %2, off sc1\n\tglobal_load_dwordx4 %1, %2, off offset:1024 sc1\n\ts_waitcnt vmcnt(0)" : "=&v"(v0), "=&v"(v1) : "v"(gp) : "memory");
                        m = max(max(v0[0], v0[2]), max(v1[0], v1[2]));
                        ok = v0[1] == call && v0[3] == call && v1[1] == call && v1[3] == call;
                    }
                    if (__all(ok)) break;
                    if (__builtin_amdgcn_s_memrealtime() - t0 > DEADLINE) { dead = true; break; }
                    __builtin_amdgcn_s_sleep(ASLEEP);
                }
                if (!dead) {
                    m = wave_max_u(m);
                    unsigned cnt = 0;
                    if (lane == 0) {
                        __hip_atomic_fetch_max((LAS unsigned*)(lds + EP_SCALE + 8), m, RLX, __HIP_MEMORY_SCOPE_WORKGROUP);
                        cnt = __hip_atomic_fetch_add((LAS unsigned*)(lds + EP_SCALE + 12), 1u, RLX, __HIP_MEMORY_SCOPE_WORKGROUP);
                    }
                    if (__builtin_amdgcn_readfirstlane(cnt) == 3) {
                        m = __hip_atomic_load((LAS unsigned*)(lds + EP_SCALE + 8), RLX, __HIP_MEMORY_SCOPE_WORKGROUP);
                        char* base = p.ws[0];
#pragma unroll
                        for (int r = 1; r < NR; ++r) base = (lane & 7) == r ? p.ws[r] : base;
                        base += WS_GG + ((call & 1) * NREP + (lane >> 3)) * 128 + p.rank * 8;
#pragma unroll
                        for (int i = 0; i < NREP / 8; ++i) __hip_atomic_store((u64*)(base + i * 1024), ((u64)call << 32) | m, RLX, SC_SYS);
                        STAMP(2);
                    }
                } else if (lane == 0) __hip_atomic_store(tmo, call, RLX, SC_AGENT);
            }
        } else {
            bool dead = false;
            {
                const unsigned nib = lane & 15; unsigned pos = lane >> 4, sel = 0;
#pragma unroll
                for (int e = 0; e < 4; ++e) { const bool k = (nib >> e) & 1; sel |= (k ? pos : 0x0cu) << (8 * e); pos += k; }
                *(LAS unsigned*)(lds + EP_LUT + wid * 512 + lane * 4) = sel;
            }
            unsigned n = 0;
            while (__hip_atomic_load((unsigned*)(myws + WS_MF + (b * 4 + w4) * 16), RLX, SC_SYS) < call) {
                if (((++n & 63u) == 0u && __hip_atomic_load(tmo, RLX, SC_AGENT) == call) || __builtin_amdgcn_s_memrealtime() - t0 > DEADLINE) { dead = true; break; }
                __builtin_amdgcn_s_sleep(32);
            }
            const u32x4 keep = ld16_sys(myws + WS_MR + (size_t)(((call & 1) * NWG + b) * 4 + w4) * 1024 + lane * 16);
            unsigned start[4], cum[4];
            stream_offsets(keep, start, cum);
            unsigned addr[32], sel[32];
#pragma unroll
            for (int k = 0; k < 4; ++k) {
                unsigned off = start[k];
#pragma unroll
                for (int u = 0; u < 8; ++u) {
                    const unsigned nib = (keep[k] >> (4 * u)) & 15u;
                    sel[k * 8 + u] = *(LAS unsigned*)(lds + EP_LUT + wid * 512 + (((off & 3u) << 4) | nib) * 4);
                    addr[k * 8 + u] = EP_STRM + wid * STRM + (off & ~3u);
                    off += __builtin_popcount(nib);
                }
            }
            if (wid == 4) STAMP(6);
            while (__hip_atomic_load((LAS unsigned*)(lds + EP_SCALE + 4), RLX, __HIP_MEMORY_SCOPE_WORKGROUP) == 0u) __builtin_amdgcn_s_sleep(8);
            const unsigned total = cum[3];
            char* o = p.out + (size_t)(peer * MS + m0 + wm * 128 + fr) * NS + cbl * BN + wn * 64 + g * 16;
            unsigned pos = 0;
#pragma unroll
            for (int k = 0; k < 4; ++k) {
                while (pos < cum[k] && !dead) {
                    u32x4 v0, v1;
                    asm volatile("global_load_dwordx4 %0, %2, off sc0 sc1\n\tglobal_load_dwordx4 %1, %2, off offset:1024 sc0 sc1\n\ts_waitcnt vmcnt(0)"
                                 : "=&v"(v0), "=&v"(v1) : "v"(rs + pos + lane * 16) : "memory");
                    const bool ok0 = pos + lane * 16 >= total || ((((v0[0] ^ tag) | (v0[1] ^ tag) | (v0[2] ^ tag) | (v0[3] ^ tag)) & 0x80808080u) == 0u);
                    const bool ok1 = pos + 1024 + lane * 16 >= total || ((((v1[0] ^ tag) | (v1[1] ^ tag) | (v1[2] ^ tag) | (v1[3] ^ tag)) & 0x80808080u) == 0u);
                    const u64 bad0 = ~__ballot(ok0), bad1 = ~__ballot(ok1);
                    *(LAS u32x4*)(lds + EP_STRM + wid * STRM + pos + lane * 16) = v0 & 0x7f7f7f7fu;
                    *(LAS u32x4*)(lds + EP_STRM + wid * STRM + pos + 1024 + lane * 16) = v1 & 0x7f7f7f7fu;
                    const unsigned adv = bad0 ? (unsigned)__builtin_ctzll(bad0) * 16u : bad1 ? 1024u + (unsigned)__builtin_ctzll(bad1) * 16u : 2048u;
                    pos += adv & ~127u;
                    const bool bad = pos < cum[k];
                    if (bad) {
                        if (((++n & 63u) == 0u && __hip_atomic_load(tmo, RLX, SC_AGENT) == call) || __builtin_amdgcn_s_memrealtime() - t0 > DEADLINE) dead = true;
                        if (k < 3) __builtin_amdgcn_s_sleep(RSLEEP);
                    }
                }
                if (k == 3 && wid == 4) STAMP(7);
#pragma unroll
                for (int ii = 0; ii < 2; ++ii) {
                    const int i = 2 * k + ii;
                    unsigned d[4];
#pragma unroll
                    for (int j = 0; j < 4; ++j) {
                        const LAS unsigned* sp = (const LAS unsigned*)(lds + addr[i * 4 + j]);
                        d[j] = __builtin_amdgcn_perm(sp[1], sp[0], sel[i * 4 + j]);
                    }
                    u32x4 v = xpose4(d[0], d[1], d[2], d[3]);
                    if (dead) v = (u32x4){0x80808080u, 0x80808080u, 0x80808080u, 0x80808080u};
                    st16_out(o + (size_t)i * 16 * NS, v);
                }
            }
            if (dead && lane == 0) __hip_atomic_store(tmo, call, RLX, SC_AGENT);
            if (wid == 4) STAMP(5);
        }
    } else {
        if (wid == 0) STAMP(0);
        const unsigned xa0 = XBASE + wm * 16384 + fr * 128 + ((g ^ (fr >> 1)) << 4);
        const int qq = fr >> 2, pp = lane & 3;
        const int f = qq | ((g & 1) << 2);
        const unsigned wa0 = WBASE + (8 * g + qq) * 256 + ((((wn * 8) + (pp >> 1)) ^ (f << 1)) << 4) + (pp & 1) * 8;
        const unsigned c0 = __hip_atomic_load((unsigned*)(myws + WS_CTR + b * 64), RLX, SC_AGENT);
        f32x4 acc[8][4];
#pragma unroll
        for (int i = 0; i < 8; ++i)
#pragma unroll
            for (int j = 0; j < 4; ++j) acc[i][j] = (f32x4){0.f, 0.f, 0.f, 0.f};
        bf16x8 XL[4], XH[4], WA[4], WB[4];
        auto ldX = [&](bf16x8 (&d)[4], unsigned base, int mf0) {
#pragma unroll
            for (int i = 0; i < 4; ++i) d[i] = *(LAS bf16x8*)(lds + base + (mf0 + i) * 2048);
        };
        auto ldW = [&](bf16x8 (&d)[4], unsigned base, int ks) {
#pragma unroll
            for (int j = 0; j < 4; ++j) {
                unsigned a = base ^ (j << 5);
                s16x4 lo = __builtin_amdgcn_ds_read_tr16_b64_v4i16((LAS s16x4*)(lds + a + ks * 8192));
                s16x4 hi = __builtin_amdgcn_ds_read_tr16_b64_v4i16((LAS s16x4*)(lds + a + ks * 8192 + 1024));
                d[j] = __builtin_bit_cast(bf16x8, __builtin_shufflevector(lo, hi, 0, 1, 2, 3, 4, 5, 6, 7));
            }
        };
        auto mma = [&](int mf0, bf16x8 (&X)[4], bf16x8 (&W)[4]) {
#pragma unroll
            for (int i = 0; i < 4; ++i)
#pragma unroll
                for (int j = 0; j < 4; ++j)
                    acc[mf0 + i][j] = __builtin_amdgcn_mfma_f32_16x16x32_bf16(W[j], X[i], acc[mf0 + i][j], 0, 0, 0);
        };
#define SGB(mask, n) __builtin_amdgcn_sched_group_barrier(mask, n, 0)
#define SCHED_A() do { _Pragma("unroll") for (int _i = 0; _i < 4; ++_i) { SGB(0x8, 1); SGB(0x100, 1); } SGB(0x8, 12); __builtin_amdgcn_sched_barrier(0); } while (0)
#define SCHED_B() do { _Pragma("unroll") for (int _i = 0; _i < 12; ++_i) { SGB(0x8, 1); SGB(0x100, 1); } SGB(0x8, 4); __builtin_amdgcn_sched_barrier(0); } while (0)
        const unsigned call = __builtin_amdgcn_readfirstlane(c0) + 1u;
        BARRIER();
        ldW(WA, wa0, 0); ldX(XL, xa0, 0);
        __builtin_amdgcn_sched_barrier(0);
        int xs = 0, ws = 0;
        __builtin_amdgcn_s_waitcnt(0xC07F);
        for (int t = 0; t < NT; ++t) {
            const unsigned xb = xa0 + xs * XSLOT, wb = wa0 + ws * WSLOT;
            xs = (xs == NXS - 1) ? 0 : xs + 1; ws = (ws == NWS - 1) ? 0 : ws + 1;
            const unsigned xbn = xa0 + xs * XSLOT, wbn = wa0 + ws * WSLOT;
            ldX(XH, xb, 4); mma(0, XL, WA); SCHED_A();
            ldW(WB, wb, 1); ldX(XL, xb ^ 64, 0); mma(4, XH, WA); SCHED_B();
            ldX(XH, xb ^ 64, 4); mma(0, XL, WB); SCHED_A();
            __builtin_amdgcn_s_waitcnt(0xC07F);
            BARRIER();
            __builtin_amdgcn_sched_barrier(0);
            ldW(WA, wbn, 0); ldX(XL, xbn, 0); mma(4, XH, WB); SCHED_B();
        }
        __builtin_amdgcn_s_waitcnt(0xC07F);
        BARRIER();
        float mx = 0.f;
#pragma unroll
        for (int i = 0; i < 8; ++i)
#pragma unroll
            for (int j = 0; j < 4; ++j)
                mx = __builtin_fmaxf(__builtin_fmaxf(mx, __builtin_fmaxf(acc[i][j][0], acc[i][j][1])), __builtin_fmaxf(acc[i][j][2], acc[i][j][3]));
        const unsigned wmax = wave_max_u(__float_as_uint(mx));
        if (lane == 0) {
            __hip_atomic_store((u64*)(myws + WS_LG) + b * 4 + wid, ((u64)call << 32) | wmax, RLX, SC_AGENT);
            if (wid == 0) __hip_atomic_store((unsigned*)(myws + WS_CTR + b * 64), call, RLX, SC_AGENT);
        }
        if (wid == 0) STAMP(1);
        const u64 t0 = __builtin_amdgcn_s_memrealtime();
        auto wait_scale = [&](bool& dead) {
            LAS u64* sw = (LAS u64*)(lds + EP_SCALE);
            if (wid == 0) {
                unsigned gm, n = 0;
                for (;;) {
                    const u64 v = __hip_atomic_load((u64*)(myws + WS_GG + ((call & 1) * NREP + (idx & (NREP - 1))) * 128) + (lane & 7), RLX, SC_SYS);
                    gm = (unsigned)v;
                    if (__all((unsigned)(v >> 32) == call)) break;
                    if (((++n & 63u) == 0u && __hip_atomic_load(tmo, RLX, SC_AGENT) == call) || __builtin_amdgcn_s_memrealtime() - t0 > DEADLINE) { dead = true; break; }
                    __builtin_amdgcn_s_sleep(GSLEEP);
                }
                if (dead && lane == 0) __hip_atomic_store(tmo, call, RLX, SC_AGENT);
                const float am = __uint_as_float(wave_max_u(gm));
                const float inv = am > 0.f ? 127.0f / am : 0.f;
                __hip_atomic_store(sw, ((u64)(dead ? 2u : 1u) << 32) | __float_as_uint(inv), RLX, __HIP_MEMORY_SCOPE_WORKGROUP);
                return inv;
            }
            u64 v;
            while (((v = __hip_atomic_load(sw, RLX, __HIP_MEMORY_SCOPE_WORKGROUP)) >> 32) == 0u) __builtin_amdgcn_s_sleep(1);
            dead = (v >> 32) == 2u;
            return __uint_as_float((unsigned)v);
        };
        bool dead = false;
        if (local) {
            const float inv = wait_scale(dead);
            if (wid == 0) STAMP(3);
            char* o = p.out + (size_t)(p.rank * MS + m0 + wm * 128 + fr) * NS + cbl * BN + wn * 64 + g * 16;
#pragma unroll
            for (int i = 0; i < 8; ++i) {
                u32x4 v = xpose4(q4(acc[i][0], inv), q4(acc[i][1], inv), q4(acc[i][2], inv), q4(acc[i][3], inv));
                if (dead) v = (u32x4){0x80808080u, 0x80808080u, 0x80808080u, 0x80808080u};
                st16_out(o + (size_t)i * 16 * NS, v);
            }
        } else {
            const int slot = (idx * 8 + p.rank) * 4 + wid;
            char* const pws = p.ws[peer];
            const unsigned tag = stream_tag(call);
            u32x4 keep;
#pragma unroll
            for (int k = 0; k < 4; ++k) {
                unsigned m = 0;
#pragma unroll
                for (int u = 7; u >= 0; --u)
#pragma unroll
                    for (int e = 3; e >= 0; --e) m = __builtin_amdgcn_alignbit(m, __float_as_uint(acc[2 * k + (u >> 2)][u & 3][e]), 31);
                keep[k] = ~m;
            }
            {
                const unsigned nib = lane & 15; unsigned pos = lane >> 4; u64 sel = 0x0c0c0c0c0c0c0c0cull;
#pragma unroll
                for (int e = 0; e < 4; ++e) { const bool k = (nib >> e) & 1; const u64 x = (u64)(0x0cu ^ (unsigned)e) << (8 * pos); sel ^= k ? x : 0ull; pos += k; }
                *(LAS u64*)(lds + EP_LUT + wid * 512 + lane * 8) = sel;
            }
#pragma unroll
            for (int i = 0; i < 9; ++i) *(LAS u32x4*)(lds + EP_STRM + wid * STRM + i * 1024 + lane * 16) = (u32x4){tag, tag, tag, tag};
            unsigned start[4], cum[4];
            stream_offsets(keep, start, cum);
            unsigned addr[32]; u32x2 sel[32];
#pragma unroll
            for (int k = 0; k < 4; ++k) {
                unsigned off = start[k];
#pragma unroll
                for (int u = 0; u < 8; ++u) {
                    const unsigned nib = (keep[k] >> (4 * u)) & 15u;
                    sel[k * 8 + u] = *(LAS u32x2*)(lds + EP_LUT + wid * 512 + (((off & 3u) << 4) | nib) * 8);
                    addr[k * 8 + u] = EP_STRM + wid * STRM + (off & ~3u);
                    off += __builtin_popcount(nib);
                }
            }
#pragma unroll
            for (int i = 0; i < 32; ++i) asm volatile("" : "+v"(addr[i]), "+v"(sel[i]));
            if (wid == 0) STAMP(8);
            st16_sys(pws + WS_MR + (size_t)((call & 1) * NWG * 4 + slot) * 1024 + lane * 16, keep);
            WAIT_V(0);
            if (lane == 0) __hip_atomic_store((unsigned*)(pws + WS_MF + slot * 16), call, RLX, SC_SYS);
            const float inv = wait_scale(dead);
            if (wid == 0) STAMP(3);
            if (!dead) {
                char* const rs = pws + WS_RS + (size_t)slot * STRM + lane * 16;
                unsigned sent = 0;
#pragma unroll
                for (int k = 0; k < 4; ++k) {
#pragma unroll
                    for (int u = 0; u < 8; ++u) {
                        const unsigned qd = q4(acc[2 * k + (u >> 2)][u & 3], inv) & 0x7f7f7f7fu;
                        LAS unsigned* d = (LAS unsigned*)(lds + addr[k * 8 + u]);
                        __hip_atomic_fetch_or(d, __builtin_amdgcn_perm(0u, qd, sel[k * 8 + u][0]), RLX, __HIP_MEMORY_SCOPE_WORKGROUP);
                        __hip_atomic_fetch_or(d + 1, __builtin_amdgcn_perm(0u, qd, sel[k * 8 + u][1]), RLX, __HIP_MEMORY_SCOPE_WORKGROUP);
                    }
                    asm volatile("" ::: "memory");
                    const unsigned lim = k < 3 ? (cum[k] & ~127u) : ((cum[3] + 63u) & ~63u);
                    WAIT_V(0);
                    while (sent < lim) {
                        if (sent + lane * 16 < lim) st16_sys(rs + sent, *(LAS u32x4*)(lds + EP_STRM + wid * STRM + sent + lane * 16));
                        sent = min(sent + 1024u, lim);
                    }
                }
                if (wid == 0) STAMP(9);
            }
        }
    }
}

extern "C" void kernel_launch(int rank, int nranks, ncclComm_t comm, hipStream_t stream,
                              void* const* d_in, const int* in_sizes, int n_in,
                              void* d_out, int out_size, const int* in_dtypes, int out_dtype,
                              void* const* d_out_all, void* const* d_ws_all, size_t ws_bytes) {
    if (nranks != NR || (unsigned)rank >= (unsigned)NR || ws_bytes < WS_END) return;
    static bool init[NR];
    if (!init[rank]) {
        HIP_CHECK(hipFuncSetAttribute((const void*)fused_kernel, hipFuncAttributeMaxDynamicSharedMemorySize, LDS_BYTES));
        init[rank] = true;
    }
    Params p;
    p.x = (const u16*)d_in[0]; p.w = (const u16*)d_in[1]; p.out = (char*)d_out;
    for (int r = 0; r < NR; ++r) p.ws[r] = (char*)d_ws_all[r];
    p.rank = rank;
    fused_kernel<<<NWG, 512, LDS_BYTES, stream>>>(p);
    HIP_CHECK(hipGetLastError());
}
```

```cpp
#include <hip/hip_runtime.h>
#include <rccl/rccl.h>
#include <cstdio>

#define HIP_CHECK(expr) do { hipError_t _e = (expr); if (_e != hipSuccess) { \
    fprintf(stderr, "HIP error %s at %s:%d\n", hipGetErrorString(_e), __FILE__, __LINE__); \
    return; } } while (0)

#define LAS __attribute__((address_space(3)))
typedef short s16x4 __attribute__((ext_vector_type(4)));
typedef __bf16 bf16x8 __attribute__((ext_vector_type(8)));
typedef float f32x4 __attribute__((ext_vector_type(4)));
typedef unsigned u32x4 __attribute__((ext_vector_type(4)));
typedef unsigned u32x2 __attribute__((ext_vector_type(2)));
typedef unsigned long long u64;
typedef unsigned short u16;

constexpr int NR = 8;
constexpr int MS = 512, NG = 16384, KD = 4096;
constexpr int NS = NG / NR;
constexpr int BM = 256, BN = 128, BK = 64, NT = KD / BK;
constexpr int NXS = 3, NWS = 4;
constexpr int XSLOT = BM * BK * 2, WSLOT = BK * BN * 2;
constexpr int XBASE = 0, WBASE = NXS * XSLOT;
constexpr int LDS_BYTES = NXS * XSLOT + NWS * WSLOT;
constexpr int NWG = 256;
constexpr int STRM = 8192 + 1024;
constexpr int EP_STRM = 0;
constexpr int RCVB = 8192 + 5120;
constexpr int EP_RCV = 4 * STRM;
constexpr int EP_LUT = EP_RCV + 4 * RCVB;
constexpr int EP_SCALE = EP_LUT + 4096;

constexpr size_t WS_CTR = 0;
constexpr size_t WS_TMO = 16384;
constexpr size_t WS_LG = 20480;
#define NREPL 16
constexpr int NREP = NREPL;
constexpr size_t WS_GG = 32768;
constexpr size_t WS_MF = 49152;
constexpr size_t WS_DIAG = 65536;
constexpr size_t WS_MR = 1u << 20;
constexpr size_t WS_RS = 3u << 20;
constexpr size_t WS_END = WS_RS + (size_t)NWG * 4 * STRM;
constexpr u64 DEADLINE = 50000000ull;
#define WAUX 2

#define PACE 1
#define OUTWT 1
#define NAGG 2
#define ASLEEP 1
#define PROG 0
#define PLACE 0
#define TK0 3
#define RSLEEP 1
#define LINK_BPT 590u
#define SLOT0 6
#define RWAIT 60
#define GSLEEP 1

struct Params {
    const u16* x; const u16* w;
    char* out; char* myws;
    int rank; int pad;
    char* ws[NR];
};

#define WAIT_V(n) asm volatile("s_waitcnt vmcnt(" #n ")" ::: "memory")
#define BARRIER() asm volatile("s_barrier" ::: "memory")
#define RLX __ATOMIC_RELAXED
#define SC_AGENT __HIP_MEMORY_SCOPE_AGENT
#define SC_SYS __HIP_MEMORY_SCOPE_SYSTEM

template <int AUX> __device__ __forceinline__ void dma16(__amdgpu_buffer_rsrc_t rsrc, LAS char* dst, unsigned voff, unsigned soff) {
    __builtin_amdgcn_raw_ptr_buffer_load_lds(rsrc, (LAS void*)dst, 16, voff, soff, 0, AUX);
}
__device__ __forceinline__ void st16_sys(char* p, u32x4 v) {
    asm volatile("global_store_dwordx4 %0, %1, off sc0 sc1\n\ts_nop 1" :: "v"(p), "v"(v) : "memory");
}
__device__ __forceinline__ void st16_out(char* p, u32x4 v) {
    asm volatile("global_store_dwordx4 %0, %1, off sc1\n\ts_nop 1" :: "v"(p), "v"(v) : "memory");
}
__device__ __forceinline__ u32x4 ld16_sys(const char* p) {
    u32x4 v; asm volatile("global_load_dwordx4 %0, %1, off sc0 sc1\n\ts_waitcnt vmcnt(0)" : "=&v"(v) : "v"(p) : "memory"); return v;
}
__device__ __forceinline__ unsigned q4(f32x4 v, float inv) {
    unsigned r = 0;
    r = __builtin_amdgcn_cvt_pk_u8_f32(v[0] * inv, 0, r);
    r = __builtin_amdgcn_cvt_pk_u8_f32(v[1] * inv, 1, r);
    r = __builtin_amdgcn_cvt_pk_u8_f32(v[2] * inv, 2, r);
    r = __builtin_amdgcn_cvt_pk_u8_f32(v[3] * inv, 3, r);
    return r;
}
template <int CTRL, int RMASK> __device__ __forceinline__ unsigned dpp0(unsigned x) { return (unsigned)__builtin_amdgcn_update_dpp(0, (int)x, CTRL, RMASK, 0xf, true); }
__device__ __forceinline__ unsigned wave_scan_add(unsigned x) {
    x += dpp0<0x111, 0xf>(x); x += dpp0<0x112, 0xf>(x); x += dpp0<0x114, 0xf>(x); x += dpp0<0x118, 0xf>(x);
    x += dpp0<0x142, 0xa>(x); x += dpp0<0x143, 0xc>(x); return x;
}
__device__ __forceinline__ unsigned wave_max_u(unsigned x) {
    x = max(x, dpp0<0x111, 0xf>(x)); x = max(x, dpp0<0x112, 0xf>(x)); x = max(x, dpp0<0x114, 0xf>(x)); x = max(x, dpp0<0x118, 0xf>(x));
    x = max(x, dpp0<0x142, 0xa>(x)); x = max(x, dpp0<0x143, 0xc>(x)); return (unsigned)__builtin_amdgcn_readlane((int)x, 63);
}
__device__ __forceinline__ void stream_offsets(const u32x4 keep, unsigned (&start)[4], unsigned (&cum)[4]) {
    const unsigned c0 = __builtin_popcount(keep[0]), c1 = __builtin_popcount(keep[1]), c2 = __builtin_popcount(keep[2]), c3 = __builtin_popcount(keep[3]);
    const unsigned a = wave_scan_add(c0 | (c1 << 16)), b = wave_scan_add(c2 | (c3 << 16));
    const unsigned ta = (unsigned)__builtin_amdgcn_readlane((int)a, 63), tb = (unsigned)__builtin_amdgcn_readlane((int)b, 63);
    cum[0] = ta & 0xffffu; cum[1] = cum[0] + (ta >> 16); cum[2] = cum[1] + (tb & 0xffffu); cum[3] = cum[2] + (tb >> 16);
    start[0] = (a & 0xffffu) - c0; start[1] = cum[0] + (a >> 16) - c1; start[2] = cum[1] + (b & 0xffffu) - c2; start[3] = cum[2] + (b >> 16) - c3;
}
__device__ __forceinline__ unsigned stream_tag(unsigned call) {
    const unsigned n = call % 15u + 1u;
    return ((n & 1u) << 7) | ((n & 2u) << 14) | ((n & 4u) << 21) | ((n & 8u) << 28);
}
__device__ __forceinline__ u64 slot_ticks(int pos, unsigned bytes) {
    return (u64)(((unsigned)(32 - SLOT0 + 3 * pos) * bytes) / LINK_BPT);
}
__device__ __forceinline__ u32x4 xpose4(unsigned d0, unsigned d1, unsigned d2, unsigned d3) {
    auto a = __builtin_amdgcn_permlane32_swap(d0, d2, false, false); auto b = __builtin_amdgcn_permlane32_swap(d1, d3, false, false);
    auto c = __builtin_amdgcn_permlane16_swap(a[0], b[0], false, false); auto d = __builtin_amdgcn_permlane16_swap(a[1], b[1], false, false);
    return (u32x4){c[0], c[1], d[0], d[1]};
}
#define STAMP(i) do { } while (0)

__global__ __launch_bounds__(512, 2) void fused_kernel(const Params p) {
    extern __shared__ __attribute__((aligned(16))) char smem[];
    LAS char* lds = (LAS char*)smem;
    asm volatile("" :: "s"(p.x), "s"(p.w), "s"(p.myws), "s"(p.rank));
    const int tid = threadIdx.x, lane = tid & 63;
    const int wid = __builtin_amdgcn_readfirstlane(tid >> 6);
    const int peer = blockIdx.x & 7, idx = blockIdx.x >> 3;
    const int b = idx * 8 + peer;
    auto col_block = [](int i) { return (int)((0xd951fecba8764320ull >> (4 * i)) & 15u); };
    const int cbl = col_block(idx >> 1), mh = idx & 1;
    const int m0 = mh * BM, n0 = (peer * 16 + cbl) * BN;
    const bool local = peer == p.rank;
    char* const myws = p.myws;
    unsigned* const tmo = (unsigned*)(myws + WS_TMO);
    const int fr = lane & 15, g = lane >> 4;
    const int w4 = wid & 3, wm = w4 >> 1, wn = w4 & 1;

    if (__builtin_expect(wid >= 4, 1)) {
        const int lw = w4;
        __amdgpu_buffer_rsrc_t rx = __builtin_amdgcn_make_buffer_rsrc((void*)p.x, 0, MS * KD * 2, 0x00020000);
        __amdgpu_buffer_rsrc_t rw = __builtin_amdgcn_make_buffer_rsrc((void*)p.w, 0, (unsigned)((size_t)KD * NG * 2), 0x00020000);
        const unsigned xrow = m0 + lw * 64 + (lane >> 3);
        const unsigned xv0 = xrow * (KD * 2) + (((lane & 7) ^ (lane >> 4)) << 4);
        const unsigned xv1 = xrow * (KD * 2) + (((lane & 7) ^ (4 + (lane >> 4))) << 4);
        const unsigned wrow = lw * 16 + (lane >> 4);
        const unsigned wv0 = wrow * (NG * 2) + n0 * 2 + (((lane & 15) ^ ((lane >> 4) << 1)) << 4);
        const unsigned wv1 = wrow * (NG * 2) + n0 * 2 + (((lane & 15) ^ (((lane >> 4) | 4) << 1)) << 4);
        auto issueX = [&](int t, int slot) {
            LAS char* d = lds + XBASE + slot * XSLOT + lw * 64 * 128;
#pragma unroll
            for (int i = 0; i < 8; ++i) dma16<0>(rx, d + i * 1024, (i & 1) ? xv1 : xv0, (unsigned)(t * 128 + i * 8 * KD * 2));
        };
        auto issueW = [&](int t, int slot) {
            LAS char* d = lds + WBASE + slot * WSLOT + lw * 16 * 256;
#pragma unroll
            for (int i = 0; i < 4; ++i) dma16<WAUX>(rw, d + i * 1024, (i & 2) ? wv1 : wv0, (unsigned)((t * 64 + i * 4) * NG * 2));
        };
        unsigned touch = 0;
        const unsigned tv = b * 16384 + (lw >> 1) * 8192 + ((lw & 1) * 8 + (lane & 7)) * 128;
        auto touchX = [&](unsigned kt0) { asm volatile("buffer_load_dword %0, %1, %2, %3 offen" : "+v"(touch) : "v"(tv), "s"(rx), "s"(kt0 * 128u) : "memory"); };
        issueW(0, 0); issueX(0, 0); issueW(1, 1); issueX(1, 1);
        WAIT_V(8);
        BARRIER();
        issueW(2, 2); touchX(TK0); issueX(2, 2); issueW(3, 3);
        WAIT_V(13); BARRIER();
        issueX(3, 0); issueW(4, 0); touchX(TK0 + 16);
        WAIT_V(13); BARRIER();
        issueX(4, 1); issueW(5, 1); issueW(6, 2);
        WAIT_V(16); BARRIER();
        int xs = 2, ws = 3;
        for (int t = 3; t < NT - 4; ++t) {
            issueX(t + 2, xs); issueW(t + 4, ws);
            if (t == 4) touchX(TK0 + 32);
            if (t == 6) touchX(48);
            xs = (xs == NXS - 1) ? 0 : xs + 1; ws = (ws == NWS - 1) ? 0 : ws + 1;
            WAIT_V(16);
            BARRIER();
        }
        issueX(NT - 2, xs); xs = (xs == NXS - 1) ? 0 : xs + 1;
        WAIT_V(12); BARRIER();
        issueX(NT - 1, xs);
        char* const rs = myws + WS_RS + (size_t)(b * 4 + w4) * STRM;
        asm volatile("global_load_dword %0, %1, off sc1" : "+v"(touch) : "v"(myws + WS_CTR + b * 64) : "memory");
        WAIT_V(9); BARRIER();
        WAIT_V(0);
        asm volatile("" : "+v"(touch));
        const unsigned call = __builtin_amdgcn_readfirstlane(touch) + 1u;
        st16_sys(rs + (call & 7u) * 1024 + lane * 16, (u32x4){0u, 0u, 0u, 0u});
        BARRIER();
        BARRIER();
        if (wid == 4) asm volatile("ds_write_b128 %0, %1\n\ts_waitcnt lgkmcnt(0)" :: "v"((unsigned)EP_SCALE), "v"((u32x4){0u, 0u, 0u, 0u}) : "memory");
        WAIT_V(0);
        BARRIER();
        const unsigned tag = stream_tag(call);
        const u64 t0 = __builtin_amdgcn_s_memrealtime();
        if (local) {
            if (idx < NAGG) {
                bool dead = false, ok = false; unsigned m = 0;
                const char* gp = myws + WS_LG + w4 * 2048 + lane * 16;
                for (;;) {
                    if (!ok) {
                        u32x4 v0, v1;
                        asm volatile("global_load_dwordx4 %0, %2, off sc1\n\tglobal_load_dwordx4 %1, %2, off offset:1024 sc1\n\ts_waitcnt vmcnt(0)" : "=&v"(v0), "=&v"(v1) : "v"(gp) : "memory");
                        m = max(max(v0[0], v0[2]), max(v1[0], v1[2]));
                        ok = v0[1] == call && v0[3] == call && v1[1] == call && v1[3] == call;
                    }
                    if (__all(ok)) break;
                    if (__builtin_amdgcn_s_memrealtime() - t0 > DEADLINE) { dead = true; break; }
                    __builtin_amdgcn_s_sleep(ASLEEP);
                }
                if (!dead) {
                    m = wave_max_u(m);
                    unsigned cnt = 0;
                    if (lane == 0) {
                        __hip_atomic_fetch_max((LAS unsigned*)(lds + EP_SCALE + 8), m, RLX, __HIP_MEMORY_SCOPE_WORKGROUP);
                        cnt = __hip_atomic_fetch_add((LAS unsigned*)(lds + EP_SCALE + 12), 1u, RLX, __HIP_MEMORY_SCOPE_WORKGROUP);
                    }
                    if (__builtin_amdgcn_readfirstlane(cnt) == 3) {
                        m = __hip_atomic_load((LAS unsigned*)(lds + EP_SCALE + 8), RLX, __HIP_MEMORY_SCOPE_WORKGROUP);
                        char* base = p.ws[0];
#pragma unroll
                        for (int r = 1; r < NR; ++r) base = (lane & 7) == r ? p.ws[r] : base;
                        base += WS_GG + ((call & 1) * NREP + (lane >> 3)) * 128 + p.rank * 8;
#pragma unroll
                        for (int i = 0; i < NREP / 8; ++i) __hip_atomic_store((u64*)(base + i * 1024), ((u64)call << 32) | m, RLX, SC_SYS);
                        STAMP(2);
                    }
                } else if (lane == 0) __hip_atomic_store(tmo, call, RLX, SC_AGENT);
            }
        } else {
            bool dead = false;
            {
                const unsigned nib = lane & 15; unsigned pos = lane >> 4, sel = 0;
#pragma unroll
                for (int e = 0; e < 4; ++e) { const bool k = (nib >> e) & 1; sel |= (k ? pos : 0x0cu) << (8 * e); pos += k; }
                *(LAS unsigned*)(lds + EP_LUT + wid * 512 + lane * 4) = sel;
            }
            unsigned n = 0;
            while (__hip_atomic_load((unsigned*)(myws + WS_MF + (b * 4 + w4) * 16), RLX, SC_SYS) < call) {
                if (((++n & 63u) == 0u && __hip_atomic_load(tmo, RLX, SC_AGENT) == call) || __builtin_amdgcn_s_memrealtime() - t0 > DEADLINE) { dead = true; break; }
                __builtin_amdgcn_s_sleep(32);
            }
            const u32x4 keep = ld16_sys(myws + WS_MR + (size_t)(((call & 1) * NWG + b) * 4 + w4) * 1024 + lane * 16);
            unsigned start[4], cum[4];
            stream_offsets(keep, start, cum);
            LAS char* const lb = lds + EP_RCV + w4 * RCVB;
            *(LAS u32x4*)(lb + lane * 32) = keep;
            *(LAS u32x4*)(lb + lane * 32 + 16) = (u32x4){start[0], start[1], start[2], start[3]};
            unsigned addr[32], sel[32];
#pragma unroll
            for (int gg = 0; gg < 4; ++gg) {
                const u32x4 kp = *(LAS u32x4*)(lb + ((lane >> 2) + 16 * gg) * 32), stt = *(LAS u32x4*)(lb + ((lane >> 2) + 16 * gg) * 32 + 16);
#pragma unroll
                for (int i = 0; i < 8; ++i) {
                    const unsigned sh = (i & 1) * 16 + (lane & 3) * 4;
                    const unsigned off = stt[i >> 1] + __builtin_popcount(kp[i >> 1] & ((1u << sh) - 1u)), nib = (kp[i >> 1] >> sh) & 15u;
                    sel[i * 4 + gg] = *(LAS unsigned*)(lds + EP_LUT + wid * 512 + (((off & 3u) << 4) | nib) * 4);
                    addr[i * 4 + gg] = EP_RCV + w4 * RCVB + (off & ~3u);
                }
            }
#pragma unroll
            for (int i = 0; i < 32; ++i) asm volatile("" : "+v"(addr[i]), "+v"(sel[i]));
            if (wid == 4) STAMP(6);
            unsigned st;
            while ((st = __builtin_amdgcn_readfirstlane(__hip_atomic_load((LAS unsigned*)(lds + EP_SCALE + 4), RLX, __HIP_MEMORY_SCOPE_WORKGROUP))) == 0u) __builtin_amdgcn_s_sleep(8);
            const unsigned total = cum[3];
            const int sidx = (idx - 4 - 8 * w4) & 31;
            {
                const unsigned dt = ((unsigned)slot_ticks(sidx, total) + RWAIT) << 2;
                while (((unsigned)__builtin_amdgcn_s_memrealtime() << 2) - (st & ~3u) < dt) __builtin_amdgcn_s_sleep(2);
            }
            char* o = p.out + (size_t)(peer * MS + m0 + wm * 128 + (lane >> 2)) * NS + col_block(sidx >> 1) * BN + wn * 64 + (lane & 3) * 16;
            const __amdgpu_buffer_rsrc_t rr = __builtin_amdgcn_make_buffer_rsrc((void*)rs, 0, (total + 15u) & ~15u, 0x00020000);
            unsigned pos = 0;
            while (pos < total && !dead) {
                u32x4 v[5];
#pragma unroll
                for (int i = 0; i < 5; ++i) v[i] = __builtin_amdgcn_raw_buffer_load_b128(rr, lane * 16 + i * 1024, pos, 17);
                unsigned adv = 5120;
#pragma unroll
                for (int i = 4; i >= 0; --i) {
                    const bool ok = pos + i * 1024 + lane * 16 >= total || ((((v[i][0] ^ tag) | (v[i][1] ^ tag) | (v[i][2] ^ tag) | (v[i][3] ^ tag)) & 0x80808080u) == 0u);
                    const u64 bad = ~__ballot(ok);
                    *(LAS u32x4*)(lb + pos + i * 1024 + lane * 16) = v[i] & 0x7f7f7f7fu;
                    if (bad) adv = i * 1024 + (unsigned)__builtin_ctzll(bad) * 16u;
                }
                pos += adv & ~127u;
                if (pos < total) {
                    if (((++n & 63u) == 0u && __hip_atomic_load(tmo, RLX, SC_AGENT) == call) || __builtin_amdgcn_s_memrealtime() - t0 > DEADLINE) dead = true;
                    __builtin_amdgcn_s_sleep(RSLEEP);
                }
            }
            if (wid == 4) STAMP(7);
            unsigned d[32];
#pragma unroll
            for (int i = 0; i < 32; ++i) {
                const LAS unsigned* sp = (const LAS unsigned*)(lds + addr[i]);
                d[i] = __builtin_amdgcn_perm(sp[1], sp[0], sel[i]);
            }
#pragma unroll
            for (int i = 0; i < 8; ++i) {
                u32x4 v = {d[i * 4], d[i * 4 + 1], d[i * 4 + 2], d[i * 4 + 3]};
                if (dead) v = (u32x4){0x80808080u, 0x80808080u, 0x80808080u, 0x80808080u};
                st16_out(o + (size_t)i * 16 * NS, v);
            }
            if (dead && lane == 0) __hip_atomic_store(tmo, call, RLX, SC_AGENT);
            if (wid == 4) STAMP(5);
        }
    } else {
        if (wid == 0) STAMP(0);
        const unsigned xa0 = XBASE + wm * 16384 + fr * 128 + ((g ^ (fr >> 1)) << 4);
        const int qq = fr >> 2, pp = lane & 3;
        const int f = qq | ((g & 1) << 2);
        const unsigned wa0 = WBASE + (8 * g + qq) * 256 + ((((wn * 8) + (pp >> 1)) ^ (f << 1)) << 4) + (pp & 1) * 8;
        const unsigned c0 = __hip_atomic_load((unsigned*)(myws + WS_CTR + b * 64), RLX, SC_AGENT);
        f32x4 acc[8][4];
#pragma unroll
        for (int i = 0; i < 8; ++i)
#pragma unroll
            for (int j = 0; j < 4; ++j) acc[i][j] = (f32x4){0.f, 0.f, 0.f, 0.f};
        bf16x8 XP[2][2], XQ[2][2], WA[4][2], WB[4][2];
        auto ldX = [&](bf16x8 (&d)[2][2], unsigned base, int mf0) {
#pragma unroll
            for (int i = 0; i < 2; ++i)
#pragma unroll
                for (int ks = 0; ks < 2; ++ks) d[i][ks] = *(LAS bf16x8*)(lds + (base ^ (ks * 64)) + (mf0 + i) * 2048);
        };
        auto ldW = [&](unsigned base, int j, int ks) {
            const unsigned a = base ^ (j << 5);
            s16x4 lo = __builtin_amdgcn_ds_read_tr16_b64_v4i16((LAS s16x4*)(lds + a + ks * 8192));
            s16x4 hi = __builtin_amdgcn_ds_read_tr16_b64_v4i16((LAS s16x4*)(lds + a + ks * 8192 + 1024));
            return __builtin_bit_cast(bf16x8, __builtin_shufflevector(lo, hi, 0, 1, 2, 3, 4, 5, 6, 7));
        };
        auto mma = [&](int mf0, bf16x8 (&X)[2][2], bf16x8 (&W)[4][2]) {
#pragma unroll
            for (int i = 0; i < 2; ++i)
#pragma unroll
                for (int j = 0; j < 4; ++j)
#pragma unroll
                    for (int ks = 0; ks < 2; ++ks)
                        acc[mf0 + i][j] = __builtin_amdgcn_mfma_f32_16x16x32_bf16(W[j][ks], X[i][ks], acc[mf0 + i][j], 0, 0, 0);
        };
#define SGB(mask, n) __builtin_amdgcn_sched_group_barrier(mask, n, 0)
#define SCHED_N(n) do { _Pragma("unroll") for (int _i = 0; _i < (n); ++_i) { SGB(0x8, 1); SGB(0x100, 1); } _Pragma("unroll") for (int _i = 0; _i < 16 - (n); ++_i) SGB(0x8, 1); __builtin_amdgcn_sched_barrier(0); } while (0)
        const unsigned call = __builtin_amdgcn_readfirstlane(c0) + 1u;
        __builtin_amdgcn_s_waitcnt(0xC07F);
        BARRIER();
#pragma unroll
        for (int j = 0; j < 4; ++j) { WA[j][0] = ldW(wa0, j, 0); WA[j][1] = ldW(wa0, j, 1); }
        ldX(XP, xa0, 0);
        __builtin_amdgcn_sched_barrier(0);
        int xs = 0, ws = 0;
        auto tile = [&](bf16x8 (&Wc)[4][2], bf16x8 (&Wn)[4][2]) {
            const unsigned xb = xa0 + xs * XSLOT;
            xs = (xs == NXS - 1) ? 0 : xs + 1; ws = (ws == NWS - 1) ? 0 : ws + 1;
            const unsigned xbn = xa0 + xs * XSLOT, wbn = wa0 + ws * WSLOT;
            ldX(XQ, xb, 2); Wn[0][0] = ldW(wbn, 0, 0); Wn[0][1] = ldW(wbn, 0, 1); Wn[1][0] = ldW(wbn, 1, 0); mma(0, XP, Wc); SCHED_N(10);
            ldX(XP, xb, 4); Wn[1][1] = ldW(wbn, 1, 1); Wn[2][0] = ldW(wbn, 2, 0); Wn[2][1] = ldW(wbn, 2, 1); mma(2, XQ, Wc); SCHED_N(10);
            ldX(XQ, xb, 6); Wn[3][0] = ldW(wbn, 3, 0); Wn[3][1] = ldW(wbn, 3, 1); mma(4, XP, Wc); SCHED_N(8);
            __builtin_amdgcn_s_waitcnt(0xC07F);
            BARRIER();
            __builtin_amdgcn_sched_barrier(0);
            ldX(XP, xbn, 0); mma(6, XQ, Wc); SCHED_N(4);
        };
        for (int t = 0; t < NT; t += 2) { tile(WA, WB); tile(WB, WA); }
        __builtin_amdgcn_s_waitcnt(0xC07F);
        BARRIER();
        float mx = 0.f;
#pragma unroll
        for (int i = 0; i < 8; ++i)
#pragma unroll
            for (int j = 0; j < 4; ++j)
                mx = __builtin_fmaxf(__builtin_fmaxf(mx, __builtin_fmaxf(acc[i][j][0], acc[i][j][1])), __builtin_fmaxf(acc[i][j][2], acc[i][j][3]));
        const unsigned wmax = wave_max_u(__float_as_uint(mx));
        if (lane == 0) {
            __hip_atomic_store((u64*)(myws + WS_LG) + b * 4 + wid, ((u64)call << 32) | wmax, RLX, SC_AGENT);
            if (wid == 0) __hip_atomic_store((unsigned*)(myws + WS_CTR + b * 64), call, RLX, SC_AGENT);
        }
        if (wid == 0) STAMP(1);
        const u64 t0 = __builtin_amdgcn_s_memrealtime();
        auto wait_scale = [&](bool& dead) {
            LAS u64* sw = (LAS u64*)(lds + EP_SCALE);
            if (wid == 0) {
                unsigned gm, n = 0;
                for (;;) {
                    const u64 v = __hip_atomic_load((u64*)(myws + WS_GG + ((call & 1) * NREP + (idx & (NREP - 1))) * 128) + (lane & 7), RLX, SC_SYS);
                    gm = (unsigned)v;
                    if (__all((unsigned)(v >> 32) == call)) break;
                    if (((++n & 63u) == 0u && __hip_atomic_load(tmo, RLX, SC_AGENT) == call) || __builtin_amdgcn_s_memrealtime() - t0 > DEADLINE) { dead = true; break; }
                    __builtin_amdgcn_s_sleep(GSLEEP);
                }
                if (dead && lane == 0) __hip_atomic_store(tmo, call, RLX, SC_AGENT);
                const float am = __uint_as_float(wave_max_u(gm));
                const float inv = am > 0.f ? 127.0f / am : 0.f;
                __hip_atomic_store(sw, ((u64)(((unsigned)__builtin_amdgcn_s_memrealtime() << 2) | (dead ? 2u : 1u)) << 32) | __float_as_uint(inv), RLX, __HIP_MEMORY_SCOPE_WORKGROUP);
                return inv;
            }
            u64 v;
            while (((v = __hip_atomic_load(sw, RLX, __HIP_MEMORY_SCOPE_WORKGROUP)) >> 32) == 0u) __builtin_amdgcn_s_sleep(1);
            dead = ((v >> 32) & 3u) == 2u;
            return __uint_as_float((unsigned)v);
        };
        bool dead = false;
        if (local) {
            const float inv = wait_scale(dead);
            if (wid == 0) STAMP(3);
            char* o = p.out + (size_t)(p.rank * MS + m0 + wm * 128 + (lane >> 2)) * NS + cbl * BN + wn * 64 + (lane & 3) * 16;
            LAS char* const lb = lds + EP_STRM + wid * STRM;
#pragma unroll
            for (int i = 0; i < 8; ++i)
#pragma unroll
                for (int j = 0; j < 4; ++j) *(LAS unsigned*)(lb + i * 1024 + fr * 64 + ((j ^ (fr >> 2)) & 3) * 16 + g * 4) = q4(acc[i][j], inv);
            u32x4 v[8];
#pragma unroll
            for (int i = 0; i < 8; ++i) v[i] = *(LAS u32x4*)(lb + i * 1024 + (lane >> 2) * 64 + ((lane ^ (lane >> 4)) & 3) * 16);
#pragma unroll
            for (int i = 0; i < 8; ++i) st16_out(o + (size_t)i * 16 * NS, dead ? (u32x4){0x80808080u, 0x80808080u, 0x80808080u, 0x80808080u} : v[i]);
        } else {
            const int slot = (((idx + 4 + 8 * wid) & 31) * 8 + p.rank) * 4 + wid;
            char* const pws = p.ws[peer];
            const unsigned tag = stream_tag(call);
            u32x4 keep;
#pragma unroll
            for (int k = 0; k < 4; ++k) {
                unsigned m = 0;
#pragma unroll
                for (int u = 7; u >= 0; --u)
#pragma unroll
                    for (int e = 3; e >= 0; --e) m = __builtin_amdgcn_alignbit(m, __float_as_uint(acc[2 * k + (u >> 2)][u & 3][e]), 31);
                keep[k] = ~m;
            }
            {
                const unsigned nib = lane & 15; unsigned pos = lane >> 4; u64 sel = 0x0c0c0c0c0c0c0c0cull;
#pragma unroll
                for (int e = 0; e < 4; ++e) { const bool k = (nib >> e) & 1; const u64 x = (u64)(0x0cu ^ (unsigned)e) << (8 * pos); sel ^= k ? x : 0ull; pos += k; }
                *(LAS u64*)(lds + EP_LUT + wid * 512 + lane * 8) = sel;
            }
#pragma unroll
            for (int i = 0; i < 9; ++i) *(LAS u32x4*)(lds + EP_STRM + wid * STRM + i * 1024 + lane * 16) = (u32x4){tag, tag, tag, tag};
            unsigned start[4], cum[4];
            stream_offsets(keep, start, cum);
            unsigned addr[32]; u32x2 sel[32];
#pragma unroll
            for (int k = 0; k < 4; ++k) {
                unsigned off = start[k];
#pragma unroll
                for (int u = 0; u < 8; ++u) {
                    const unsigned nib = (keep[k] >> (4 * u)) & 15u;
                    sel[k * 8 + u] = *(LAS u32x2*)(lds + EP_LUT + wid * 512 + (((off & 3u) << 4) | nib) * 8);
                    addr[k * 8 + u] = EP_STRM + wid * STRM + (off & ~3u);
                    off += __builtin_popcount(nib);
                }
            }
#pragma unroll
            for (int i = 0; i < 32; ++i) asm volatile("" : "+v"(addr[i]), "+v"(sel[i]));
            if (wid == 0) STAMP(8);
            st16_sys(pws + WS_MR + (size_t)((call & 1) * NWG * 4 + slot) * 1024 + lane * 16, keep);
            WAIT_V(0);
            if (lane == 0) __hip_atomic_store((unsigned*)(pws + WS_MF + slot * 16), call, RLX, SC_SYS);
            const float inv = wait_scale(dead);
            if (wid == 0) STAMP(3);
            if (!dead) {
                char* const rs = pws + WS_RS + (size_t)slot * STRM + lane * 16;
                const u64 ta = __builtin_amdgcn_s_memrealtime();
                const u64 dt = slot_ticks(idx, cum[3]);
                unsigned sent = 0;
#pragma unroll
                for (int k = 0; k < 4; ++k) {
#pragma unroll
                    for (int u = 0; u < 8; ++u) {
                        const unsigned qd = q4(acc[2 * k + (u >> 2)][u & 3], inv) & 0x7f7f7f7fu;
                        LAS unsigned* d = (LAS unsigned*)(lds + addr[k * 8 + u]);
                        __hip_atomic_fetch_or(d, __builtin_amdgcn_perm(0u, qd, sel[k * 8 + u][0]), RLX, __HIP_MEMORY_SCOPE_WORKGROUP);
                        __hip_atomic_fetch_or(d + 1, __builtin_amdgcn_perm(0u, qd, sel[k * 8 + u][1]), RLX, __HIP_MEMORY_SCOPE_WORKGROUP);
                    }
                    asm volatile("" ::: "memory");
                    const unsigned lim = k < 3 ? (cum[k] & ~127u) : ((cum[3] + 63u) & ~63u);
                    if (k == 3) while (__builtin_amdgcn_s_memrealtime() - ta < dt) __builtin_amdgcn_s_sleep(1);
                    if (k == 0 || k == 3 || __builtin_amdgcn_s_memrealtime() - ta >= dt) {
                        while (sent < lim) {
                            if (sent + lane * 16 < lim) st16_sys(rs + sent, *(LAS u32x4*)(lds + EP_STRM + wid * STRM + sent + lane * 16));
                            sent = min(sent + 1024u, lim);
                        }
                    }
                }
                if (wid == 0) STAMP(9);
            }
        }
    }
}

extern "C" void kernel_launch(int rank, int nranks, ncclComm_t comm, hipStream_t stream,
                              void* const* d_in, const int* in_sizes, int n_in,
                              void* d_out, int out_size, const int* in_dtypes, int out_dtype,
                              void* const* d_out_all, void* const* d_ws_all, size_t ws_bytes) {
    if (nranks != NR || (unsigned)rank >= (unsigned)NR || ws_bytes < WS_END) return;
    static bool init[NR];
    if (!init[rank]) {
        HIP_CHECK(hipFuncSetAttribute((const void*)fused_kernel, hipFuncAttributeMaxDynamicSharedMemorySize, LDS_BYTES));
        init[rank] = true;
    }
    Params p;
    p.x = (const u16*)d_in[0]; p.w = (const u16*)d_in[1]; p.out = (char*)d_out;
    for (int r = 0; r < NR; ++r) p.ws[r] = (char*)d_ws_all[r];
    p.rank = rank; p.pad = 0; p.myws = p.ws[rank];
    fused_kernel<<<NWG, 512, LDS_BYTES, stream>>>(p);
    HIP_CHECK(hipGetLastError());
}
```

```cpp
#include <hip/hip_runtime.h>
#include <rccl/rccl.h>
#include <cstdio>

#define HIP_CHECK(expr) do { hipError_t _e = (expr); if (_e != hipSuccess) { \
    fprintf(stderr, "HIP error %s at %s:%d\n", hipGetErrorString(_e), __FILE__, __LINE__); \
    return; } } while (0)

#define LAS __attribute__((address_space(3)))
typedef short s16x4 __attribute__((ext_vector_type(4)));
typedef __bf16 bf16x8 __attribute__((ext_vector_type(8)));
typedef float f32x4 __attribute__((ext_vector_type(4)));
typedef unsigned u32x4 __attribute__((ext_vector_type(4)));
typedef unsigned u32x2 __attribute__((ext_vector_type(2)));
typedef unsigned long long u64;
typedef unsigned short u16;

constexpr int NR = 8;
constexpr int MS = 512, NG = 16384, KD = 4096;
constexpr int NS = NG / NR;
constexpr int BM = 256, BN = 128, BK = 64, NT = KD / BK;
constexpr int NXS = 3, NWS = 4;
constexpr int XSLOT = BM * BK * 2, WSLOT = BK * BN * 2;
constexpr int XBASE = 0, WBASE = NXS * XSLOT;
constexpr int LDS_BYTES = NXS * XSLOT + NWS * WSLOT;
constexpr int NWG = 256;
constexpr int STRM = 8192 + 1024;
constexpr int EP_STRM = 0;
constexpr int RCVB = 8192 + 5120;
constexpr int EP_RCV = 4 * STRM;
constexpr int EP_LUT = EP_RCV + 4 * RCVB;
constexpr int EP_SCALE = EP_LUT + 4096;

constexpr size_t WS_CTR = 0;
constexpr size_t WS_TMO = 16384;
constexpr size_t WS_LG = 20480;
#define NREPL 16
constexpr int NREP = NREPL;
constexpr size_t WS_GG = 32768;
constexpr size_t WS_MF = 49152;
constexpr size_t WS_DIAG = 65536;
constexpr size_t WS_MR = 1u << 20;
constexpr size_t WS_RS = 3u << 20;
constexpr size_t WS_END = WS_RS + (size_t)NWG * 4 * STRM;
constexpr u64 DEADLINE = 50000000ull;
#define WAUX 2

#define PACE 1
#define OUTWT 1
#define NAGG 2
#define ASLEEP 1
#define PROG 0
#define PLACE 0
#define TK0 3
#define SAUX 0
#define SLOWALL 0
#define MEARLY 1
#define RSLEEP 1
#define LINK_BPT 590u
#define SLOT0 6
#define RWAIT 60
#define GSLEEP 1

struct Params {
    const u16* x; const u16* w;
    char* out; char* myws;
    int rank; int pad;
    char* ws[NR];
};

#define WAIT_V(n) asm volatile("s_waitcnt vmcnt(" #n ")" ::: "memory")
#define BARRIER() asm volatile("s_barrier" ::: "memory")
#define RLX __ATOMIC_RELAXED
#define SC_AGENT __HIP_MEMORY_SCOPE_AGENT
#define SC_SYS __HIP_MEMORY_SCOPE_SYSTEM

template <int AUX> __device__ __forceinline__ void dma16(__amdgpu_buffer_rsrc_t rsrc, LAS char* dst, unsigned voff, unsigned soff) {
    __builtin_amdgcn_raw_ptr_buffer_load_lds(rsrc, (LAS void*)dst, 16, voff, soff, 0, AUX);
}
__device__ __forceinline__ void st16_sys(char* p, u32x4 v) {
    asm volatile("global_store_dwordx4 %0, %1, off sc0 sc1\n\ts_nop 1" :: "v"(p), "v"(v) : "memory");
}
__device__ __forceinline__ void st16_out(char* p, u32x4 v) {
    asm volatile("global_store_dwordx4 %0, %1, off sc1\n\ts_nop 1" :: "v"(p), "v"(v) : "memory");
}
__device__ __forceinline__ u32x4 ld16_sys(const char* p) {
    u32x4 v; asm volatile("global_load_dwordx4 %0, %1, off sc0 sc1\n\ts_waitcnt vmcnt(0)" : "=&v"(v) : "v"(p) : "memory"); return v;
}
__device__ __forceinline__ unsigned q4(f32x4 v, float inv) {
    unsigned r = 0;
    r = __builtin_amdgcn_cvt_pk_u8_f32(v[0] * inv, 0, r);
    r = __builtin_amdgcn_cvt_pk_u8_f32(v[1] * inv, 1, r);
    r = __builtin_amdgcn_cvt_pk_u8_f32(v[2] * inv, 2, r);
    r = __builtin_amdgcn_cvt_pk_u8_f32(v[3] * inv, 3, r);
    return r;
}
template <int CTRL, int RMASK> __device__ __forceinline__ unsigned dpp0(unsigned x) { return (unsigned)__builtin_amdgcn_update_dpp(0, (int)x, CTRL, RMASK, 0xf, true); }
__device__ __forceinline__ unsigned wave_scan_add(unsigned x) {
    x += dpp0<0x111, 0xf>(x); x += dpp0<0x112, 0xf>(x); x += dpp0<0x114, 0xf>(x); x += dpp0<0x118, 0xf>(x);
    x += dpp0<0x142, 0xa>(x); x += dpp0<0x143, 0xc>(x); return x;
}
__device__ __forceinline__ unsigned wave_max_u(unsigned x) {
    x = max(x, dpp0<0x111, 0xf>(x)); x = max(x, dpp0<0x112, 0xf>(x)); x = max(x, dpp0<0x114, 0xf>(x)); x = max(x, dpp0<0x118, 0xf>(x));
    x = max(x, dpp0<0x142, 0xa>(x)); x = max(x, dpp0<0x143, 0xc>(x)); return (unsigned)__builtin_amdgcn_readlane((int)x, 63);
}
__device__ __forceinline__ void stream_offsets(const u32x4 keep, unsigned (&start)[4], unsigned (&cum)[4]) {
    const unsigned c0 = __builtin_popcount(keep[0]), c1 = __builtin_popcount(keep[1]), c2 = __builtin_popcount(keep[2]), c3 = __builtin_popcount(keep[3]);
    const unsigned a = wave_scan_add(c0 | (c1 << 16)), b = wave_scan_add(c2 | (c3 << 16));
    const unsigned ta = (unsigned)__builtin_amdgcn_readlane((int)a, 63), tb = (unsigned)__builtin_amdgcn_readlane((int)b, 63);
    cum[0] = ta & 0xffffu; cum[1] = cum[0] + (ta >> 16); cum[2] = cum[1] + (tb & 0xffffu); cum[3] = cum[2] + (tb >> 16);
    start[0] = (a & 0xffffu) - c0; start[1] = cum[0] + (a >> 16) - c1; start[2] = cum[1] + (b & 0xffffu) - c2; start[3] = cum[2] + (b >> 16) - c3;
}
__device__ __forceinline__ unsigned stream_tag(unsigned call) {
    const unsigned n = call % 15u + 1u;
    return ((n & 1u) << 7) | ((n & 2u) << 14) | ((n & 4u) << 21) | ((n & 8u) << 28);
}
__device__ __forceinline__ u64 slot_ticks(int pos, unsigned bytes) {
    return (u64)(((unsigned)(32 - SLOT0 + 3 * pos) * bytes) / LINK_BPT);
}
__device__ __forceinline__ u32x4 xpose4(unsigned d0, unsigned d1, unsigned d2, unsigned d3) {
    auto a = __builtin_amdgcn_permlane32_swap(d0, d2, false, false); auto b = __builtin_amdgcn_permlane32_swap(d1, d3, false, false);
    auto c = __builtin_amdgcn_permlane16_swap(a[0], b[0], false, false); auto d = __builtin_amdgcn_permlane16_swap(a[1], b[1], false, false);
    return (u32x4){c[0], c[1], d[0], d[1]};
}
#define STAMP(i) do { } while (0)

__global__ __launch_bounds__(512, 2) void fused_kernel(const Params p) {
    extern __shared__ __attribute__((aligned(16))) char smem[];
    LAS char* const lds = (LAS char*)0;
    asm volatile("" :: "s"((LAS char*)smem));
    asm volatile("" :: "s"(p.x), "s"(p.w), "s"(p.myws), "s"(p.rank));
    const int tid = threadIdx.x, lane = tid & 63;
    const int wid = __builtin_amdgcn_readfirstlane(tid >> 6);
    const int peer = blockIdx.x & 7, idx = blockIdx.x >> 3;
    const int b = idx * 8 + peer;
    auto col_block = [](int i) { return (int)((0xd951fecba8764320ull >> (4 * i)) & 15u); };
    const int cbl = col_block(idx >> 1), mh = idx & 1;
    const int m0 = mh * BM, n0 = (peer * 16 + cbl) * BN;
    const bool local = peer == p.rank;
    char* const myws = p.myws;
    unsigned* const tmo = (unsigned*)(myws + WS_TMO);
    const int fr = lane & 15, g = lane >> 4;
    const int w4 = wid & 3, wm = w4 >> 1, wn = w4 & 1;

    if (__builtin_expect(wid >= 4, 1)) {
        const int lw = w4;
        __amdgpu_buffer_rsrc_t rx = __builtin_amdgcn_make_buffer_rsrc((void*)p.x, 0, MS * KD * 2, 0x00020000);
        __amdgpu_buffer_rsrc_t rw = __builtin_amdgcn_make_buffer_rsrc((void*)p.w, 0, (unsigned)((size_t)KD * NG * 2), 0x00020000);
        const unsigned xrow = m0 + lw * 64 + (lane >> 3);
        const unsigned xv0 = xrow * (KD * 2) + (((lane & 7) ^ (lane >> 4)) << 4);
        const unsigned xv1 = xrow * (KD * 2) + (((lane & 7) ^ (4 + (lane >> 4))) << 4);
        const unsigned wv = (lw * 4 + (lane >> 4)) * (NG * 2) + n0 * 2 + (((lane & 15) ^ (((lane >> 4) | ((lw & 2) << 1)) << 1)) << 4);
        auto issueX = [&](int t, int slot) {
            LAS char* d = lds + XBASE + slot * XSLOT + lw * 64 * 128;
#pragma unroll
            for (int i = 0; i < 8; ++i) dma16<0>(rx, d + i * 1024, (i & 1) ? xv1 : xv0, (unsigned)(t * 128 + i * 8 * KD * 2));
        };
        auto issueW = [&](int t) {
#pragma unroll
            for (int i = 0; i < 4; ++i) dma16<WAUX>(rw, lds + WBASE + (t & 3) * WSLOT + i * 4096 + lw * 1024, wv, (unsigned)(t * 64 + i * 16) * (NG * 2));
        };
        auto issueWs = [&](int t) {
#pragma unroll
            for (int i = 0; i < 4; ++i) dma16<SAUX>(rw, lds + WBASE + (t & 3) * WSLOT + i * 4096 + lw * 1024, wv, (unsigned)(t * 64 + i * 16) * (NG * 2));
        };
        unsigned touch = 0;
        const unsigned tv = b * 16384 + (lw >> 1) * 8192 + ((lw & 1) * 8 + (lane & 7)) * 128;
        auto touchX = [&](unsigned kt0) { asm volatile("buffer_load_dword %0, %1, %2, %3 offen" : "+v"(touch) : "v"(tv), "s"(rx), "s"(kt0 * 128u) : "memory"); };
        int xs = 1;
        auto nextX = [&](int t) { issueX(t, xs); xs = (xs == NXS - 1) ? 0 : xs + 1; };
        issueW(0); issueX(0, 0); issueW(1); issueX(1, 1);
        WAIT_V(8);
        BARRIER();
        issueW(2); touchX(3); issueX(2, 2); issueW(3);
        WAIT_V(13); BARRIER();
        issueX(3, 0); issueW(4); touchX(19); touchX(35); touchX(48);
        WAIT_V(15); BARRIER();
        int t = 2;
        const bool slow = SLOWALL || idx >= 24 || (((unsigned)n0 * 2u + 255u) & 16383u) == 8191u;
        const bool shift = slow && mh;
        if (shift) { nextX(4); issueW(5); WAIT_V(12); BARRIER(); t = 3; }
        if (slow) {
            for (; t < NT - 4; t += 2) {
                nextX(t + 2); issueWs(t + 3); issueWs(t + 4);
                WAIT_V(16); BARRIER();
                nextX(t + 3);
                WAIT_V(12); BARRIER();
            }
        } else {
            for (; t < NT - 4; t += 2) {
                nextX(t + 2); issueW(t + 3); issueW(t + 4);
                WAIT_V(16); BARRIER();
                nextX(t + 3);
                WAIT_V(12); BARRIER();
            }
        }
        if (!shift) { nextX(NT - 2); issueW(NT - 1); WAIT_V(12); BARRIER(); }
        nextX(NT - 1);
        char* const rs = myws + WS_RS + (size_t)(b * 4 + w4) * STRM;
        asm volatile("global_load_dword %0, %1, off sc1" : "+v"(touch) : "v"(myws + WS_CTR + b * 64) : "memory");
        WAIT_V(9); BARRIER();
        WAIT_V(0);
        asm volatile("" : "+v"(touch));
        const unsigned call = __builtin_amdgcn_readfirstlane(touch) + 1u;
        st16_sys(rs + (call & 7u) * 1024 + lane * 16, (u32x4){0u, 0u, 0u, 0u});
        BARRIER();
        BARRIER();
        if (wid == 4) asm volatile("ds_write_b128 %0, %1\n\ts_waitcnt lgkmcnt(0)" :: "v"((unsigned)EP_SCALE), "v"((u32x4){0u, 0u, 0u, 0u}) : "memory");
        WAIT_V(0);
        BARRIER();
        const unsigned tag = stream_tag(call);
        const u64 t0 = __builtin_amdgcn_s_memrealtime();
        if (local) {
            if (idx < NAGG) {
                bool dead = false, ok = false; unsigned m = 0;
                const char* gp = myws + WS_LG + w4 * 2048 + lane * 16;
                for (;;) {
                    if (!ok) {
                        u32x4 v0, v1;
                        asm volatile("global_load_dwordx4 %0, %2, off sc1\n\tglobal_load_dwordx4 %1, %2, off offset:1024 sc1\n\ts_waitcnt vmcnt(0)" : "=&v"(v0), "=&v"(v1) : "v"(gp) : "memory");
                        m = max(max(v0[0], v0[2]), max(v1[0], v1[2]));
                        ok = v0[1] == call && v0[3] == call && v1[1] == call && v1[3] == call;
                    }
                    if (__all(ok)) break;
                    if (__builtin_amdgcn_s_memrealtime() - t0 > DEADLINE) { dead = true; break; }
                    __builtin_amdgcn_s_sleep(ASLEEP);
                }
                if (!dead) {
                    m = wave_max_u(m);
                    unsigned cnt = 0;
                    if (lane == 0) {
                        __hip_atomic_fetch_max((LAS unsigned*)(lds + EP_SCALE + 8), m, RLX, __HIP_MEMORY_SCOPE_WORKGROUP);
                        cnt = __hip_atomic_fetch_add((LAS unsigned*)(lds + EP_SCALE + 12), 1u, RLX, __HIP_MEMORY_SCOPE_WORKGROUP);
                    }
                    if (__builtin_amdgcn_readfirstlane(cnt) == 3) {
                        m = __hip_atomic_load((LAS unsigned*)(lds + EP_SCALE + 8), RLX, __HIP_MEMORY_SCOPE_WORKGROUP);
                        char* base = p.ws[0];
#pragma unroll
                        for (int r = 1; r < NR; ++r) base = (lane & 7) == r ? p.ws[r] : base;
                        base += WS_GG + ((call & 1) * NREP + (lane >> 3)) * 128 + p.rank * 8;
#pragma unroll
                        for (int i = 0; i < NREP / 8; ++i) __hip_atomic_store((u64*)(base + i * 1024), ((u64)call << 32) | m, RLX, SC_SYS);
                        STAMP(2);
                    }
                } else if (lane == 0) __hip_atomic_store(tmo, call, RLX, SC_AGENT);
            }
        } else {
            bool dead = false;
            {
                const unsigned nib = lane & 15; unsigned pos = lane >> 4, sel = 0;
#pragma unroll
                for (int e = 0; e < 4; ++e) { const bool k = (nib >> e) & 1; sel |= (k ? pos : 0x0cu) << (8 * e); pos += k; }
                *(LAS unsigned*)(lds + EP_LUT + wid * 512 + lane * 4) = sel;
            }
            unsigned n = 0;
            while (__hip_atomic_load((unsigned*)(myws + WS_MF + (b * 4 + w4) * 16), RLX, SC_SYS) < call) {
                if (((++n & 63u) == 0u && __hip_atomic_load(tmo, RLX, SC_AGENT) == call) || __builtin_amdgcn_s_memrealtime() - t0 > DEADLINE) { dead = true; break; }
                __builtin_amdgcn_s_sleep(32);
            }
            const u32x4 keep = ld16_sys(myws + WS_MR + (size_t)(((call & 1) * NWG + b) * 4 + w4) * 1024 + lane * 16);
            unsigned start[4], cum[4];
            stream_offsets(keep, start, cum);
            LAS char* const lb = lds + EP_RCV + w4 * RCVB;
            *(LAS u32x4*)(lb + lane * 32) = keep;
            *(LAS u32x4*)(lb + lane * 32 + 16) = (u32x4){start[0], start[1], start[2], start[3]};
            unsigned addr[32], sel[32];
#pragma unroll
            for (int gg = 0; gg < 4; ++gg) {
                const u32x4 kp = *(LAS u32x4*)(lb + ((lane >> 2) + 16 * gg) * 32), stt = *(LAS u32x4*)(lb + ((lane >> 2) + 16 * gg) * 32 + 16);
#pragma unroll
                for (int i = 0; i < 8; ++i) {
                    const unsigned sh = (i & 1) * 16 + (lane & 3) * 4;
                    const unsigned off = stt[i >> 1] + __builtin_popcount(kp[i >> 1] & ((1u << sh) - 1u)), nib = (kp[i >> 1] >> sh) & 15u;
                    sel[i * 4 + gg] = *(LAS unsigned*)(lds + EP_LUT + wid * 512 + (((off & 3u) << 4) | nib) * 4);
                    addr[i * 4 + gg] = EP_RCV + w4 * RCVB + (off & ~3u);
                }
            }
#pragma unroll
            for (int i = 0; i < 32; ++i) asm volatile("" : "+v"(addr[i]), "+v"(sel[i]));
            if (wid == 4) STAMP(6);
            unsigned st;
            while ((st = __builtin_amdgcn_readfirstlane(__hip_atomic_load((LAS unsigned*)(lds + EP_SCALE + 4), RLX, __HIP_MEMORY_SCOPE_WORKGROUP))) == 0u) __builtin_amdgcn_s_sleep(8);
            const unsigned total = cum[3];
            const int sidx = (idx - 4 - 8 * w4) & 31;
            {
                const unsigned dt = ((unsigned)slot_ticks(sidx, total) + RWAIT) << 2;
                while (((unsigned)__builtin_amdgcn_s_memrealtime() << 2) - (st & ~3u) < dt) __builtin_amdgcn_s_sleep(2);
            }
            char* o = p.out + (size_t)(peer * MS + m0 + wm * 128 + (lane >> 2)) * NS + col_block(sidx >> 1) * BN + wn * 64 + (lane & 3) * 16;
            const __amdgpu_buffer_rsrc_t rr = __builtin_amdgcn_make_buffer_rsrc((void*)rs, 0, (total + 15u) & ~15u, 0x00020000);
            unsigned pos = 0;
            while (pos < total && !dead) {
                u32x4 v[5];
#pragma unroll
                for (int i = 0; i < 5; ++i) v[i] = __builtin_amdgcn_raw_buffer_load_b128(rr, lane * 16 + i * 1024, pos, 17);
                unsigned adv = 5120;
#pragma unroll
                for (int i = 4; i >= 0; --i) {
                    const bool ok = pos + i * 1024 + lane * 16 >= total || ((((v[i][0] ^ tag) | (v[i][1] ^ tag) | (v[i][2] ^ tag) | (v[i][3] ^ tag)) & 0x80808080u) == 0u);
                    const u64 bad = ~__ballot(ok);
                    *(LAS u32x4*)(lb + pos + i * 1024 + lane * 16) = v[i] & 0x7f7f7f7fu;
                    if (bad) adv = i * 1024 + (unsigned)__builtin_ctzll(bad) * 16u;
                }
                pos += adv & ~127u;
                if (pos < total) {
                    if (((++n & 63u) == 0u && __hip_atomic_load(tmo, RLX, SC_AGENT) == call) || __builtin_amdgcn_s_memrealtime() - t0 > DEADLINE) dead = true;
                    __builtin_amdgcn_s_sleep(RSLEEP);
                }
            }
            if (wid == 4) STAMP(7);
            unsigned d[32];
#pragma unroll
            for (int i = 0; i < 32; ++i) {
                const LAS unsigned* sp = (const LAS unsigned*)(lds + addr[i]);
                d[i] = __builtin_amdgcn_perm(sp[1], sp[0], sel[i]);
            }
#pragma unroll
            for (int i = 0; i < 8; ++i) {
                u32x4 v = {d[i * 4], d[i * 4 + 1], d[i * 4 + 2], d[i * 4 + 3]};
                if (dead) v = (u32x4){0x80808080u, 0x80808080u, 0x80808080u, 0x80808080u};
                st16_out(o + (size_t)i * 16 * NS, v);
            }
            if (dead && lane == 0) __hip_atomic_store(tmo, call, RLX, SC_AGENT);
            if (wid == 4) STAMP(5);
        }
    } else {
        if (wid == 0) STAMP(0);
        const unsigned xa0 = XBASE + wm * 16384 + fr * 128 + ((g ^ (fr >> 1)) << 4);
        const int qq = fr >> 2, pp = lane & 3;
        const int f = qq | ((g & 1) << 2);
        const unsigned wa0 = WBASE + (8 * g + qq) * 256 + ((((wn * 8) + (pp >> 1)) ^ (f << 1)) << 4) + (pp & 1) * 8;
        const unsigned c0 = __hip_atomic_load((unsigned*)(myws + WS_CTR + b * 64), RLX, SC_AGENT);
        f32x4 acc[8][4];
#pragma unroll
        for (int i = 0; i < 8; ++i)
#pragma unroll
            for (int j = 0; j < 4; ++j) acc[i][j] = (f32x4){0.f, 0.f, 0.f, 0.f};
        bf16x8 XP[2][2], XQ[2][2], WA[4][2], WB[4][2];
#define PIN() __builtin_amdgcn_sched_barrier(0)
#define LGKM(n) do { __builtin_amdgcn_s_waitcnt(0xC07F | ((n) << 8)); PIN(); } while (0)
        auto rdX = [&](unsigned a, int off) { bf16x8 v = *(LAS bf16x8*)(lds + a + off); PIN(); return v; };
        auto rdW = [&](unsigned a, int off) { s16x4 v = __builtin_amdgcn_ds_read_tr16_b64_v4i16((LAS s16x4*)(lds + a + off)); PIN(); return v; };
        auto cat = [&](s16x4 lo, s16x4 hi) { return __builtin_bit_cast(bf16x8, __builtin_shufflevector(lo, hi, 0, 1, 2, 3, 4, 5, 6, 7)); };
        auto quarter = [&](int mf, bf16x8 (&X)[2][2], bf16x8 (&W)[4][2], auto fill) {
#pragma unroll
            for (int n = 0; n < 16; ++n) {
                const int i = n >> 3, j = (n >> 1) & 3, ks = n & 1;
                asm volatile("v_mfma_f32_16x16x32_bf16 %0, %1, %2, %0" : "+v"(acc[mf + i][j]) : "v"(W[j][ks]), "v"(X[i][ks])); PIN();
                fill(n);
            }
        };
        int xs = 0, ws = WSLOT;
        unsigned xc0 = xa0, xc1 = xa0 ^ 64, wq[4];
#pragma unroll
        for (int j = 0; j < 4; ++j) wq[j] = (wa0 ^ (j << 5)) + WSLOT;
        auto tile = [&](bf16x8 (&Wc)[4][2], bf16x8 (&Wn)[4][2]) {
            s16x4 lo;
            auto fx = [&](bf16x8 (&D)[2][2], int mf, int n) { if (n < 4) D[n >> 1][n & 1] = rdX((n & 1) ? xc1 : xc0, (mf + (n >> 1)) * 2048); };
            auto fw = [&](int f0, int cnt, int n) {
                const int m = n - 4;
                if (m >= 0 && m < 2 * cnt) {
                    const int f = f0 + (m >> 1), j = f >> 1, ks = f & 1;
                    if (!(m & 1)) lo = rdW(wq[j], ks * 8192); else Wn[j][ks] = cat(lo, rdW(wq[j], ks * 8192 + 1024));
                }
            };
            quarter(0, XP, Wc, [&](int n) { fx(XQ, 2, n); fw(0, 3, n); if (n == 14) LGKM(6); });
            quarter(2, XQ, Wc, [&](int n) { fx(XP, 4, n); fw(3, 3, n); if (n == 14) LGKM(6); });
            quarter(4, XP, Wc, [&](int n) {
                fx(XQ, 6, n); fw(6, 2, n);
                if (n == 9) { xs = (xs == NXS - 1) ? 0 : xs + 1; asm volatile("" : "+s"(xs)); PIN(); }
                if (n == 11) { xc0 = xa0 + xs * XSLOT; asm volatile("" : "+v"(xc0)); PIN(); }
                if (n == 12) { xc1 = (xa0 ^ 64) + xs * XSLOT; asm volatile("" : "+v"(xc1)); PIN(); }
                if (n == 14) LGKM(0);
            });
            BARRIER(); PIN();
            quarter(6, XQ, Wc, [&](int n) {
                fx(XP, 0, n);
                if (n == 5) { ws = (ws + WSLOT) & (NWS * WSLOT - 1); asm volatile("" : "+s"(ws)); PIN(); }
                if (n >= 7 && n < 11) { wq[n - 7] = (wa0 ^ ((n - 7) << 5)) + ws; asm volatile("" : "+v"(wq[n - 7])); PIN(); }
                if (n == 14) LGKM(0);
            });
        };
        const unsigned call = __builtin_amdgcn_readfirstlane(c0) + 1u;
#pragma unroll
        for (int i = 0; i < 8; ++i) asm volatile("" : "+v"(acc[i][0]), "+v"(acc[i][1]), "+v"(acc[i][2]), "+v"(acc[i][3]));
        LGKM(0);
        BARRIER(); PIN();
#pragma unroll
        for (int j = 0; j < 4; ++j)
#pragma unroll
            for (int ks = 0; ks < 2; ++ks) { const s16x4 lo = rdW(wa0 ^ (j << 5), ks * 8192); WA[j][ks] = cat(lo, rdW(wa0 ^ (j << 5), ks * 8192 + 1024)); }
#pragma unroll
        for (int n = 0; n < 4; ++n) XP[n >> 1][n & 1] = rdX((n & 1) ? xc1 : xc0, (n >> 1) * 2048);
        LGKM(0);
#pragma unroll 1
        for (int t = 0; t < NT; t += 2) { tile(WA, WB); tile(WB, WA); }
        asm volatile("s_nop 15" ::: "memory");
#pragma unroll
        for (int i = 0; i < 8; ++i) asm volatile("" : "+v"(acc[i][0]), "+v"(acc[i][1]), "+v"(acc[i][2]), "+v"(acc[i][3]));
        __builtin_amdgcn_s_waitcnt(0xC07F);
        BARRIER();
        float mx = 0.f;
#pragma unroll
        for (int i = 0; i < 8; ++i)
#pragma unroll
            for (int j = 0; j < 4; ++j)
                mx = __builtin_fmaxf(__builtin_fmaxf(mx, __builtin_fmaxf(acc[i][j][0], acc[i][j][1])), __builtin_fmaxf(acc[i][j][2], acc[i][j][3]));
        const unsigned wmax = wave_max_u(__float_as_uint(mx));
        if (lane == 0) {
            __hip_atomic_store((u64*)(myws + WS_LG) + b * 4 + wid, ((u64)call << 32) | wmax, RLX, SC_AGENT);
            if (wid == 0) __hip_atomic_store((unsigned*)(myws + WS_CTR + b * 64), call, RLX, SC_AGENT);
        }
        if (wid == 0) STAMP(1);
        const u64 t0 = __builtin_amdgcn_s_memrealtime();
        auto wait_scale = [&](bool& dead) {
            LAS u64* sw = (LAS u64*)(lds + EP_SCALE);
            if (wid == 0) {
                unsigned gm, n = 0;
                for (;;) {
                    const u64 v = __hip_atomic_load((u64*)(myws + WS_GG + ((call & 1) * NREP + (idx & (NREP - 1))) * 128) + (lane & 7), RLX, SC_SYS);
                    gm = (unsigned)v;
                    if (__all((unsigned)(v >> 32) == call)) break;
                    if (((++n & 63u) == 0u && __hip_atomic_load(tmo, RLX, SC_AGENT) == call) || __builtin_amdgcn_s_memrealtime() - t0 > DEADLINE) { dead = true; break; }
                    __builtin_amdgcn_s_sleep(GSLEEP);
                }
                if (dead && lane == 0) __hip_atomic_store(tmo, call, RLX, SC_AGENT);
                const float am = __uint_as_float(wave_max_u(gm));
                const float inv = am > 0.f ? 127.0f / am : 0.f;
                __hip_atomic_store(sw, ((u64)(((unsigned)__builtin_amdgcn_s_memrealtime() << 2) | (dead ? 2u : 1u)) << 32) | __float_as_uint(inv), RLX, __HIP_MEMORY_SCOPE_WORKGROUP);
                return inv;
            }
            u64 v;
            while (((v = __hip_atomic_load(sw, RLX, __HIP_MEMORY_SCOPE_WORKGROUP)) >> 32) == 0u) __builtin_amdgcn_s_sleep(1);
            dead = ((v >> 32) & 3u) == 2u;
            return __uint_as_float((unsigned)v);
        };
        bool dead = false;
        if (local) {
            const float inv = wait_scale(dead);
            if (wid == 0) STAMP(3);
            char* o = p.out + (size_t)(p.rank * MS + m0 + wm * 128 + (lane >> 2)) * NS + cbl * BN + wn * 64 + (lane & 3) * 16;
            LAS char* const lb = lds + EP_STRM + wid * STRM;
#pragma unroll
            for (int i = 0; i < 8; ++i)
#pragma unroll
                for (int j = 0; j < 4; ++j) *(LAS unsigned*)(lb + i * 1024 + fr * 64 + ((j ^ (fr >> 2)) & 3) * 16 + g * 4) = q4(acc[i][j], inv);
            u32x4 v[8];
#pragma unroll
            for (int i = 0; i < 8; ++i) v[i] = *(LAS u32x4*)(lb + i * 1024 + (lane >> 2) * 64 + ((lane ^ (lane >> 4)) & 3) * 16);
#pragma unroll
            for (int i = 0; i < 8; ++i) st16_out(o + (size_t)i * 16 * NS, dead ? (u32x4){0x80808080u, 0x80808080u, 0x80808080u, 0x80808080u} : v[i]);
        } else {
            const int slot = (((idx + 4 + 8 * wid) & 31) * 8 + p.rank) * 4 + wid;
            char* const pws = p.ws[peer];
            const unsigned tag = stream_tag(call);
            u32x4 keep;
#pragma unroll
            for (int k = 0; k < 4; ++k) {
                unsigned m = 0;
#pragma unroll
                for (int u = 7; u >= 0; --u)
#pragma unroll
                    for (int e = 3; e >= 0; --e) m = __builtin_amdgcn_alignbit(m, __float_as_uint(acc[2 * k + (u >> 2)][u & 3][e]), 31);
                keep[k] = ~m;
            }
            WAIT_V(0);
            st16_sys(pws + WS_MR + (size_t)((call & 1) * NWG * 4 + slot) * 1024 + lane * 16, keep);
            {
                const unsigned nib = lane & 15; unsigned pos = lane >> 4; u64 sel = 0x0c0c0c0c0c0c0c0cull;
#pragma unroll
                for (int e = 0; e < 4; ++e) { const bool k = (nib >> e) & 1; const u64 x = (u64)(0x0cu ^ (unsigned)e) << (8 * pos); sel ^= k ? x : 0ull; pos += k; }
                *(LAS u64*)(lds + EP_LUT + wid * 512 + lane * 8) = sel;
            }
#pragma unroll
            for (int i = 0; i < 9; ++i) *(LAS u32x4*)(lds + EP_STRM + wid * STRM + i * 1024 + lane * 16) = (u32x4){tag, tag, tag, tag};
            unsigned start[4], cum[4];
            stream_offsets(keep, start, cum);
            unsigned addr[32]; u32x2 sel[32];
#pragma unroll
            for (int k = 0; k < 4; ++k) {
                unsigned off = start[k];
#pragma unroll
                for (int u = 0; u < 8; ++u) {
                    const unsigned nib = (keep[k] >> (4 * u)) & 15u;
                    sel[k * 8 + u] = *(LAS u32x2*)(lds + EP_LUT + wid * 512 + (((off & 3u) << 4) | nib) * 8);
                    addr[k * 8 + u] = EP_STRM + wid * STRM + (off & ~3u);
                    off += __builtin_popcount(nib);
                }
            }
#pragma unroll
            for (int i = 0; i < 32; ++i) asm volatile("" : "+v"(addr[i]), "+v"(sel[i]));
            if (wid == 0) STAMP(8);
            WAIT_V(0);
            if (lane == 0) __hip_atomic_store((unsigned*)(pws + WS_MF + slot * 16), call, RLX, SC_SYS);
            const float inv = wait_scale(dead);
            if (wid == 0) STAMP(3);
            if (!dead) {
                char* const rs = pws + WS_RS + (size_t)slot * STRM + lane * 16;
                const u64 ta = __builtin_amdgcn_s_memrealtime();
                const u64 dt = slot_ticks(idx, cum[3]);
                unsigned sent = 0;
#pragma unroll
                for (int k = 0; k < 4; ++k) {
#pragma unroll
                    for (int u = 0; u < 8; ++u) {
                        const unsigned qd = q4(acc[2 * k + (u >> 2)][u & 3], inv) & 0x7f7f7f7fu;
                        LAS unsigned* d = (LAS unsigned*)(lds + addr[k * 8 + u]);
                        __hip_atomic_fetch_or(d, __builtin_amdgcn_perm(0u, qd, sel[k * 8 + u][0]), RLX, __HIP_MEMORY_SCOPE_WORKGROUP);
                        __hip_atomic_fetch_or(d + 1, __builtin_amdgcn_perm(0u, qd, sel[k * 8 + u][1]), RLX, __HIP_MEMORY_SCOPE_WORKGROUP);
                    }
                    asm volatile("" ::: "memory");
                    const unsigned lim = k < 3 ? (cum[k] & ~127u) : ((cum[3] + 63u) & ~63u);
                    if (k == 3) while (__builtin_amdgcn_s_memrealtime() - ta < dt) __builtin_amdgcn_s_sleep(1);
                    if (k == 0 || k == 3 || __builtin_amdgcn_s_memrealtime() - ta >= dt) {
                        while (sent < lim) {
                            if (sent + lane * 16 < lim) st16_sys(rs + sent, *(LAS u32x4*)(lds + EP_STRM + wid * STRM + sent + lane * 16));
                            sent = min(sent + 1024u, lim);
                        }
                    }
                }
                if (wid == 0) STAMP(9);
            }
        }
    }
}

extern "C" void kernel_launch(int rank, int nranks, ncclComm_t comm, hipStream_t stream,
                              void* const* d_in, const int* in_sizes, int n_in,
                              void* d_out, int out_size, const int* in_dtypes, int out_dtype,
                              void* const* d_out_all, void* const* d_ws_all, size_t ws_bytes) {
    if (nranks != NR || (unsigned)rank >= (unsigned)NR || ws_bytes < WS_END) return;
    static bool init[NR];
    if (!init[rank]) {
        HIP_CHECK(hipFuncSetAttribute((const void*)fused_kernel, hipFuncAttributeMaxDynamicSharedMemorySize, LDS_BYTES));
        init[rank] = true;
    }
    Params p;
    p.x = (const u16*)d_in[0]; p.w = (const u16*)d_in[1]; p.out = (char*)d_out;
    for (int r = 0; r < NR; ++r) p.ws[r] = (char*)d_ws_all[r];
    p.rank = rank; p.pad = 0; p.myws = p.ws[rank];
    fused_kernel<<<NWG, 512, LDS_BYTES, stream>>>(p);
    HIP_CHECK(hipGetLastError());
}
```

```cpp
#include <hip/hip_runtime.h>
#include <rccl/rccl.h>
#include <cstdio>

#define HIP_CHECK(expr) do { hipError_t _e = (expr); if (_e != hipSuccess) { \
    fprintf(stderr, "HIP error %s at %s:%d\n", hipGetErrorString(_e), __FILE__, __LINE__); \
    return; } } while (0)

#define LAS __attribute__((address_space(3)))
typedef short s16x4 __attribute__((ext_vector_type(4)));
typedef __bf16 bf16x8 __attribute__((ext_vector_type(8)));
typedef float f32x4 __attribute__((ext_vector_type(4)));
typedef unsigned u32x4 __attribute__((ext_vector_type(4)));
typedef unsigned u32x2 __attribute__((ext_vector_type(2)));
typedef unsigned long long u64;
typedef unsigned short u16;

constexpr int NR = 8;
constexpr int MS = 512, NG = 16384, KD = 4096;
constexpr int NS = NG / NR;
constexpr int BM = 256, BN = 128, BK = 64, NT = KD / BK;
constexpr int NXS = 3, NWS = 4;
constexpr int XSLOT = BM * BK * 2, WSLOT = BK * BN * 2;
constexpr int XBASE = 0, WBASE = NXS * XSLOT;
constexpr int LDS_BYTES = NXS * XSLOT + NWS * WSLOT;
constexpr int NWG = 256;
constexpr int STRM = 8192 + 1024;
constexpr int EP_STRM = 0;
constexpr int RCVB = 8192 + 5120;
constexpr int EP_RCV = 4 * STRM;
constexpr int EP_LUT = EP_RCV + 4 * RCVB;
constexpr int EP_SCALE = EP_LUT + 4096;

constexpr size_t WS_CTR = 0;
constexpr size_t WS_TMO = 16384;
constexpr size_t WS_LG = 20480;
constexpr int NREP = 16;
constexpr size_t WS_GG = 32768;
constexpr size_t WS_MF = 49152;
constexpr size_t WS_MR = 1u << 20;
constexpr size_t WS_RS = 3u << 20;
constexpr size_t WS_END = WS_RS + (size_t)NWG * 4 * STRM;
constexpr u64 DEADLINE = 100000000ull;
constexpr int WAUX = 2;
constexpr int NAGG = 2;
constexpr unsigned LINK_BPT = 590;
constexpr unsigned SLOT0 = 6, RWAIT = 60;

struct Params {
    const u16* x; const u16* w;
    char* out; char* myws;
    int rank; int pad;
    char* ws[NR];
};

struct LdsBase { __device__ __forceinline__ LAS char* operator+(unsigned a) const { return (LAS char*)(__UINTPTR_TYPE__)a; } };

#define WAIT_V(n) asm volatile("s_waitcnt vmcnt(" #n ")" ::: "memory")
#define BARRIER() asm volatile("s_barrier" ::: "memory")
#define RLX __ATOMIC_RELAXED
#define SC_AGENT __HIP_MEMORY_SCOPE_AGENT
#define SC_SYS __HIP_MEMORY_SCOPE_SYSTEM

template <int AUX> __device__ __forceinline__ void dma16(__amdgpu_buffer_rsrc_t rsrc, LAS char* dst, unsigned voff, unsigned soff) {
    __builtin_amdgcn_raw_ptr_buffer_load_lds(rsrc, (LAS void*)dst, 16, voff, soff, 0, AUX);
}
__device__ __forceinline__ void st16_sys(char* p, u32x4 v) {
    asm volatile("global_store_dwordx4 %0, %1, off sc0 sc1\n\ts_nop 1" :: "v"(p), "v"(v) : "memory");
}
__device__ __forceinline__ void st16_out(char* p, u32x4 v) {
    asm volatile("global_store_dwordx4 %0, %1, off sc1\n\ts_nop 1" :: "v"(p), "v"(v) : "memory");
}
__device__ __forceinline__ u32x4 ld16_sys(const char* p) {
    u32x4 v; asm volatile("global_load_dwordx4 %0, %1, off sc0 sc1\n\ts_waitcnt vmcnt(0)" : "=&v"(v) : "v"(p) : "memory"); return v;
}
__device__ __forceinline__ unsigned q4(f32x4 v, float inv) {
    unsigned r = 0;
    r = __builtin_amdgcn_cvt_pk_u8_f32(v[0] * inv, 0, r);
    r = __builtin_amdgcn_cvt_pk_u8_f32(v[1] * inv, 1, r);
    r = __builtin_amdgcn_cvt_pk_u8_f32(v[2] * inv, 2, r);
    r = __builtin_amdgcn_cvt_pk_u8_f32(v[3] * inv, 3, r);
    const unsigned h = r & 0x80808080u;
    return (r | (h - (h >> 7))) & 0x7f7f7f7fu;
}
template <int CTRL, int RMASK> __device__ __forceinline__ unsigned dpp0(unsigned x) { return (unsigned)__builtin_amdgcn_update_dpp(0, (int)x, CTRL, RMASK, 0xf, true); }
__device__ __forceinline__ unsigned wave_scan_add(unsigned x) {
    x += dpp0<0x111, 0xf>(x); x += dpp0<0x112, 0xf>(x); x += dpp0<0x114, 0xf>(x); x += dpp0<0x118, 0xf>(x);
    x += dpp0<0x142, 0xa>(x); x += dpp0<0x143, 0xc>(x); return x;
}
__device__ __forceinline__ unsigned wave_max_u(unsigned x) {
    x = max(x, dpp0<0x111, 0xf>(x)); x = max(x, dpp0<0x112, 0xf>(x)); x = max(x, dpp0<0x114, 0xf>(x)); x = max(x, dpp0<0x118, 0xf>(x));
    x = max(x, dpp0<0x142, 0xa>(x)); x = max(x, dpp0<0x143, 0xc>(x)); return (unsigned)__builtin_amdgcn_readlane((int)x, 63);
}
__device__ __forceinline__ void stream_offsets(const u32x4 keep, unsigned (&start)[4], unsigned (&cum)[4]) {
    const unsigned c0 = __builtin_popcount(keep[0]), c1 = __builtin_popcount(keep[1]), c2 = __builtin_popcount(keep[2]), c3 = __builtin_popcount(keep[3]);
    const unsigned a = wave_scan_add(c0 | (c1 << 16)), b = wave_scan_add(c2 | (c3 << 16));
    const unsigned ta = (unsigned)__builtin_amdgcn_readlane((int)a, 63), tb = (unsigned)__builtin_amdgcn_readlane((int)b, 63);
    cum[0] = ta & 0xffffu; cum[1] = cum[0] + (ta >> 16); cum[2] = cum[1] + (tb & 0xffffu); cum[3] = cum[2] + (tb >> 16);
    start[0] = (a & 0xffffu) - c0; start[1] = cum[0] + (a >> 16) - c1; start[2] = cum[1] + (b & 0xffffu) - c2; start[3] = cum[2] + (b >> 16) - c3;
}
__device__ __forceinline__ unsigned stream_tag(unsigned call) {
    const unsigned n = call % 15u + 1u;
    return ((n & 1u) << 7) | ((n & 2u) << 14) | ((n & 4u) << 21) | ((n & 8u) << 28);
}
__device__ __forceinline__ u64 slot_ticks(int pos, unsigned bytes) {
    return (u64)(((unsigned)(32 - SLOT0 + 3 * pos) * bytes) / LINK_BPT);
}
__global__ __launch_bounds__(512, 2) void fused_kernel(const Params p) {
    extern __shared__ __attribute__((aligned(16))) char smem[];
    asm volatile("" :: "s"((LAS char*)smem));
    const LdsBase lds;
    asm volatile("" :: "s"(p.x), "s"(p.w), "s"(p.myws), "s"(p.rank));
    const int tid = threadIdx.x, lane = tid & 63;
    const int wid = __builtin_amdgcn_readfirstlane(tid >> 6);
    const int b = blockIdx.x, peer = b & 7, idx = b >> 3;
    auto col_block = [](int i) { return (int)((0xd951fecba8764320ull >> (4 * i)) & 15u); };
    const int cbl = col_block(idx >> 1), mh = idx & 1;
    const int m0 = mh * BM, n0 = (peer * 16 + cbl) * BN;
    const bool local = peer == p.rank;
    char* const myws = p.myws;
    unsigned* const tmo = (unsigned*)(myws + WS_TMO);
    const int fr = lane & 15, g = lane >> 4;
    const int w4 = wid & 3, wm = w4 >> 1, wn = w4 & 1;

    if (__builtin_expect(wid >= 4, 1)) {
        const int lw = w4;
        __amdgpu_buffer_rsrc_t rx = __builtin_amdgcn_make_buffer_rsrc((void*)p.x, 0, MS * KD * 2, 0x00020000);
        __amdgpu_buffer_rsrc_t rw = __builtin_amdgcn_make_buffer_rsrc((void*)p.w, 0, (unsigned)((size_t)KD * NG * 2), 0x00020000);
        const unsigned xrow = m0 + lw * 64 + (lane >> 3);
        const unsigned xv0 = xrow * (KD * 2) + (((lane & 7) ^ (lane >> 4)) << 4);
        const unsigned xv1 = xrow * (KD * 2) + (((lane & 7) ^ (4 + (lane >> 4))) << 4);
        const unsigned wv = (lw * 4 + (lane >> 4)) * (NG * 2) + n0 * 2 + (((lane & 15) ^ (((lane >> 4) | ((lw & 2) << 1)) << 1)) << 4);
        auto issueX = [&](int t, int slot) {
            LAS char* d = lds + XBASE + slot * XSLOT + lw * 64 * 128;
#pragma unroll
            for (int i = 0; i < 8; ++i) dma16<0>(rx, d + i * 1024, (i & 1) ? xv1 : xv0, (unsigned)(t * 128 + i * 8 * KD * 2));
        };
        auto issueW = [&](int t) {
#pragma unroll
            for (int i = 0; i < 4; ++i) dma16<WAUX>(rw, lds + WBASE + (t & 3) * WSLOT + i * 4096 + lw * 1024, wv, (unsigned)(t * 64 + i * 16) * (NG * 2));
        };
        auto issueWs = [&](int t) {
#pragma unroll
            for (int i = 0; i < 4; ++i) dma16<0>(rw, lds + WBASE + (t & 3) * WSLOT + i * 4096 + lw * 1024, wv, (unsigned)(t * 64 + i * 16) * (NG * 2));
        };
        unsigned touch = 0;
        const unsigned tv = b * 16384 + (lw >> 1) * 8192 + ((lw & 1) * 8 + (lane & 7)) * 128;
        auto touchX = [&](unsigned kt0) { asm volatile("buffer_load_dword %0, %1, %2, %3 offen" : "+v"(touch) : "v"(tv), "s"(rx), "s"(kt0 * 128u) : "memory"); };
        int xs = 1;
        auto nextX = [&](int t) { issueX(t, xs); xs = (xs == NXS - 1) ? 0 : xs + 1; };
        issueW(0); issueX(0, 0); issueW(1); issueX(1, 1);
        WAIT_V(8);
        BARRIER();
        issueW(2); touchX(3); issueX(2, 2); issueW(3);
        WAIT_V(13); BARRIER();
        issueX(3, 0); issueW(4); touchX(19); touchX(35); touchX(48);
        WAIT_V(15); BARRIER();
        int t = 2;
        const unsigned wl = ((unsigned)(__UINTPTR_TYPE__)p.w + (unsigned)n0 * 2u) >> 7;
        const bool slow = ((wl - 2u) & 7u) < 2u || ((wl - 62u) & 127u) < 2u;
        const bool shift = slow ? mh : (((unsigned)(__UINTPTR_TYPE__)p.x >> 7) & 1u);
        if (shift) { nextX(4); issueW(5); WAIT_V(12); BARRIER(); t = 3; }
        if (slow) {
            for (; t < NT - 4; t += 2) {
                nextX(t + 2); issueWs(t + 3); issueWs(t + 4);
                WAIT_V(16); BARRIER();
                nextX(t + 3);
                WAIT_V(12); BARRIER();
            }
        } else {
            for (; t < NT - 4; t += 2) {
                nextX(t + 2); issueW(t + 3); issueW(t + 4);
                WAIT_V(16); BARRIER();
                nextX(t + 3);
                WAIT_V(12); BARRIER();
            }
        }
        if (!shift) { nextX(NT - 2); issueW(NT - 1); WAIT_V(12); BARRIER(); }
        nextX(NT - 1);
        char* const rs = myws + WS_RS + (size_t)(b * 4 + w4) * STRM;
        asm volatile("global_load_dword %0, %1, off sc1" : "+v"(touch) : "v"(myws + WS_CTR + b * 64) : "memory");
        WAIT_V(9); BARRIER();
        WAIT_V(0);
        asm volatile("" : "+v"(touch));
        const unsigned call = __builtin_amdgcn_readfirstlane(touch) + 1u;
        st16_sys(rs + (call & 7u) * 1024 + lane * 16, (u32x4){0u, 0u, 0u, 0u});
        BARRIER();
        BARRIER();
        if (wid == 4) asm volatile("ds_write_b128 %0, %1\n\ts_waitcnt lgkmcnt(0)" :: "v"((unsigned)EP_SCALE), "v"((u32x4){0u, 0u, 0u, 0u}) : "memory");
        WAIT_V(0);
        BARRIER();
        const unsigned tag = stream_tag(call);
        const u64 t0 = __builtin_amdgcn_s_memrealtime();
        if (local) {
            if (idx < NAGG) {
                bool dead = false, ok = false; unsigned m = 0;
                const char* gp = myws + WS_LG + w4 * 2048 + lane * 16;
                char* base = p.ws[0];
#pragma unroll
                for (int r = 1; r < NR; ++r) base = (lane & 7) == r ? p.ws[r] : base;
                base += WS_GG + ((call & 1) * NREP + (lane >> 3)) * 128 + p.rank * 8;
                asm volatile("" : "+v"(base));
                for (;;) {
                    if (!ok) {
                        u32x4 v0, v1;
                        asm volatile("global_load_dwordx4 %0, %2, off sc1\n\tglobal_load_dwordx4 %1, %2, off offset:1024 sc1\n\ts_waitcnt vmcnt(0)" : "=&v"(v0), "=&v"(v1) : "v"(gp) : "memory");
                        m = max(max(v0[0], v0[2]), max(v1[0], v1[2]));
                        ok = v0[1] == call && v0[3] == call && v1[1] == call && v1[3] == call;
                    }
                    if (__all(ok)) break;
                    if (__builtin_amdgcn_s_memrealtime() - t0 > DEADLINE) { dead = true; break; }
                    __builtin_amdgcn_s_sleep(1);
                }
                if (!dead) {
                    m = wave_max_u(m);
                    unsigned cnt = 0;
                    if (lane == 0) {
                        __hip_atomic_fetch_max((LAS unsigned*)(lds + EP_SCALE + 8), m, RLX, __HIP_MEMORY_SCOPE_WORKGROUP);
                        cnt = __hip_atomic_fetch_add((LAS unsigned*)(lds + EP_SCALE + 12), 1u, RLX, __HIP_MEMORY_SCOPE_WORKGROUP);
                    }
                    if (__builtin_amdgcn_readfirstlane(cnt) == 3) {
                        m = __hip_atomic_load((LAS unsigned*)(lds + EP_SCALE + 8), RLX, __HIP_MEMORY_SCOPE_WORKGROUP);
#pragma unroll
                        for (int i = 0; i < NREP / 8; ++i) __hip_atomic_store((u64*)(base + i * 1024), ((u64)call << 32) | m, RLX, SC_SYS);
                    }
                } else if (lane == 0) __hip_atomic_store(tmo, call, RLX, SC_AGENT);
            }
        } else {
            bool dead = false;
            {
                const unsigned nib = lane & 15; unsigned pos = lane >> 4, sel = 0;
#pragma unroll
                for (int e = 0; e < 4; ++e) { const bool k = (nib >> e) & 1; sel |= (k ? pos : 0x0cu) << (8 * e); pos += k; }
                *(LAS unsigned*)(lds + EP_LUT + wid * 512 + lane * 4) = sel;
            }
            unsigned n = 0;
            while (__hip_atomic_load((unsigned*)(myws + WS_MF + (b * 4 + w4) * 16), RLX, SC_SYS) < call) {
                if (((++n & 63u) == 0u && __hip_atomic_load(tmo, RLX, SC_AGENT) == call) || __builtin_amdgcn_s_memrealtime() - t0 > DEADLINE) { dead = true; break; }
                __builtin_amdgcn_s_sleep(32);
            }
            const u32x4 keep = ld16_sys(myws + WS_MR + (size_t)(((call & 1) * NWG + b) * 4 + w4) * 1024 + lane * 16);
            unsigned start[4], cum[4];
            stream_offsets(keep, start, cum);
            LAS char* const lb = lds + EP_RCV + w4 * RCVB;
            *(LAS u32x4*)(lb + lane * 32) = keep;
            *(LAS u32x4*)(lb + lane * 32 + 16) = (u32x4){start[0], start[1], start[2], start[3]};
            unsigned addr[32], sel[32];
#pragma unroll
            for (int gg = 0; gg < 4; ++gg) {
                const u32x4 kp = *(LAS u32x4*)(lb + ((lane >> 2) + 16 * gg) * 32), stt = *(LAS u32x4*)(lb + ((lane >> 2) + 16 * gg) * 32 + 16);
#pragma unroll
                for (int i = 0; i < 8; ++i) {
                    const unsigned sh = (i & 1) * 16 + (lane & 3) * 4;
                    const unsigned off = stt[i >> 1] + __builtin_popcount(kp[i >> 1] & ((1u << sh) - 1u)), nib = (kp[i >> 1] >> sh) & 15u;
                    sel[i * 4 + gg] = *(LAS unsigned*)(lds + EP_LUT + wid * 512 + (((off & 3u) << 4) | nib) * 4);
                    addr[i * 4 + gg] = EP_RCV + w4 * RCVB + (off & ~3u);
                }
            }
#pragma unroll
            for (int i = 0; i < 32; ++i) asm volatile("" : "+v"(addr[i]), "+v"(sel[i]));
            unsigned st;
            while ((st = __builtin_amdgcn_readfirstlane(__hip_atomic_load((LAS unsigned*)(lds + EP_SCALE + 4), RLX, __HIP_MEMORY_SCOPE_WORKGROUP))) == 0u) __builtin_amdgcn_s_sleep(8);
            const unsigned total = cum[3];
            const int sidx = (idx - 4 - 8 * w4) & 31;
            {
                const unsigned dt = ((unsigned)slot_ticks(sidx, total) + RWAIT) << 2;
                while (((unsigned)__builtin_amdgcn_s_memrealtime() << 2) - (st & ~3u) < dt) __builtin_amdgcn_s_sleep(2);
            }
            char* o = p.out + (size_t)(peer * MS + m0 + wm * 128 + (lane >> 2)) * NS + col_block(sidx >> 1) * BN + wn * 64 + (lane & 3) * 16;
            const __amdgpu_buffer_rsrc_t rr = __builtin_amdgcn_make_buffer_rsrc((void*)rs, 0, (total + 15u) & ~15u, 0x00020000);
            unsigned pos = 0;
            while (pos < total && !dead) {
                u32x4 v[5];
#pragma unroll
                for (int i = 0; i < 5; ++i) v[i] = __builtin_amdgcn_raw_buffer_load_b128(rr, pos + lane * 16 + i * 1024, 0, 17);
                unsigned adv = 5120;
#pragma unroll
                for (int i = 4; i >= 0; --i) {
                    const bool ok = pos + i * 1024 + lane * 16 >= total || ((((v[i][0] ^ tag) | (v[i][1] ^ tag) | (v[i][2] ^ tag) | (v[i][3] ^ tag)) & 0x80808080u) == 0u);
                    const u64 bad = ~__ballot(ok);
                    *(LAS u32x4*)(lb + pos + i * 1024 + lane * 16) = v[i] & 0x7f7f7f7fu;
                    if (bad) adv = i * 1024 + (unsigned)__builtin_ctzll(bad) * 16u;
                }
                pos += adv & ~127u;
                if (pos < total) {
                    if (((++n & 63u) == 0u && __hip_atomic_load(tmo, RLX, SC_AGENT) == call) || __builtin_amdgcn_s_memrealtime() - t0 > DEADLINE) dead = true;
                    __builtin_amdgcn_s_sleep(1);
                }
            }
            unsigned d[32];
#pragma unroll
            for (int i = 0; i < 32; ++i) {
                const LAS unsigned* sp = (const LAS unsigned*)(lds + addr[i]);
                d[i] = __builtin_amdgcn_perm(sp[1], sp[0], sel[i]);
            }
#pragma unroll
            for (int i = 0; i < 8; ++i) {
                u32x4 v = {d[i * 4], d[i * 4 + 1], d[i * 4 + 2], d[i * 4 + 3]};
                if (dead) v = (u32x4){0x80808080u, 0x80808080u, 0x80808080u, 0x80808080u};
                st16_out(o + (size_t)i * 16 * NS, v);
            }
            if (dead && lane == 0) __hip_atomic_store(tmo, call, RLX, SC_AGENT);
        }
    } else {
        const unsigned xa0 = XBASE + wm * 16384 + fr * 128 + ((g ^ (fr >> 1)) << 4);
        const int qq = fr >> 2, pp = lane & 3;
        const int f = qq | ((g & 1) << 2);
        const unsigned wa0 = WBASE + (8 * g + qq) * 256 + ((((wn * 8) + (pp >> 1)) ^ (f << 1)) << 4) + (pp & 1) * 8;
        const unsigned c0 = __hip_atomic_load((unsigned*)(myws + WS_CTR + b * 64), RLX, SC_AGENT);
        f32x4 acc[8][4];
#pragma unroll
        for (int i = 0; i < 8; ++i)
#pragma unroll
            for (int j = 0; j < 4; ++j) acc[i][j] = (f32x4){0.f, 0.f, 0.f, 0.f};
        bf16x8 XP[2][2], XQ[2][2], WA[4][2], WB[4][2];
#define PIN() __builtin_amdgcn_sched_barrier(0)
#define LGKM(n) do { __builtin_amdgcn_s_waitcnt(0xC07F | ((n) << 8)); PIN(); } while (0)
        auto rdX = [&](unsigned a, int off) { bf16x8 v = *(LAS bf16x8*)(lds + a + off); PIN(); return v; };
        auto rdW = [&](unsigned a, int off) { s16x4 v = __builtin_amdgcn_ds_read_tr16_b64_v4i16((LAS s16x4*)(lds + a + off)); PIN(); return v; };
        auto cat = [&](s16x4 lo, s16x4 hi) { return __builtin_bit_cast(bf16x8, __builtin_shufflevector(lo, hi, 0, 1, 2, 3, 4, 5, 6, 7)); };
        auto quarter = [&](int mf, bf16x8 (&X)[2][2], bf16x8 (&W)[4][2], auto fill) {
#pragma unroll
            for (int n = 0; n < 16; ++n) {
                const int i = n >> 3, j = (n >> 1) & 3, ks = n & 1;
                asm volatile("v_mfma_f32_16x16x32_bf16 %0, %1, %2, %0" : "+v"(acc[mf + i][j]) : "v"(W[j][ks]), "v"(X[i][ks])); PIN();
                fill(n);
            }
        };
        int xs = 0, ws = WSLOT;
        unsigned xc0 = xa0, xc1 = xa0 ^ 64, wq[4];
#pragma unroll
        for (int j = 0; j < 4; ++j) wq[j] = (wa0 ^ (j << 5)) + WSLOT;
        auto tile = [&](bf16x8 (&Wc)[4][2], bf16x8 (&Wn)[4][2]) {
            s16x4 lo;
            auto fx = [&](bf16x8 (&D)[2][2], int mf, int n) { if (n < 4) D[n >> 1][n & 1] = rdX((n & 1) ? xc1 : xc0, (mf + (n >> 1)) * 2048); };
            auto fw = [&](int f0, int cnt, int n) {
                const int m = n - 4;
                if (m >= 0 && m < 2 * cnt) {
                    const int f = f0 + (m >> 1), j = f >> 1, ks = f & 1;
                    if (!(m & 1)) lo = rdW(wq[j], ks * 8192); else Wn[j][ks] = cat(lo, rdW(wq[j], ks * 8192 + 1024));
                }
            };
            quarter(0, XP, Wc, [&](int n) { fx(XQ, 2, n); fw(0, 3, n); if (n == 14) LGKM(6); });
            quarter(2, XQ, Wc, [&](int n) { fx(XP, 4, n); fw(3, 3, n); if (n == 14) LGKM(6); });
            quarter(4, XP, Wc, [&](int n) {
                fx(XQ, 6, n); fw(6, 2, n);
                if (n == 9) { xs = (xs == NXS - 1) ? 0 : xs + 1; asm volatile("" : "+s"(xs)); PIN(); }
                if (n == 11) { xc0 = xa0 + xs * XSLOT; asm volatile("" : "+v"(xc0)); PIN(); }
                if (n == 12) { xc1 = (xa0 ^ 64) + xs * XSLOT; asm volatile("" : "+v"(xc1)); PIN(); }
                if (n == 14) LGKM(0);
            });
            BARRIER(); PIN();
            quarter(6, XQ, Wc, [&](int n) {
                fx(XP, 0, n);
                if (n == 5) { ws = (ws + WSLOT) & (NWS * WSLOT - 1); asm volatile("" : "+s"(ws)); PIN(); }
                if (n >= 7 && n < 11) { wq[n - 7] = (wa0 ^ ((n - 7) << 5)) + ws; asm volatile("" : "+v"(wq[n - 7])); PIN(); }
                if (n == 14) LGKM(0);
            });
        };
        const unsigned call = __builtin_amdgcn_readfirstlane(c0) + 1u;
#pragma unroll
        for (int i = 0; i < 8; ++i) asm volatile("" : "+v"(acc[i][0]), "+v"(acc[i][1]), "+v"(acc[i][2]), "+v"(acc[i][3]));
        LGKM(0);
        BARRIER(); PIN();
#pragma unroll
        for (int j = 0; j < 4; ++j)
#pragma unroll
            for (int ks = 0; ks < 2; ++ks) { const s16x4 lo = rdW(wa0 ^ (j << 5), ks * 8192); WA[j][ks] = cat(lo, rdW(wa0 ^ (j << 5), ks * 8192 + 1024)); }
#pragma unroll
        for (int n = 0; n < 4; ++n) XP[n >> 1][n & 1] = rdX((n & 1) ? xc1 : xc0, (n >> 1) * 2048);
        LGKM(0);
#pragma unroll 1
        for (int t = 0; t < NT; t += 2) { tile(WA, WB); tile(WB, WA); }
        asm volatile("s_nop 15" ::: "memory");
#pragma unroll
        for (int i = 0; i < 8; ++i) asm volatile("" : "+v"(acc[i][0]), "+v"(acc[i][1]), "+v"(acc[i][2]), "+v"(acc[i][3]));
        int mx = 0;
#pragma unroll
        for (int i = 0; i < 8; ++i)
#pragma unroll
            for (int j = 0; j < 4; ++j) {
                mx = max(max(mx, __float_as_int(acc[i][j][0])), __float_as_int(acc[i][j][1]));
                mx = max(max(mx, __float_as_int(acc[i][j][2])), __float_as_int(acc[i][j][3]));
            }
        unsigned wmax = wave_max_u((unsigned)mx);
        asm volatile("" : "+s"(wmax));
        BARRIER();
        if (lane == 0) {
            __hip_atomic_store((u64*)(myws + WS_LG) + b * 4 + wid, ((u64)call << 32) | wmax, RLX, SC_AGENT);
            if (wid == 0) __hip_atomic_store((unsigned*)(myws + WS_CTR + b * 64), call, RLX, SC_AGENT);
        }
        const u64 t0 = __builtin_amdgcn_s_memrealtime();
        auto wait_scale = [&](bool& dead) {
            LAS u64* sw = (LAS u64*)(lds + EP_SCALE);
            if (wid == 0) {
                unsigned gm, n = 0;
                for (;;) {
                    const u64 v = __hip_atomic_load((u64*)(myws + WS_GG + ((call & 1) * NREP + (idx & (NREP - 1))) * 128) + (lane & 7), RLX, SC_SYS);
                    gm = (unsigned)v;
                    if (__all((unsigned)(v >> 32) == call)) break;
                    if (((++n & 63u) == 0u && __hip_atomic_load(tmo, RLX, SC_AGENT) == call) || __builtin_amdgcn_s_memrealtime() - t0 > DEADLINE) { dead = true; break; }
                    __builtin_amdgcn_s_sleep(1);
                }
                if (dead && lane == 0) __hip_atomic_store(tmo, call, RLX, SC_AGENT);
                const float am = __uint_as_float(wave_max_u(gm));
                const float inv = am > 0.f ? 127.0f / am : 0.f;
                __hip_atomic_store(sw, ((u64)(((unsigned)__builtin_amdgcn_s_memrealtime() << 2) | (dead ? 2u : 1u)) << 32) | __float_as_uint(inv), RLX, __HIP_MEMORY_SCOPE_WORKGROUP);
                return inv;
            }
            u64 v;
            while (((v = __hip_atomic_load(sw, RLX, __HIP_MEMORY_SCOPE_WORKGROUP)) >> 32) == 0u) __builtin_amdgcn_s_sleep(1);
            dead = ((v >> 32) & 3u) == 2u;
            return __uint_as_float((unsigned)v);
        };
        bool dead = false;
        if (local) {
            const float inv = wait_scale(dead);
            char* o = p.out + (size_t)(p.rank * MS + m0 + wm * 128 + (lane >> 2)) * NS + cbl * BN + wn * 64 + (lane & 3) * 16;
            LAS char* const lb = lds + EP_STRM + wid * STRM;
#pragma unroll
            for (int i = 0; i < 8; ++i)
#pragma unroll
                for (int j = 0; j < 4; ++j) *(LAS unsigned*)(lb + i * 1024 + fr * 64 + ((j ^ (fr >> 2)) & 3) * 16 + g * 4) = q4(acc[i][j], inv);
            u32x4 v[8];
#pragma unroll
            for (int i = 0; i < 8; ++i) v[i] = *(LAS u32x4*)(lb + i * 1024 + (lane >> 2) * 64 + ((lane ^ (lane >> 4)) & 3) * 16);
#pragma unroll
            for (int i = 0; i < 8; ++i) st16_out(o + (size_t)i * 16 * NS, dead ? (u32x4){0x80808080u, 0x80808080u, 0x80808080u, 0x80808080u} : v[i]);
        } else {
            const int slot = (((idx + 4 + 8 * wid) & 31) * 8 + p.rank) * 4 + wid;
            char* const pws = p.ws[peer];
            const unsigned tag = stream_tag(call);
            u32x4 keep;
#pragma unroll
            for (int k = 0; k < 4; ++k) {
                unsigned m = 0;
#pragma unroll
                for (int u = 7; u >= 0; --u)
#pragma unroll
                    for (int e = 3; e >= 0; --e) m = __builtin_amdgcn_alignbit(m, __float_as_uint(acc[2 * k + (u >> 2)][u & 3][e]), 31);
                keep[k] = ~m;
            }
            WAIT_V(0);
            st16_sys(pws + WS_MR + (size_t)((call & 1) * NWG * 4 + slot) * 1024 + lane * 16, keep);
            {
                const unsigned nib = lane & 15; unsigned pos = lane >> 4; u64 sel = 0x0c0c0c0c0c0c0c0cull;
#pragma unroll
                for (int e = 0; e < 4; ++e) { const bool k = (nib >> e) & 1; const u64 x = (u64)(0x0cu ^ (unsigned)e) << (8 * pos); sel ^= k ? x : 0ull; pos += k; }
                *(LAS u64*)(lds + EP_LUT + wid * 512 + lane * 8) = sel;
            }
#pragma unroll
            for (int i = 0; i < 9; ++i) *(LAS u32x4*)(lds + EP_STRM + wid * STRM + i * 1024 + lane * 16) = (u32x4){tag, tag, tag, tag};
            unsigned start[4], cum[4];
            stream_offsets(keep, start, cum);
            unsigned addr[32]; u32x2 sel[32];
#pragma unroll
            for (int k = 0; k < 4; ++k) {
                unsigned off = start[k];
#pragma unroll
                for (int u = 0; u < 8; ++u) {
                    const unsigned nib = (keep[k] >> (4 * u)) & 15u;
                    sel[k * 8 + u] = *(LAS u32x2*)(lds + EP_LUT + wid * 512 + (((off & 3u) << 4) | nib) * 8);
                    addr[k * 8 + u] = EP_STRM + wid * STRM + (off & ~3u);
                    off += __builtin_popcount(nib);
                }
            }
#pragma unroll
            for (int i = 0; i < 32; ++i) asm volatile("" : "+v"(addr[i]), "+v"(sel[i]));
            WAIT_V(0);
            if (lane == 0) __hip_atomic_store((unsigned*)(pws + WS_MF + slot * 16), call, RLX, SC_SYS);
            const float inv = wait_scale(dead);
            if (!dead) {
                char* const rs = pws + WS_RS + (size_t)slot * STRM + lane * 16;
                const u64 ta = __builtin_amdgcn_s_memrealtime();
                const u64 dt = slot_ticks(idx, cum[3]);
                unsigned sent = 0;
#pragma unroll
                for (int k = 0; k < 4; ++k) {
#pragma unroll
                    for (int u = 0; u < 8; ++u) {
                        const unsigned qd = q4(acc[2 * k + (u >> 2)][u & 3], inv);
                        LAS unsigned* d = (LAS unsigned*)(lds + addr[k * 8 + u]);
                        __hip_atomic_fetch_or(d, __builtin_amdgcn_perm(0u, qd, sel[k * 8 + u][0]), RLX, __HIP_MEMORY_SCOPE_WORKGROUP);
                        __hip_atomic_fetch_or(d + 1, __builtin_amdgcn_perm(0u, qd, sel[k * 8 + u][1]), RLX, __HIP_MEMORY_SCOPE_WORKGROUP);
                    }
                    asm volatile("" ::: "memory");
                    const unsigned lim = k < 3 ? (cum[k] & ~127u) : ((cum[3] + 63u) & ~63u);
                    if (k == 3) while (__builtin_amdgcn_s_memrealtime() - ta < dt) __builtin_amdgcn_s_sleep(1);
                    if (k == 0 || k == 3 || __builtin_amdgcn_s_memrealtime() - ta >= dt) {
                        while (sent < lim) {
                            if (sent + lane * 16 < lim) st16_sys(rs + sent, *(LAS u32x4*)(lds + EP_STRM + wid * STRM + sent + lane * 16));
                            sent = min(sent + 1024u, lim);
                        }
                    }
                }
            }
        }
    }
}

extern "C" void kernel_launch(int rank, int nranks, ncclComm_t comm, hipStream_t stream,
                              void* const* d_in, const int* in_sizes, int n_in,
                              void* d_out, int out_size, const int* in_dtypes, int out_dtype,
                              void* const* d_out_all, void* const* d_ws_all, size_t ws_bytes) {
    if (nranks != NR || (unsigned)rank >= (unsigned)NR || ws_bytes < WS_END) return;
    static bool init[NR];
    if (!init[rank]) {
        HIP_CHECK(hipFuncSetAttribute((const void*)fused_kernel, hipFuncAttributeMaxDynamicSharedMemorySize, LDS_BYTES));
        init[rank] = true;
    }
    Params p;
    p.x = (const u16*)d_in[0]; p.w = (const u16*)d_in[1]; p.out = (char*)d_out;
    for (int r = 0; r < NR; ++r) p.ws[r] = (char*)d_ws_all[r];
    p.rank = rank; p.pad = 0; p.myws = p.ws[rank];
    fused_kernel<<<NWG, 512, LDS_BYTES, stream>>>(p);
    HIP_CHECK(hipGetLastError());
}
```

```cpp
#include <hip/hip_runtime.h>
#include <rccl/rccl.h>
#include <cstdio>

#define HIP_CHECK(expr) do { hipError_t _e = (expr); if (_e != hipSuccess) { \
    fprintf(stderr, "HIP error %s at %s:%d\n", hipGetErrorString(_e), __FILE__, __LINE__); \
    return; } } while (0)

#define LAS __attribute__((address_space(3)))
typedef short s16x4 __attribute__((ext_vector_type(4)));
typedef __bf16 bf16x8 __attribute__((ext_vector_type(8)));
typedef float f32x4 __attribute__((ext_vector_type(4)));
typedef unsigned u32x4 __attribute__((ext_vector_type(4)));
typedef unsigned u32x2 __attribute__((ext_vector_type(2)));
typedef unsigned long long u64;
typedef unsigned short u16;

constexpr int NR = 8;
constexpr int MS = 512, NG = 16384, KD = 4096;
constexpr int NS = NG / NR;
constexpr int BM = 256, BN = 128, BK = 64, NT = KD / BK;
constexpr int NXS = 3, NWS = 4;
constexpr int XSLOT = BM * BK * 2, WSLOT = BK * BN * 2;
constexpr int XBASE = 0, WBASE = NXS * XSLOT;
constexpr int LDS_BYTES = NXS * XSLOT + NWS * WSLOT;
constexpr int NWG = 256;
constexpr int STRM = 8192 + 1024;
constexpr int EP_STRM = 0;
constexpr int RCVB = 8192 + 5120;
constexpr int EP_RCV = 4 * STRM;
constexpr int EP_LUT = EP_RCV + 4 * RCVB;
constexpr int EP_SCALE = EP_LUT + 4096;

constexpr size_t WS_CTR = 0;
constexpr size_t WS_TMO = 16384;
constexpr size_t WS_LG = 20480;
constexpr int NREP = 16;
constexpr size_t WS_GG = 32768;
constexpr size_t WS_IN = 2u << 20, INBOX = 2u << 20;
constexpr size_t IN_MF = 0;
constexpr size_t IN_MR = 4096;
constexpr size_t IN_RS = IN_MR + 2 * 128 * 1024;
static_assert(IN_RS + 128 * (size_t)STRM <= INBOX, "inbox");
constexpr size_t WS_END = WS_IN + NR * INBOX;
constexpr u64 DEADLINE = 100000000ull;
constexpr int WAUX = 2;
constexpr int NAGG = 2;
constexpr unsigned LINK_BPT = 590;
constexpr unsigned SLOT0 = 6, RWAIT = 60;

struct Params {
    const u16* x; const u16* w;
    char* out; char* myws;
    int rank; int pad;
    char* ws[NR];
};

struct LdsBase { __device__ __forceinline__ LAS char* operator+(unsigned a) const { return (LAS char*)(__UINTPTR_TYPE__)a; } };

#define WAIT_V(n) asm volatile("s_waitcnt vmcnt(" #n ")" ::: "memory")
#define BARRIER() asm volatile("s_barrier" ::: "memory")
#define RLX __ATOMIC_RELAXED
#define SC_AGENT __HIP_MEMORY_SCOPE_AGENT
#define SC_SYS __HIP_MEMORY_SCOPE_SYSTEM

template <int AUX> __device__ __forceinline__ void dma16(__amdgpu_buffer_rsrc_t rsrc, LAS char* dst, unsigned voff, unsigned soff) {
    __builtin_amdgcn_raw_ptr_buffer_load_lds(rsrc, (LAS void*)dst, 16, voff, soff, 0, AUX);
}
__device__ __forceinline__ void st16_sys(char* p, u32x4 v) {
    asm volatile("global_store_dwordx4 %0, %1, off sc0 sc1\n\ts_nop 1" :: "v"(p), "v"(v) : "memory");
}
__device__ __forceinline__ void st16_out(char* p, u32x4 v) {
    asm volatile("global_store_dwordx4 %0, %1, off sc1\n\ts_nop 1" :: "v"(p), "v"(v) : "memory");
}
__device__ __forceinline__ u32x4 ld16_sys(const char* p) {
    u32x4 v; asm volatile("global_load_dwordx4 %0, %1, off sc0 sc1\n\ts_waitcnt vmcnt(0)" : "=&v"(v) : "v"(p) : "memory"); return v;
}
__device__ __forceinline__ unsigned q4(f32x4 v, float inv) {
    unsigned r = 0;
    r = __builtin_amdgcn_cvt_pk_u8_f32(v[0] * inv, 0, r);
    r = __builtin_amdgcn_cvt_pk_u8_f32(v[1] * inv, 1, r);
    r = __builtin_amdgcn_cvt_pk_u8_f32(v[2] * inv, 2, r);
    r = __builtin_amdgcn_cvt_pk_u8_f32(v[3] * inv, 3, r);
    const unsigned h = r & 0x80808080u;
    return (r | (h - (h >> 7))) & 0x7f7f7f7fu;
}
template <int CTRL, int RMASK> __device__ __forceinline__ unsigned dpp0(unsigned x) { return (unsigned)__builtin_amdgcn_update_dpp(0, (int)x, CTRL, RMASK, 0xf, true); }
__device__ __forceinline__ unsigned wave_scan_add(unsigned x) {
    x += dpp0<0x111, 0xf>(x); x += dpp0<0x112, 0xf>(x); x += dpp0<0x114, 0xf>(x); x += dpp0<0x118, 0xf>(x);
    x += dpp0<0x142, 0xa>(x); x += dpp0<0x143, 0xc>(x); return x;
}
__device__ __forceinline__ unsigned wave_max_u(unsigned x) {
    x = max(x, dpp0<0x111, 0xf>(x)); x = max(x, dpp0<0x112, 0xf>(x)); x = max(x, dpp0<0x114, 0xf>(x)); x = max(x, dpp0<0x118, 0xf>(x));
    x = max(x, dpp0<0x142, 0xa>(x)); x = max(x, dpp0<0x143, 0xc>(x)); return (unsigned)__builtin_amdgcn_readlane((int)x, 63);
}
__device__ __forceinline__ void stream_offsets(const u32x4 keep, unsigned (&start)[4], unsigned (&cum)[4]) {
    const unsigned c0 = __builtin_popcount(keep[0]), c1 = __builtin_popcount(keep[1]), c2 = __builtin_popcount(keep[2]), c3 = __builtin_popcount(keep[3]);
    const unsigned a = wave_scan_add(c0 | (c1 << 16)), b = wave_scan_add(c2 | (c3 << 16));
    const unsigned ta = (unsigned)__builtin_amdgcn_readlane((int)a, 63), tb = (unsigned)__builtin_amdgcn_readlane((int)b, 63);
    cum[0] = ta & 0xffffu; cum[1] = cum[0] + (ta >> 16); cum[2] = cum[1] + (tb & 0xffffu); cum[3] = cum[2] + (tb >> 16);
    start[0] = (a & 0xffffu) - c0; start[1] = cum[0] + (a >> 16) - c1; start[2] = cum[1] + (b & 0xffffu) - c2; start[3] = cum[2] + (b >> 16) - c3;
}
__device__ __forceinline__ unsigned stream_tag(unsigned call) {
    const unsigned n = call % 15u + 1u;
    return ((n & 1u) << 7) | ((n & 2u) << 14) | ((n & 4u) << 21) | ((n & 8u) << 28);
}
__device__ __forceinline__ u64 slot_ticks(int pos, unsigned bytes) {
    return (u64)(((unsigned)(32 - SLOT0 + 3 * pos) * bytes) / LINK_BPT);
}
__global__ __launch_bounds__(512, 2) void fused_kernel(const Params p) {
    extern __shared__ __attribute__((aligned(16))) char smem[];
    asm volatile("" :: "s"((LAS char*)smem));
    const LdsBase lds;
    asm volatile("" :: "s"(p.x), "s"(p.w), "s"(p.myws), "s"(p.rank));
    const int tid = threadIdx.x, lane = tid & 63;
    const int wid = __builtin_amdgcn_readfirstlane(tid >> 6);
    const int b = blockIdx.x, peer = b & 7, idx = b >> 3;
    auto col_block = [](int i) { return (int)((0xd951fecba8764320ull >> (4 * i)) & 15u); };
    const int cbl = col_block(idx >> 1), mh = idx & 1;
    const int m0 = mh * BM, n0 = (peer * 16 + cbl) * BN;
    const bool local = peer == p.rank;
    char* const myws = p.myws;
    unsigned* const tmo = (unsigned*)(myws + WS_TMO);
    const int fr = lane & 15, g = lane >> 4;
    const int w4 = wid & 3, wm = w4 >> 1, wn = w4 & 1;

    if (__builtin_expect(wid >= 4, 1)) {
        const int lw = w4;
        __amdgpu_buffer_rsrc_t rx = __builtin_amdgcn_make_buffer_rsrc((void*)p.x, 0, MS * KD * 2, 0x00020000);
        __amdgpu_buffer_rsrc_t rw = __builtin_amdgcn_make_buffer_rsrc((void*)p.w, 0, (unsigned)((size_t)KD * NG * 2), 0x00020000);
        const unsigned xrow = m0 + lw * 64 + (lane >> 3);
        const unsigned xv0 = xrow * (KD * 2) + (((lane & 7) ^ (lane >> 4)) << 4);
        const unsigned xv1 = xrow * (KD * 2) + (((lane & 7) ^ (4 + (lane >> 4))) << 4);
        const unsigned wv = (lw * 4 + (lane >> 4)) * (NG * 2) + n0 * 2 + (((lane & 15) ^ (((lane >> 4) | ((lw & 2) << 1)) << 1)) << 4);
        auto issueX = [&](int t, int slot) {
            LAS char* d = lds + XBASE + slot * XSLOT + lw * 64 * 128;
#pragma unroll
            for (int i = 0; i < 8; ++i) dma16<0>(rx, d + i * 1024, (i & 1) ? xv1 : xv0, (unsigned)(t * 128 + i * 8 * KD * 2));
        };
        auto issueW = [&](int t) {
#pragma unroll
            for (int i = 0; i < 4; ++i) dma16<WAUX>(rw, lds + WBASE + (t & 3) * WSLOT + i * 4096 + lw * 1024, wv, (unsigned)(t * 64 + i * 16) * (NG * 2));
        };
        auto issueWs = [&](int t) {
#pragma unroll
            for (int i = 0; i < 4; ++i) dma16<0>(rw, lds + WBASE + (t & 3) * WSLOT + i * 4096 + lw * 1024, wv, (unsigned)(t * 64 + i * 16) * (NG * 2));
        };
        unsigned touch = 0;
        const unsigned tv = b * 16384 + (lw >> 1) * 8192 + ((lw & 1) * 8 + (lane & 7)) * 128;
        auto touchX = [&](unsigned kt0) { asm volatile("buffer_load_dword %0, %1, %2, %3 offen" : "+v"(touch) : "v"(tv), "s"(rx), "s"(kt0 * 128u) : "memory"); };
        int xs = 1;
        auto nextX = [&](int t) { issueX(t, xs); xs = (xs == NXS - 1) ? 0 : xs + 1; };
        issueW(0); issueX(0, 0); issueW(1); issueX(1, 1);
        WAIT_V(8);
        BARRIER();
        unsigned cv;
        issueW(2); touchX(3);
        asm volatile("global_load_dword %0, %1, off sc1" : "=v"(cv) : "v"(myws + WS_CTR + b * 64) : "memory");
        issueX(2, 2); issueW(3);
        WAIT_V(14); BARRIER();
        issueX(3, 0); issueW(4); touchX(19); touchX(35); touchX(48);
        asm volatile("s_waitcnt vmcnt(15)" : "+v"(cv) :: "memory");
        BARRIER();
        const unsigned call = __builtin_amdgcn_readfirstlane(cv) + 1u;
        {
            const char* wp;
            if (lw == 0) wp = local ? p.ws[idx & 7] + WS_GG : p.ws[peer] + WS_IN + p.rank * INBOX + (idx & 1) * (INBOX - 64);
            else if (lw == 1) wp = (idx & 2) ? myws + WS_IN + peer * INBOX + (idx & 1) * (INBOX - 64) : p.out + (size_t)(peer * MS + (idx & 1) * (MS - 1)) * NS;
            else wp = (const char*)p.w + (size_t)(idx * 2 + (lw & 1)) * (BK * NG * 2) + 64 * 1024;
            asm volatile("s_load_dword s95, %0, 0x0" :: "s"(wp) : "s95");
        }
        int t = 2;
        const unsigned wl = ((unsigned)(__UINTPTR_TYPE__)p.w + (unsigned)n0 * 2u) >> 7;
        const bool slow = ((wl - 2u) & 7u) < 2u || ((wl - 62u) & 127u) < 2u;
        const bool shift = slow ? mh : (((unsigned)(__UINTPTR_TYPE__)p.x >> 7) & 1u);
        if (shift) { nextX(4); issueW(5); WAIT_V(12); BARRIER(); t = 3; }
        if (slow) {
            for (; t < NT - 4; t += 2) {
                nextX(t + 2); issueWs(t + 3); issueWs(t + 4);
                WAIT_V(16); BARRIER();
                nextX(t + 3);
                WAIT_V(12); BARRIER();
            }
        } else {
            for (; t < NT - 4; t += 2) {
                nextX(t + 2); issueW(t + 3); issueW(t + 4);
                WAIT_V(16); BARRIER();
                nextX(t + 3);
                WAIT_V(12); BARRIER();
            }
        }
        if (!shift) { nextX(NT - 2); issueW(NT - 1); WAIT_V(12); BARRIER(); }
        nextX(NT - 1);
        char* const inbox = myws + WS_IN + peer * INBOX;
        char* const rs = inbox + IN_RS + (size_t)(idx * 4 + w4) * STRM;
        st16_sys(rs + (call & 7u) * 1024 + lane * 16, (u32x4){0u, 0u, 0u, 0u});
        WAIT_V(9); BARRIER();
        WAIT_V(1); BARRIER();
        BARRIER();
        if (wid == 4) asm volatile("ds_write_b128 %0, %1\n\ts_waitcnt lgkmcnt(0)" :: "v"((unsigned)EP_SCALE), "v"((u32x4){0u, 0u, 0u, 0u}) : "memory");
        WAIT_V(0);
        asm volatile("s_waitcnt lgkmcnt(0)" :: "v"(touch) : "s95");
        BARRIER();
        const unsigned tag = stream_tag(call);
        const u64 t0 = __builtin_amdgcn_s_memrealtime();
        if (local) {
            if (idx < NAGG) {
                bool dead = false, ok = false; unsigned m = 0;
                const char* gp = myws + WS_LG + w4 * 2048 + lane * 16;
                char* base = p.ws[0];
#pragma unroll
                for (int r = 1; r < NR; ++r) base = (lane & 7) == r ? p.ws[r] : base;
                base += WS_GG + ((call & 1) * NREP + (lane >> 3)) * 128 + p.rank * 8;
                asm volatile("" : "+v"(base));
                for (;;) {
                    if (!ok) {
                        u32x4 v0, v1;
                        asm volatile("global_load_dwordx4 %0, %2, off sc1\n\tglobal_load_dwordx4 %1, %2, off offset:1024 sc1\n\ts_waitcnt vmcnt(0)" : "=&v"(v0), "=&v"(v1) : "v"(gp) : "memory");
                        m = max(max(v0[0], v0[2]), max(v1[0], v1[2]));
                        ok = v0[1] == call && v0[3] == call && v1[1] == call && v1[3] == call;
                    }
                    if (__all(ok)) break;
                    if (__builtin_amdgcn_s_memrealtime() - t0 > DEADLINE) { dead = true; break; }
                    __builtin_amdgcn_s_sleep(1);
                }
                if (!dead) {
                    m = wave_max_u(m);
                    unsigned cnt = 0;
                    if (lane == 0) {
                        __hip_atomic_fetch_max((LAS unsigned*)(lds + EP_SCALE + 8), m, RLX, __HIP_MEMORY_SCOPE_WORKGROUP);
                        cnt = __hip_atomic_fetch_add((LAS unsigned*)(lds + EP_SCALE + 12), 1u, RLX, __HIP_MEMORY_SCOPE_WORKGROUP);
                    }
                    if (__builtin_amdgcn_readfirstlane(cnt) == 3) {
                        m = __hip_atomic_load((LAS unsigned*)(lds + EP_SCALE + 8), RLX, __HIP_MEMORY_SCOPE_WORKGROUP);
#pragma unroll
                        for (int i = 0; i < NREP / 8; ++i) __hip_atomic_store((u64*)(base + i * 1024), ((u64)call << 32) | m, RLX, SC_SYS);
                    }
                } else if (lane == 0) __hip_atomic_store(tmo, call, RLX, SC_AGENT);
            }
        } else {
            bool dead = false;
            {
                const unsigned nib = lane & 15; unsigned pos = lane >> 4, sel = 0;
#pragma unroll
                for (int e = 0; e < 4; ++e) { const bool k = (nib >> e) & 1; sel |= (k ? pos : 0x0cu) << (8 * e); pos += k; }
                *(LAS unsigned*)(lds + EP_LUT + wid * 512 + lane * 4) = sel;
            }
            unsigned n = 0;
            while (__hip_atomic_load((unsigned*)(inbox + IN_MF + (idx * 4 + w4) * 16), RLX, SC_SYS) < call) {
                if (((++n & 63u) == 0u && __hip_atomic_load(tmo, RLX, SC_AGENT) == call) || __builtin_amdgcn_s_memrealtime() - t0 > DEADLINE) { dead = true; break; }
                __builtin_amdgcn_s_sleep(32);
            }
            const u32x4 keep = ld16_sys(inbox + IN_MR + (size_t)((call & 1) * 128 + idx * 4 + w4) * 1024 + lane * 16);
            unsigned start[4], cum[4];
            stream_offsets(keep, start, cum);
            LAS char* const lb = lds + EP_RCV + w4 * RCVB;
            *(LAS u32x4*)(lb + lane * 32) = keep;
            *(LAS u32x4*)(lb + lane * 32 + 16) = (u32x4){start[0], start[1], start[2], start[3]};
            unsigned addr[32], sel[32];
#pragma unroll
            for (int gg = 0; gg < 4; ++gg) {
                const u32x4 kp = *(LAS u32x4*)(lb + ((lane >> 2) + 16 * gg) * 32), stt = *(LAS u32x4*)(lb + ((lane >> 2) + 16 * gg) * 32 + 16);
#pragma unroll
                for (int i = 0; i < 8; ++i) {
                    const unsigned sh = (i & 1) * 16 + (lane & 3) * 4;
                    const unsigned off = stt[i >> 1] + __builtin_popcount(kp[i >> 1] & ((1u << sh) - 1u)), nib = (kp[i >> 1] >> sh) & 15u;
                    sel[i * 4 + gg] = *(LAS unsigned*)(lds + EP_LUT + wid * 512 + (((off & 3u) << 4) | nib) * 4);
                    addr[i * 4 + gg] = EP_RCV + w4 * RCVB + (off & ~3u);
                }
            }
#pragma unroll
            for (int i = 0; i < 32; ++i) asm volatile("" : "+v"(addr[i]), "+v"(sel[i]));
            unsigned st;
            while ((st = __builtin_amdgcn_readfirstlane(__hip_atomic_load((LAS unsigned*)(lds + EP_SCALE + 4), RLX, __HIP_MEMORY_SCOPE_WORKGROUP))) == 0u) __builtin_amdgcn_s_sleep(8);
            const unsigned total = cum[3];
            const int sidx = (idx - 4 - 8 * w4) & 31;
            {
                const unsigned dt = ((unsigned)slot_ticks(sidx, total) + RWAIT) << 2;
                while (((unsigned)__builtin_amdgcn_s_memrealtime() << 2) - (st & ~3u) < dt) __builtin_amdgcn_s_sleep(2);
            }
            char* o = p.out + (size_t)(peer * MS + m0 + wm * 128 + (lane >> 2)) * NS + col_block(sidx >> 1) * BN + wn * 64 + (lane & 3) * 16;
            const __amdgpu_buffer_rsrc_t rr = __builtin_amdgcn_make_buffer_rsrc((void*)rs, 0, (total + 15u) & ~15u, 0x00020000);
            unsigned pos = 0;
            while (pos < total && !dead) {
                u32x4 v[5];
#pragma unroll
                for (int i = 0; i < 5; ++i) v[i] = __builtin_amdgcn_raw_buffer_load_b128(rr, pos + lane * 16 + i * 1024, 0, 17);
                unsigned adv = 5120;
#pragma unroll
                for (int i = 4; i >= 0; --i) {
                    const bool ok = pos + i * 1024 + lane * 16 >= total || ((((v[i][0] ^ tag) | (v[i][1] ^ tag) | (v[i][2] ^ tag) | (v[i][3] ^ tag)) & 0x80808080u) == 0u);
                    const u64 bad = ~__ballot(ok);
                    *(LAS u32x4*)(lb + pos + i * 1024 + lane * 16) = v[i] & 0x7f7f7f7fu;
                    if (bad) adv = i * 1024 + (unsigned)__builtin_ctzll(bad) * 16u;
                }
                pos += adv & ~127u;
                if (pos < total) {
                    if (((++n & 63u) == 0u && __hip_atomic_load(tmo, RLX, SC_AGENT) == call) || __builtin_amdgcn_s_memrealtime() - t0 > DEADLINE) dead = true;
                    __builtin_amdgcn_s_sleep(1);
                }
            }
            unsigned d[32];
#pragma unroll
            for (int i = 0; i < 32; ++i) {
                const LAS unsigned* sp = (const LAS unsigned*)(lds + addr[i]);
                d[i] = __builtin_amdgcn_perm(sp[1], sp[0], sel[i]);
            }
#pragma unroll
            for (int i = 0; i < 8; ++i) {
                u32x4 v = {d[i * 4], d[i * 4 + 1], d[i * 4 + 2], d[i * 4 + 3]};
                if (dead) v = (u32x4){0x80808080u, 0x80808080u, 0x80808080u, 0x80808080u};
                st16_out(o + (size_t)i * 16 * NS, v);
            }
            if (dead && lane == 0) __hip_atomic_store(tmo, call, RLX, SC_AGENT);
        }
    } else {
        const unsigned xa0 = XBASE + wm * 16384 + fr * 128 + ((g ^ (fr >> 1)) << 4);
        const int qq = fr >> 2, pp = lane & 3;
        const int f = qq | ((g & 1) << 2);
        const unsigned wa0 = WBASE + (8 * g + qq) * 256 + ((((wn * 8) + (pp >> 1)) ^ (f << 1)) << 4) + (pp & 1) * 8;
        const unsigned c0 = __hip_atomic_load((unsigned*)(myws + WS_CTR + b * 64), RLX, SC_AGENT);
        f32x4 acc[8][4];
#pragma unroll
        for (int i = 0; i < 8; ++i)
#pragma unroll
            for (int j = 0; j < 4; ++j) acc[i][j] = (f32x4){0.f, 0.f, 0.f, 0.f};
        bf16x8 XP[2][2], XQ[2][2], WA[4][2], WB[4][2];
#define PIN() __builtin_amdgcn_sched_barrier(0)
#define LGKM(n) do { __builtin_amdgcn_s_waitcnt(0xC07F | ((n) << 8)); PIN(); } while (0)
        auto rdX = [&](unsigned a, int off) { bf16x8 v = *(LAS bf16x8*)(lds + a + off); PIN(); return v; };
        auto rdW = [&](unsigned a, int off) { s16x4 v = __builtin_amdgcn_ds_read_tr16_b64_v4i16((LAS s16x4*)(lds + a + off)); PIN(); return v; };
        auto cat = [&](s16x4 lo, s16x4 hi) { return __builtin_bit_cast(bf16x8, __builtin_shufflevector(lo, hi, 0, 1, 2, 3, 4, 5, 6, 7)); };
        auto quarter = [&](int mf, bf16x8 (&X)[2][2], bf16x8 (&W)[4][2], auto fill) {
#pragma unroll
            for (int n = 0; n < 16; ++n) {
                const int i = n >> 3, j = (n >> 1) & 3, ks = n & 1;
                asm volatile("v_mfma_f32_16x16x32_bf16 %0, %1, %2, %0" : "+v"(acc[mf + i][j]) : "v"(W[j][ks]), "v"(X[i][ks])); PIN();
                fill(n);
            }
        };
        int xs = 0, ws = WSLOT;
        unsigned xc0 = xa0, xc1 = xa0 ^ 64, wq[4];
#pragma unroll
        for (int j = 0; j < 4; ++j) wq[j] = (wa0 ^ (j << 5)) + WSLOT;
        auto tile = [&](bf16x8 (&Wc)[4][2], bf16x8 (&Wn)[4][2]) {
            s16x4 lo;
            auto fx = [&](bf16x8 (&D)[2][2], int mf, int n) { if (n < 4) D[n >> 1][n & 1] = rdX((n & 1) ? xc1 : xc0, (mf + (n >> 1)) * 2048); };
            auto fw = [&](int f0, int cnt, int n) {
                const int m = n - 4;
                if (m >= 0 && m < 2 * cnt) {
                    const int f = f0 + (m >> 1), j = f >> 1, ks = f & 1;
                    if (!(m & 1)) lo = rdW(wq[j], ks * 8192); else Wn[j][ks] = cat(lo, rdW(wq[j], ks * 8192 + 1024));
                }
            };
            quarter(0, XP, Wc, [&](int n) { fx(XQ, 2, n); fw(0, 3, n); if (n == 14) LGKM(6); });
            quarter(2, XQ, Wc, [&](int n) { fx(XP, 4, n); fw(3, 3, n); if (n == 14) LGKM(6); });
            quarter(4, XP, Wc, [&](int n) {
                fx(XQ, 6, n); fw(6, 2, n);
                if (n == 9) { xs = (xs == NXS - 1) ? 0 : xs + 1; asm volatile("" : "+s"(xs)); PIN(); }
                if (n == 11) { xc0 = xa0 + xs * XSLOT; asm volatile("" : "+v"(xc0)); PIN(); }
                if (n == 12) { xc1 = (xa0 ^ 64) + xs * XSLOT; asm volatile("" : "+v"(xc1)); PIN(); }
                if (n == 14) LGKM(0);
            });
            BARRIER(); PIN();
            quarter(6, XQ, Wc, [&](int n) {
                fx(XP, 0, n);
                if (n == 5) { ws = (ws + WSLOT) & (NWS * WSLOT - 1); asm volatile("" : "+s"(ws)); PIN(); }
                if (n >= 7 && n < 11) { wq[n - 7] = (wa0 ^ ((n - 7) << 5)) + ws; asm volatile("" : "+v"(wq[n - 7])); PIN(); }
                if (n == 14) LGKM(0);
            });
        };
        const unsigned call = __builtin_amdgcn_readfirstlane(c0) + 1u;
#pragma unroll
        for (int i = 0; i < 8; ++i) asm volatile("" : "+v"(acc[i][0]), "+v"(acc[i][1]), "+v"(acc[i][2]), "+v"(acc[i][3]));
        LGKM(0);
        BARRIER(); PIN();
#pragma unroll
        for (int j = 0; j < 4; ++j)
#pragma unroll
            for (int ks = 0; ks < 2; ++ks) { const s16x4 lo = rdW(wa0 ^ (j << 5), ks * 8192); WA[j][ks] = cat(lo, rdW(wa0 ^ (j << 5), ks * 8192 + 1024)); }
#pragma unroll
        for (int n = 0; n < 4; ++n) XP[n >> 1][n & 1] = rdX((n & 1) ? xc1 : xc0, (n >> 1) * 2048);
        LGKM(0);
#pragma unroll 1
        for (int t = 0; t < NT; t += 2) { tile(WA, WB); tile(WB, WA); }
        asm volatile("s_nop 15" ::: "memory");
#pragma unroll
        for (int i = 0; i < 8; ++i) asm volatile("" : "+v"(acc[i][0]), "+v"(acc[i][1]), "+v"(acc[i][2]), "+v"(acc[i][3]));
        int mx = 0;
#pragma unroll
        for (int i = 0; i < 8; ++i)
#pragma unroll
            for (int j = 0; j < 4; ++j) {
                mx = max(max(mx, __float_as_int(acc[i][j][0])), __float_as_int(acc[i][j][1]));
                mx = max(max(mx, __float_as_int(acc[i][j][2])), __float_as_int(acc[i][j][3]));
            }
        unsigned wmax = wave_max_u((unsigned)mx);
        asm volatile("" : "+s"(wmax));
        BARRIER();
        if (lane == 0) {
            __hip_atomic_store((u64*)(myws + WS_LG) + b * 4 + wid, ((u64)call << 32) | wmax, RLX, SC_AGENT);
            if (wid == 0) __hip_atomic_store((unsigned*)(myws + WS_CTR + b * 64), call, RLX, SC_AGENT);
        }
        const u64 t0 = __builtin_amdgcn_s_memrealtime();
        auto wait_scale = [&](bool& dead) {
            LAS u64* sw = (LAS u64*)(lds + EP_SCALE);
            if (wid == 0) {
                unsigned gm, n = 0;
                for (;;) {
                    const u64 v = __hip_atomic_load((u64*)(myws + WS_GG + ((call & 1) * NREP + (idx & (NREP - 1))) * 128) + (lane & 7), RLX, SC_SYS);
                    gm = (unsigned)v;
                    if (__all((unsigned)(v >> 32) == call)) break;
                    if (((++n & 63u) == 0u && __hip_atomic_load(tmo, RLX, SC_AGENT) == call) || __builtin_amdgcn_s_memrealtime() - t0 > DEADLINE) { dead = true; break; }
                    __builtin_amdgcn_s_sleep(1);
                }
                if (dead && lane == 0) __hip_atomic_store(tmo, call, RLX, SC_AGENT);
                const float am = __uint_as_float(wave_max_u(gm));
                const float inv = am > 0.f ? 127.0f / am : 0.f;
                __hip_atomic_store(sw, ((u64)(((unsigned)__builtin_amdgcn_s_memrealtime() << 2) | (dead ? 2u : 1u)) << 32) | __float_as_uint(inv), RLX, __HIP_MEMORY_SCOPE_WORKGROUP);
                return inv;
            }
            u64 v;
            while (((v = __hip_atomic_load(sw, RLX, __HIP_MEMORY_SCOPE_WORKGROUP)) >> 32) == 0u) __builtin_amdgcn_s_sleep(1);
            dead = ((v >> 32) & 3u) == 2u;
            return __uint_as_float((unsigned)v);
        };
        bool dead = false;
        if (local) {
            const float inv = wait_scale(dead);
            char* o = p.out + (size_t)(p.rank * MS + m0 + wm * 128 + (lane >> 2)) * NS + cbl * BN + wn * 64 + (lane & 3) * 16;
            LAS char* const lb = lds + EP_STRM + wid * STRM;
#pragma unroll
            for (int i = 0; i < 8; ++i)
#pragma unroll
                for (int j = 0; j < 4; ++j) *(LAS unsigned*)(lb + i * 1024 + fr * 64 + ((j ^ (fr >> 2)) & 3) * 16 + g * 4) = q4(acc[i][j], inv);
            u32x4 v[8];
#pragma unroll
            for (int i = 0; i < 8; ++i) v[i] = *(LAS u32x4*)(lb + i * 1024 + (lane >> 2) * 64 + ((lane ^ (lane >> 4)) & 3) * 16);
#pragma unroll
            for (int i = 0; i < 8; ++i) st16_out(o + (size_t)i * 16 * NS, dead ? (u32x4){0x80808080u, 0x80808080u, 0x80808080u, 0x80808080u} : v[i]);
        } else {
            const int slot = ((idx + 4 + 8 * wid) & 31) * 4 + wid;
            char* const pws = p.ws[peer] + WS_IN + p.rank * INBOX;
            const unsigned tag = stream_tag(call);
            u32x4 keep;
#pragma unroll
            for (int k = 0; k < 4; ++k) {
                unsigned m = 0;
#pragma unroll
                for (int u = 7; u >= 0; --u)
#pragma unroll
                    for (int e = 3; e >= 0; --e) m = __builtin_amdgcn_alignbit(m, __float_as_uint(acc[2 * k + (u >> 2)][u & 3][e]), 31);
                keep[k] = ~m;
            }
            WAIT_V(0);
            st16_sys(pws + IN_MR + (size_t)((call & 1) * 128 + slot) * 1024 + lane * 16, keep);
            {
                const unsigned nib = lane & 15; unsigned pos = lane >> 4; u64 sel = 0x0c0c0c0c0c0c0c0cull;
#pragma unroll
                for (int e = 0; e < 4; ++e) { const bool k = (nib >> e) & 1; const u64 x = (u64)(0x0cu ^ (unsigned)e) << (8 * pos); sel ^= k ? x : 0ull; pos += k; }
                *(LAS u64*)(lds + EP_LUT + wid * 512 + lane * 8) = sel;
            }
#pragma unroll
            for (int i = 0; i < 9; ++i) *(LAS u32x4*)(lds + EP_STRM + wid * STRM + i * 1024 + lane * 16) = (u32x4){tag, tag, tag, tag};
            unsigned start[4], cum[4];
            stream_offsets(keep, start, cum);
            unsigned addr[32]; u32x2 sel[32];
#pragma unroll
            for (int k = 0; k < 4; ++k) {
                unsigned off = start[k];
#pragma unroll
                for (int u = 0; u < 8; ++u) {
                    const unsigned nib = (keep[k] >> (4 * u)) & 15u;
                    sel[k * 8 + u] = *(LAS u32x2*)(lds + EP_LUT + wid * 512 + (((off & 3u) << 4) | nib) * 8);
                    addr[k * 8 + u] = EP_STRM + wid * STRM + (off & ~3u);
                    off += __builtin_popcount(nib);
                }
            }
#pragma unroll
            for (int i = 0; i < 32; ++i) asm volatile("" : "+v"(addr[i]), "+v"(sel[i]));
            WAIT_V(0);
            if (lane == 0) __hip_atomic_store((unsigned*)(pws + IN_MF + slot * 16), call, RLX, SC_SYS);
            const float inv = wait_scale(dead);
            if (!dead) {
                char* const rs = pws + IN_RS + (size_t)slot * STRM + lane * 16;
                const u64 ta = __builtin_amdgcn_s_memrealtime();
                const u64 dt = slot_ticks(idx, cum[3]);
                unsigned sent = 0;
#pragma unroll
                for (int k = 0; k < 4; ++k) {
#pragma unroll
                    for (int u = 0; u < 8; ++u) {
                        const unsigned qd = q4(acc[2 * k + (u >> 2)][u & 3], inv);
                        LAS unsigned* d = (LAS unsigned*)(lds + addr[k * 8 + u]);
                        __hip_atomic_fetch_or(d, __builtin_amdgcn_perm(0u, qd, sel[k * 8 + u][0]), RLX, __HIP_MEMORY_SCOPE_WORKGROUP);
                        __hip_atomic_fetch_or(d + 1, __builtin_amdgcn_perm(0u, qd, sel[k * 8 + u][1]), RLX, __HIP_MEMORY_SCOPE_WORKGROUP);
                    }
                    asm volatile("" ::: "memory");
                    const unsigned lim = k < 3 ? (cum[k] & ~127u) : ((cum[3] + 63u) & ~63u);
                    if (k == 3) while (__builtin_amdgcn_s_memrealtime() - ta < dt) __builtin_amdgcn_s_sleep(1);
                    if (k == 0 || k == 3 || __builtin_amdgcn_s_memrealtime() - ta >= dt) {
                        while (sent < lim) {
                            if (sent + lane * 16 < lim) st16_sys(rs + sent, *(LAS u32x4*)(lds + EP_STRM + wid * STRM + sent + lane * 16));
                            sent = min(sent + 1024u, lim);
                        }
                    }
                }
            }
        }
    }
}

extern "C" void kernel_launch(int rank, int nranks, ncclComm_t comm, hipStream_t stream,
                              void* const* d_in, const int* in_sizes, int n_in,
                              void* d_out, int out_size, const int* in_dtypes, int out_dtype,
                              void* const* d_out_all, void* const* d_ws_all, size_t ws_bytes) {
    if (nranks != NR || (unsigned)rank >= (unsigned)NR || ws_bytes < WS_END) return;
    static bool init[NR];
    if (!init[rank]) {
        HIP_CHECK(hipFuncSetAttribute((const void*)fused_kernel, hipFuncAttributeMaxDynamicSharedMemorySize, LDS_BYTES));
        init[rank] = true;
    }
    Params p;
    p.x = (const u16*)d_in[0]; p.w = (const u16*)d_in[1]; p.out = (char*)d_out;
    for (int r = 0; r < NR; ++r) p.ws[r] = (char*)d_ws_all[r];
    p.rank = rank; p.pad = 0; p.myws = p.ws[rank];
    fused_kernel<<<NWG, 512, LDS_BYTES, stream>>>(p);
    HIP_CHECK(hipGetLastError());
}
```

```cpp
#include <hip/hip_runtime.h>
#include <rccl/rccl.h>
#include <cstdio>

#define HIP_CHECK(expr) do { hipError_t _e = (expr); if (_e != hipSuccess) { \
    fprintf(stderr, "HIP error %s at %s:%d\n", hipGetErrorString(_e), __FILE__, __LINE__); \
    return; } } while (0)

#define LAS __attribute__((address_space(3)))
typedef short s16x4 __attribute__((ext_vector_type(4)));
typedef __bf16 bf16x8 __attribute__((ext_vector_type(8)));
typedef float f32x4 __attribute__((ext_vector_type(4)));
typedef unsigned u32x4 __attribute__((ext_vector_type(4)));
typedef unsigned u32x2 __attribute__((ext_vector_type(2)));
typedef unsigned long long u64;
typedef unsigned short u16;

constexpr int NR = 8;
constexpr int MS = 512, NG = 16384, KD = 4096;
constexpr int NS = NG / NR;
constexpr int BM = 256, BN = 128, BK = 64, NT = KD / BK;
constexpr int NXS = 3, NWS = 4;
constexpr int XSLOT = BM * BK * 2, WSLOT = BK * BN * 2;
constexpr int XBASE = 0, WBASE = NXS * XSLOT;
constexpr int LDS_BYTES = NXS * XSLOT + NWS * WSLOT;
constexpr int NWG = 256;
constexpr int STRM = 8192 + 1024;
constexpr int EP_STRM = 0;
constexpr int RCVB = 8192 + 5120;
constexpr int EP_RCV = 4 * STRM;
constexpr int EP_LUT = EP_RCV + 4 * RCVB;
constexpr int EP_SCALE = EP_LUT + 4096;

constexpr size_t WS_CTR = 0;
constexpr size_t WS_TMO = 16384;
constexpr size_t WS_LG = 20480;
constexpr int NREP = 16;
constexpr size_t WS_GG = 32768;
constexpr size_t WS_IN = 2u << 20, INBOX = 2u << 20;
constexpr size_t IN_MF = 0;
constexpr size_t IN_MR = 4096;
constexpr size_t IN_RS = IN_MR + 2 * 128 * 1024;
static_assert(IN_RS + 128 * (size_t)STRM <= INBOX, "inbox");
constexpr size_t WS_END = WS_IN + NR * INBOX;
constexpr u64 DEADLINE = 100000000ull;
constexpr int WAUX = 2;
constexpr int NAGG = 2;
constexpr unsigned LINK_BPT = 590;
constexpr unsigned SLOT0 = 6, RWAIT = 60;
constexpr unsigned PACE_T0 = 260, PACE_TAU = 72, PACE_MIN = 132;

struct Params {
    const u16* x; const u16* w;
    char* out; char* myws;
    int rank; int pad;
    char* ws[NR];
};

struct LdsBase { __device__ __forceinline__ LAS char* operator+(unsigned a) const { return (LAS char*)(__UINTPTR_TYPE__)a; } };

#define WAIT_V(n) asm volatile("s_waitcnt vmcnt(" #n ")" ::: "memory")
#define BARRIER() asm volatile("s_barrier" ::: "memory")
#define RLX __ATOMIC_RELAXED
#define SC_AGENT __HIP_MEMORY_SCOPE_AGENT
#define SC_SYS __HIP_MEMORY_SCOPE_SYSTEM

template <int AUX> __device__ __forceinline__ void dma16(__amdgpu_buffer_rsrc_t rsrc, LAS char* dst, unsigned voff, unsigned soff) {
    __builtin_amdgcn_raw_ptr_buffer_load_lds(rsrc, (LAS void*)dst, 16, voff, soff, 0, AUX);
}
__device__ __forceinline__ void st16_sys(char* p, u32x4 v) {
    asm volatile("global_store_dwordx4 %0, %1, off sc0 sc1\n\ts_nop 1" :: "v"(p), "v"(v) : "memory");
}
__device__ __forceinline__ void st16_out(char* p, u32x4 v) {
    asm volatile("global_store_dwordx4 %0, %1, off sc1\n\ts_nop 1" :: "v"(p), "v"(v) : "memory");
}
__device__ __forceinline__ u32x4 ld16_sys(const char* p) {
    u32x4 v; asm volatile("global_load_dwordx4 %0, %1, off sc0 sc1\n\ts_waitcnt vmcnt(0)" : "=&v"(v) : "v"(p) : "memory"); return v;
}
__device__ __forceinline__ unsigned q4(f32x4 v, float inv) {
    unsigned r = 0;
    r = __builtin_amdgcn_cvt_pk_u8_f32(v[0] * inv, 0, r);
    r = __builtin_amdgcn_cvt_pk_u8_f32(v[1] * inv, 1, r);
    r = __builtin_amdgcn_cvt_pk_u8_f32(v[2] * inv, 2, r);
    r = __builtin_amdgcn_cvt_pk_u8_f32(v[3] * inv, 3, r);
    const unsigned h = r & 0x80808080u;
    return (r | (h - (h >> 7))) & 0x7f7f7f7fu;
}
template <int CTRL, int RMASK> __device__ __forceinline__ unsigned dpp0(unsigned x) { return (unsigned)__builtin_amdgcn_update_dpp(0, (int)x, CTRL, RMASK, 0xf, true); }
__device__ __forceinline__ unsigned wave_scan_add(unsigned x) {
    x += dpp0<0x111, 0xf>(x); x += dpp0<0x112, 0xf>(x); x += dpp0<0x114, 0xf>(x); x += dpp0<0x118, 0xf>(x);
    x += dpp0<0x142, 0xa>(x); x += dpp0<0x143, 0xc>(x); return x;
}
__device__ __forceinline__ unsigned wave_max_u(unsigned x) {
    x = max(x, dpp0<0x111, 0xf>(x)); x = max(x, dpp0<0x112, 0xf>(x)); x = max(x, dpp0<0x114, 0xf>(x)); x = max(x, dpp0<0x118, 0xf>(x));
    x = max(x, dpp0<0x142, 0xa>(x)); x = max(x, dpp0<0x143, 0xc>(x)); return (unsigned)__builtin_amdgcn_readlane((int)x, 63);
}
__device__ __forceinline__ void stream_offsets(const u32x4 keep, unsigned (&start)[4], unsigned (&cum)[4]) {
    const unsigned c0 = __builtin_popcount(keep[0]), c1 = __builtin_popcount(keep[1]), c2 = __builtin_popcount(keep[2]), c3 = __builtin_popcount(keep[3]);
    const unsigned a = wave_scan_add(c0 | (c1 << 16)), b = wave_scan_add(c2 | (c3 << 16));
    const unsigned ta = (unsigned)__builtin_amdgcn_readlane((int)a, 63), tb = (unsigned)__builtin_amdgcn_readlane((int)b, 63);
    cum[0] = ta & 0xffffu; cum[1] = cum[0] + (ta >> 16); cum[2] = cum[1] + (tb & 0xffffu); cum[3] = cum[2] + (tb >> 16);
    start[0] = (a & 0xffffu) - c0; start[1] = cum[0] + (a >> 16) - c1; start[2] = cum[1] + (b & 0xffffu) - c2; start[3] = cum[2] + (b >> 16) - c3;
}
__device__ __forceinline__ unsigned stream_tag(unsigned call) {
    const unsigned n = call % 15u + 1u;
    return ((n & 1u) << 7) | ((n & 2u) << 14) | ((n & 4u) << 21) | ((n & 8u) << 28);
}
__device__ __forceinline__ u64 slot_ticks(int pos, unsigned bytes) {
    return (u64)(((unsigned)(32 - SLOT0 + 3 * pos) * bytes) / LINK_BPT);
}
__global__ __launch_bounds__(512, 2) void fused_kernel(const Params p) {
    extern __shared__ __attribute__((aligned(16))) char smem[];
    asm volatile("" :: "s"((LAS char*)smem));
    const LdsBase lds;
    asm volatile("" :: "s"(p.x), "s"(p.w), "s"(p.myws), "s"(p.rank));
    const int tid = threadIdx.x, lane = tid & 63;
    const int wid = __builtin_amdgcn_readfirstlane(tid >> 6);
    const int b = blockIdx.x, peer = b & 7, idx = b >> 3;
    auto col_block = [](int i) { return (int)((0xd951fecba8764320ull >> (4 * i)) & 15u); };
    const int cbl = col_block(idx >> 1), mh = idx & 1;
    const int m0 = mh * BM, n0 = (peer * 16 + cbl) * BN;
    const bool local = peer == p.rank;
    char* const myws = p.myws;
    unsigned* const tmo = (unsigned*)(myws + WS_TMO);
    const int fr = lane & 15, g = lane >> 4;
    const int w4 = wid & 3, wm = w4 >> 1, wn = w4 & 1;

    if (__builtin_expect(wid >= 4, 1)) {
        const int lw = w4;
        const u64 tstart = __builtin_amdgcn_s_memrealtime();
        __amdgpu_buffer_rsrc_t rx = __builtin_amdgcn_make_buffer_rsrc((void*)p.x, 0, MS * KD * 2, 0x00020000);
        __amdgpu_buffer_rsrc_t rw = __builtin_amdgcn_make_buffer_rsrc((void*)p.w, 0, (unsigned)((size_t)KD * NG * 2), 0x00020000);
        const unsigned xrow = m0 + lw * 64 + (lane >> 3);
        const unsigned xv0 = xrow * (KD * 2) + (((lane & 7) ^ (lane >> 4)) << 4);
        const unsigned xv1 = xrow * (KD * 2) + (((lane & 7) ^ (4 + (lane >> 4))) << 4);
        const unsigned wv = (lw * 4 + (lane >> 4)) * (NG * 2) + n0 * 2 + (((lane & 15) ^ (((lane >> 4) | ((lw & 2) << 1)) << 1)) << 4);
        auto issueX = [&](int t, int slot) {
            LAS char* d = lds + XBASE + slot * XSLOT + lw * 64 * 128;
#pragma unroll
            for (int i = 0; i < 8; ++i) dma16<0>(rx, d + i * 1024, (i & 1) ? xv1 : xv0, (unsigned)(t * 128 + i * 8 * KD * 2));
        };
        auto issueW = [&](int t) {
#pragma unroll
            for (int i = 0; i < 4; ++i) dma16<WAUX>(rw, lds + WBASE + (t & 3) * WSLOT + i * 4096 + lw * 1024, wv, (unsigned)(t * 64 + i * 16) * (NG * 2));
        };
        auto issueWs = [&](int t) {
#pragma unroll
            for (int i = 0; i < 4; ++i) dma16<0>(rw, lds + WBASE + (t & 3) * WSLOT + i * 4096 + lw * 1024, wv, (unsigned)(t * 64 + i * 16) * (NG * 2));
        };
        unsigned touch = 0;
        const unsigned tv = b * 16384 + (lw >> 1) * 8192 + ((lw & 1) * 8 + (lane & 7)) * 128;
        auto touchX = [&](unsigned kt0) { asm volatile("buffer_load_dword %0, %1, %2, %3 offen" : "+v"(touch) : "v"(tv), "s"(rx), "s"(kt0 * 128u) : "memory"); };
        int xs = 1;
        auto nextX = [&](int t) { issueX(t, xs); xs = (xs == NXS - 1) ? 0 : xs + 1; };
        issueW(0); issueX(0, 0); issueW(1); issueX(1, 1);
        WAIT_V(8);
        BARRIER();
        unsigned cv;
        issueW(2); touchX(3);
        asm volatile("global_load_dword %0, %1, off sc1" : "=v"(cv) : "v"(myws + WS_CTR + b * 64) : "memory");
        issueX(2, 2); issueW(3);
        WAIT_V(14); BARRIER();
        issueX(3, 0); issueW(4); touchX(19); touchX(35); touchX(48);
        asm volatile("s_waitcnt vmcnt(15)" : "+v"(cv) :: "memory");
        BARRIER();
        const unsigned call = __builtin_amdgcn_readfirstlane(cv) + 1u;
        {
            const char* wp;
            if (lw == 0) wp = local ? p.ws[idx & 7] + WS_GG : p.ws[peer] + WS_IN + p.rank * INBOX + (idx & 1) * (INBOX - 64);
            else if (lw == 1) wp = (idx & 2) ? myws + WS_IN + peer * INBOX + (idx & 1) * (INBOX - 64) : p.out + (size_t)(peer * MS + (idx & 1) * (MS - 1)) * NS;
            else wp = (const char*)p.w + (size_t)(idx * 2 + (lw & 1)) * (BK * NG * 2) + 64 * 1024;
            asm volatile("s_load_dword s95, %0, 0x0" :: "s"(wp) : "s95");
        }
        u64 tlast = 0;
        auto pace = [&](int n) {
            if (lw == 1) {
                const u64 a = tstart + PACE_T0 + (unsigned)n * PACE_TAU, c = tlast + PACE_MIN, tgt = a > c ? a : c;
                while (__builtin_amdgcn_s_memrealtime() < tgt) __builtin_amdgcn_s_sleep(1);
            }
        };
        int t = 2;
        const unsigned wl = ((unsigned)(__UINTPTR_TYPE__)p.w + (unsigned)n0 * 2u) >> 7;
        const bool slow = ((wl - 2u) & 7u) < 2u || ((wl - 62u) & 127u) < 2u;
        const bool shift = slow ? mh : (((unsigned)(__UINTPTR_TYPE__)p.x >> 7) & 1u);
        if (shift) { nextX(4); issueW(5); WAIT_V(12); BARRIER(); t = 3; }
        if (slow) {
            for (; t < NT - 4; t += 2) {
                nextX(t + 2); issueWs(t + 3); issueWs(t + 4);
                WAIT_V(16); BARRIER();
                nextX(t + 3);
                pace(t + 2);
                WAIT_V(12); BARRIER();
                tlast = __builtin_amdgcn_s_memrealtime();
            }
        } else {
            for (; t < NT - 4; t += 2) {
                nextX(t + 2); issueW(t + 3); issueW(t + 4);
                WAIT_V(16); BARRIER();
                nextX(t + 3);
                pace(t + 2);
                WAIT_V(12); BARRIER();
                tlast = __builtin_amdgcn_s_memrealtime();
            }
        }
        if (!shift) { nextX(NT - 2); issueW(NT - 1); WAIT_V(12); BARRIER(); }
        nextX(NT - 1);
        char* const inbox = myws + WS_IN + peer * INBOX;
        char* const rs = inbox + IN_RS + (size_t)(idx * 4 + w4) * STRM;
        st16_sys(rs + (call & 7u) * 1024 + lane * 16, (u32x4){0u, 0u, 0u, 0u});
        WAIT_V(9); BARRIER();
        WAIT_V(1); BARRIER();
        BARRIER();
        if (wid == 4) asm volatile("ds_write_b128 %0, %1\n\ts_waitcnt lgkmcnt(0)" :: "v"((unsigned)EP_SCALE), "v"((u32x4){0u, 0u, 0u, 0u}) : "memory");
        WAIT_V(0);
        asm volatile("s_waitcnt lgkmcnt(0)" :: "v"(touch) : "s95");
        BARRIER();
        const unsigned tag = stream_tag(call);
        const u64 t0 = __builtin_amdgcn_s_memrealtime();
        if (local) {
            if (idx < NAGG) {
                bool dead = false, ok = false; unsigned m = 0;
                const char* gp = myws + WS_LG + w4 * 2048 + lane * 16;
                char* base = p.ws[0];
#pragma unroll
                for (int r = 1; r < NR; ++r) base = (lane & 7) == r ? p.ws[r] : base;
                base += WS_GG + ((call & 1) * NREP + (lane >> 3)) * 128 + p.rank * 8;
                asm volatile("" : "+v"(base));
                for (;;) {
                    if (!ok) {
                        u32x4 v0, v1;
                        asm volatile("global_load_dwordx4 %0, %2, off sc1\n\tglobal_load_dwordx4 %1, %2, off offset:1024 sc1\n\ts_waitcnt vmcnt(0)" : "=&v"(v0), "=&v"(v1) : "v"(gp) : "memory");
                        m = max(max(v0[0], v0[2]), max(v1[0], v1[2]));
                        ok = v0[1] == call && v0[3] == call && v1[1] == call && v1[3] == call;
                    }
                    if (__all(ok)) break;
                    if (__builtin_amdgcn_s_memrealtime() - t0 > DEADLINE) { dead = true; break; }
                    __builtin_amdgcn_s_sleep(1);
                }
                if (!dead) {
                    m = wave_max_u(m);
                    unsigned cnt = 0;
                    if (lane == 0) {
                        __hip_atomic_fetch_max((LAS unsigned*)(lds + EP_SCALE + 8), m, RLX, __HIP_MEMORY_SCOPE_WORKGROUP);
                        cnt = __hip_atomic_fetch_add((LAS unsigned*)(lds + EP_SCALE + 12), 1u, RLX, __HIP_MEMORY_SCOPE_WORKGROUP);
                    }
                    if (__builtin_amdgcn_readfirstlane(cnt) == 3) {
                        m = __hip_atomic_load((LAS unsigned*)(lds + EP_SCALE + 8), RLX, __HIP_MEMORY_SCOPE_WORKGROUP);
#pragma unroll
                        for (int i = 0; i < NREP / 8; ++i) __hip_atomic_store((u64*)(base + i * 1024), ((u64)call << 32) | m, RLX, SC_SYS);
                    }
                } else if (lane == 0) __hip_atomic_store(tmo, call, RLX, SC_AGENT);
            }
        } else {
            bool dead = false;
            {
                const unsigned nib = lane & 15; unsigned pos = lane >> 4, sel = 0;
#pragma unroll
                for (int e = 0; e < 4; ++e) { const bool k = (nib >> e) & 1; sel |= (k ? pos : 0x0cu) << (8 * e); pos += k; }
                *(LAS unsigned*)(lds + EP_LUT + wid * 512 + lane * 4) = sel;
            }
            unsigned n = 0;
            while (__hip_atomic_load((unsigned*)(inbox + IN_MF + (idx * 4 + w4) * 16), RLX, SC_SYS) < call) {
                if (((++n & 63u) == 0u && __hip_atomic_load(tmo, RLX, SC_AGENT) == call) || __builtin_amdgcn_s_memrealtime() - t0 > DEADLINE) { dead = true; break; }
                __builtin_amdgcn_s_sleep(32);
            }
            const u32x4 keep = ld16_sys(inbox + IN_MR + (size_t)((call & 1) * 128 + idx * 4 + w4) * 1024 + lane * 16);
            unsigned start[4], cum[4];
            stream_offsets(keep, start, cum);
            LAS char* const lb = lds + EP_RCV + w4 * RCVB;
            *(LAS u32x4*)(lb + lane * 32) = keep;
            *(LAS u32x4*)(lb + lane * 32 + 16) = (u32x4){start[0], start[1], start[2], start[3]};
            unsigned addr[32], sel[32];
#pragma unroll
            for (int gg = 0; gg < 4; ++gg) {
                const u32x4 kp = *(LAS u32x4*)(lb + ((lane >> 2) + 16 * gg) * 32), stt = *(LAS u32x4*)(lb + ((lane >> 2) + 16 * gg) * 32 + 16);
#pragma unroll
                for (int i = 0; i < 8; ++i) {
                    const unsigned sh = (i & 1) * 16 + (lane & 3) * 4;
                    const unsigned off = stt[i >> 1] + __builtin_popcount(kp[i >> 1] & ((1u << sh) - 1u)), nib = (kp[i >> 1] >> sh) & 15u;
                    sel[i * 4 + gg] = *(LAS unsigned*)(lds + EP_LUT + wid * 512 + (((off & 3u) << 4) | nib) * 4);
                    addr[i * 4 + gg] = EP_RCV + w4 * RCVB + (off & ~3u);
                }
            }
#pragma unroll
            for (int i = 0; i < 32; ++i) asm volatile("" : "+v"(addr[i]), "+v"(sel[i]));
            unsigned st;
            while ((st = __builtin_amdgcn_readfirstlane(__hip_atomic_load((LAS unsigned*)(lds + EP_SCALE + 4), RLX, __HIP_MEMORY_SCOPE_WORKGROUP))) == 0u) __builtin_amdgcn_s_sleep(8);
            const unsigned total = cum[3];
            const int sidx = (idx - 4 - 8 * w4) & 31;
            {
                const unsigned dt = ((unsigned)slot_ticks(sidx, total) + RWAIT) << 2;
                while (((unsigned)__builtin_amdgcn_s_memrealtime() << 2) - (st & ~3u) < dt) __builtin_amdgcn_s_sleep(2);
            }
            char* o = p.out + (size_t)(peer * MS + m0 + wm * 128 + (lane >> 2)) * NS + col_block(sidx >> 1) * BN + wn * 64 + (lane & 3) * 16;
            const __amdgpu_buffer_rsrc_t rr = __builtin_amdgcn_make_buffer_rsrc((void*)rs, 0, (total + 15u) & ~15u, 0x00020000);
            unsigned pos = 0;
            while (pos < total && !dead) {
                u32x4 v[5];
#pragma unroll
                for (int i = 0; i < 5; ++i) v[i] = __builtin_amdgcn_raw_buffer_load_b128(rr, pos + lane * 16 + i * 1024, 0, 17);
                unsigned adv = 5120;
#pragma unroll
                for (int i = 4; i >= 0; --i) {
                    const bool ok = pos + i * 1024 + lane * 16 >= total || ((((v[i][0] ^ tag) | (v[i][1] ^ tag) | (v[i][2] ^ tag) | (v[i][3] ^ tag)) & 0x80808080u) == 0u);
                    const u64 bad = ~__ballot(ok);
                    *(LAS u32x4*)(lb + pos + i * 1024 + lane * 16) = v[i] & 0x7f7f7f7fu;
                    if (bad) adv = i * 1024 + (unsigned)__builtin_ctzll(bad) * 16u;
                }
                pos += adv & ~127u;
                if (pos < total) {
                    if (((++n & 63u) == 0u && __hip_atomic_load(tmo, RLX, SC_AGENT) == call) || __builtin_amdgcn_s_memrealtime() - t0 > DEADLINE) dead = true;
                    __builtin_amdgcn_s_sleep(1);
                }
            }
            unsigned d[32];
#pragma unroll
            for (int i = 0; i < 32; ++i) {
                const LAS unsigned* sp = (const LAS unsigned*)(lds + addr[i]);
                d[i] = __builtin_amdgcn_perm(sp[1], sp[0], sel[i]);
            }
#pragma unroll
            for (int i = 0; i < 8; ++i) {
                u32x4 v = {d[i * 4], d[i * 4 + 1], d[i * 4 + 2], d[i * 4 + 3]};
                if (dead) v = (u32x4){0x80808080u, 0x80808080u, 0x80808080u, 0x80808080u};
                st16_out(o + (size_t)i * 16 * NS, v);
            }
            if (dead && lane == 0) __hip_atomic_store(tmo, call, RLX, SC_AGENT);
        }
    } else {
        const unsigned xa0 = XBASE + wm * 16384 + fr * 128 + ((g ^ (fr >> 1)) << 4);
        const int qq = fr >> 2, pp = lane & 3;
        const int f = qq | ((g & 1) << 2);
        const unsigned wa0 = WBASE + (8 * g + qq) * 256 + ((((wn * 8) + (pp >> 1)) ^ (f << 1)) << 4) + (pp & 1) * 8;
        const unsigned c0 = __hip_atomic_load((unsigned*)(myws + WS_CTR + b * 64), RLX, SC_AGENT);
        f32x4 acc[8][4];
#pragma unroll
        for (int i = 0; i < 8; ++i)
#pragma unroll
            for (int j = 0; j < 4; ++j) acc[i][j] = (f32x4){0.f, 0.f, 0.f, 0.f};
        bf16x8 XP[2][2], XQ[2][2], WA[4][2], WB[4][2];
#define PIN() __builtin_amdgcn_sched_barrier(0)
#define LGKM(n) do { __builtin_amdgcn_s_waitcnt(0xC07F | ((n) << 8)); PIN(); } while (0)
        auto rdX = [&](unsigned a, int off) { bf16x8 v = *(LAS bf16x8*)(lds + a + off); PIN(); return v; };
        auto rdW = [&](unsigned a, int off) { s16x4 v = __builtin_amdgcn_ds_read_tr16_b64_v4i16((LAS s16x4*)(lds + a + off)); PIN(); return v; };
        auto cat = [&](s16x4 lo, s16x4 hi) { return __builtin_bit_cast(bf16x8, __builtin_shufflevector(lo, hi, 0, 1, 2, 3, 4, 5, 6, 7)); };
        auto quarter = [&](int mf, bf16x8 (&X)[2][2], bf16x8 (&W)[4][2], auto fill) {
#pragma unroll
            for (int n = 0; n < 16; ++n) {
                const int i = n >> 3, j = (n >> 1) & 3, ks = n & 1;
                asm volatile("v_mfma_f32_16x16x32_bf16 %0, %1, %2, %0" : "+v"(acc[mf + i][j]) : "v"(W[j][ks]), "v"(X[i][ks])); PIN();
                fill(n);
            }
        };
        int xs = 0, ws = WSLOT;
        unsigned xc0 = xa0, xc1 = xa0 ^ 64, wq[4];
#pragma unroll
        for (int j = 0; j < 4; ++j) wq[j] = (wa0 ^ (j << 5)) + WSLOT;
        auto tile = [&](bf16x8 (&Wc)[4][2], bf16x8 (&Wn)[4][2]) {
            s16x4 lo;
            auto fx = [&](bf16x8 (&D)[2][2], int mf, int n) { if (n < 4) D[n >> 1][n & 1] = rdX((n & 1) ? xc1 : xc0, (mf + (n >> 1)) * 2048); };
            auto fw = [&](int f0, int cnt, int n) {
                const int m = n - 4;
                if (m >= 0 && m < 2 * cnt) {
                    const int f = f0 + (m >> 1), j = f >> 1, ks = f & 1;
                    if (!(m & 1)) lo = rdW(wq[j], ks * 8192); else Wn[j][ks] = cat(lo, rdW(wq[j], ks * 8192 + 1024));
                }
            };
            quarter(0, XP, Wc, [&](int n) { fx(XQ, 2, n); fw(0, 3, n); if (n == 14) LGKM(6); });
            quarter(2, XQ, Wc, [&](int n) { fx(XP, 4, n); fw(3, 3, n); if (n == 14) LGKM(6); });
            quarter(4, XP, Wc, [&](int n) {
                fx(XQ, 6, n); fw(6, 2, n);
                if (n == 9) { xs = (xs == NXS - 1) ? 0 : xs + 1; asm volatile("" : "+s"(xs)); PIN(); }
                if (n == 11) { xc0 = xa0 + xs * XSLOT; asm volatile("" : "+v"(xc0)); PIN(); }
                if (n == 12) { xc1 = (xa0 ^ 64) + xs * XSLOT; asm volatile("" : "+v"(xc1)); PIN(); }
                if (n == 14) LGKM(0);
            });
            BARRIER(); PIN();
            quarter(6, XQ, Wc, [&](int n) {
                fx(XP, 0, n);
                if (n == 5) { ws = (ws + WSLOT) & (NWS * WSLOT - 1); asm volatile("" : "+s"(ws)); PIN(); }
                if (n >= 7 && n < 11) { wq[n - 7] = (wa0 ^ ((n - 7) << 5)) + ws; asm volatile("" : "+v"(wq[n - 7])); PIN(); }
                if (n == 14) LGKM(0);
            });
        };
        const unsigned call = __builtin_amdgcn_readfirstlane(c0) + 1u;
#pragma unroll
        for (int i = 0; i < 8; ++i) asm volatile("" : "+v"(acc[i][0]), "+v"(acc[i][1]), "+v"(acc[i][2]), "+v"(acc[i][3]));
        LGKM(0);
        BARRIER(); PIN();
#pragma unroll
        for (int j = 0; j < 4; ++j)
#pragma unroll
            for (int ks = 0; ks < 2; ++ks) { const s16x4 lo = rdW(wa0 ^ (j << 5), ks * 8192); WA[j][ks] = cat(lo, rdW(wa0 ^ (j << 5), ks * 8192 + 1024)); }
#pragma unroll
        for (int n = 0; n < 4; ++n) XP[n >> 1][n & 1] = rdX((n & 1) ? xc1 : xc0, (n >> 1) * 2048);
        LGKM(0);
#pragma unroll 1
        for (int t = 0; t < NT; t += 2) { tile(WA, WB); tile(WB, WA); }
        asm volatile("s_nop 15" ::: "memory");
#pragma unroll
        for (int i = 0; i < 8; ++i) asm volatile("" : "+v"(acc[i][0]), "+v"(acc[i][1]), "+v"(acc[i][2]), "+v"(acc[i][3]));
        int mx = 0;
#pragma unroll
        for (int i = 0; i < 8; ++i)
#pragma unroll
            for (int j = 0; j < 4; ++j) {
                mx = max(max(mx, __float_as_int(acc[i][j][0])), __float_as_int(acc[i][j][1]));
                mx = max(max(mx, __float_as_int(acc[i][j][2])), __float_as_int(acc[i][j][3]));
            }
        unsigned wmax = wave_max_u((unsigned)mx);
        asm volatile("" : "+s"(wmax));
        BARRIER();
        if (lane == 0) {
            __hip_atomic_store((u64*)(myws + WS_LG) + b * 4 + wid, ((u64)call << 32) | wmax, RLX, SC_AGENT);
            if (wid == 0) __hip_atomic_store((unsigned*)(myws + WS_CTR + b * 64), call, RLX, SC_AGENT);
        }
        const u64 t0 = __builtin_amdgcn_s_memrealtime();
        auto wait_scale = [&](bool& dead) {
            LAS u64* sw = (LAS u64*)(lds + EP_SCALE);
            if (wid == 0) {
                unsigned gm, n = 0;
                for (;;) {
                    const u64 v = __hip_atomic_load((u64*)(myws + WS_GG + ((call & 1) * NREP + (idx & (NREP - 1))) * 128) + (lane & 7), RLX, SC_SYS);
                    gm = (unsigned)v;
                    if (__all((unsigned)(v >> 32) == call)) break;
                    if (((++n & 63u) == 0u && __hip_atomic_load(tmo, RLX, SC_AGENT) == call) || __builtin_amdgcn_s_memrealtime() - t0 > DEADLINE) { dead = true; break; }
                    __builtin_amdgcn_s_sleep(1);
                }
                if (dead && lane == 0) __hip_atomic_store(tmo, call, RLX, SC_AGENT);
                const float am = __uint_as_float(wave_max_u(gm));
                const float inv = am > 0.f ? 127.0f / am : 0.f;
                __hip_atomic_store(sw, ((u64)(((unsigned)__builtin_amdgcn_s_memrealtime() << 2) | (dead ? 2u : 1u)) << 32) | __float_as_uint(inv), RLX, __HIP_MEMORY_SCOPE_WORKGROUP);
                return inv;
            }
            u64 v;
            while (((v = __hip_atomic_load(sw, RLX, __HIP_MEMORY_SCOPE_WORKGROUP)) >> 32) == 0u) __builtin_amdgcn_s_sleep(1);
            dead = ((v >> 32) & 3u) == 2u;
            return __uint_as_float((unsigned)v);
        };
        bool dead = false;
        if (local) {
            const float inv = wait_scale(dead);
            char* o = p.out + (size_t)(p.rank * MS + m0 + wm * 128 + (lane >> 2)) * NS + cbl * BN + wn * 64 + (lane & 3) * 16;
            LAS char* const lb = lds + EP_STRM + wid * STRM;
#pragma unroll
            for (int i = 0; i < 8; ++i)
#pragma unroll
                for (int j = 0; j < 4; ++j) *(LAS unsigned*)(lb + i * 1024 + fr * 64 + ((j ^ (fr >> 2)) & 3) * 16 + g * 4) = q4(acc[i][j], inv);
            u32x4 v[8];
#pragma unroll
            for (int i = 0; i < 8; ++i) v[i] = *(LAS u32x4*)(lb + i * 1024 + (lane >> 2) * 64 + ((lane ^ (lane >> 4)) & 3) * 16);
#pragma unroll
            for (int i = 0; i < 8; ++i) st16_out(o + (size_t)i * 16 * NS, dead ? (u32x4){0x80808080u, 0x80808080u, 0x80808080u, 0x80808080u} : v[i]);
        } else {
            const int slot = ((idx + 4 + 8 * wid) & 31) * 4 + wid;
            char* const pws = p.ws[peer] + WS_IN + p.rank * INBOX;
            const unsigned tag = stream_tag(call);
            u32x4 keep;
#pragma unroll
            for (int k = 0; k < 4; ++k) {
                unsigned m = 0;
#pragma unroll
                for (int u = 7; u >= 0; --u)
#pragma unroll
                    for (int e = 3; e >= 0; --e) m = __builtin_amdgcn_alignbit(m, __float_as_uint(acc[2 * k + (u >> 2)][u & 3][e]), 31);
                keep[k] = ~m;
            }
            WAIT_V(0);
            st16_sys(pws + IN_MR + (size_t)((call & 1) * 128 + slot) * 1024 + lane * 16, keep);
            {
                const unsigned nib = lane & 15; unsigned pos = lane >> 4; u64 sel = 0x0c0c0c0c0c0c0c0cull;
#pragma unroll
                for (int e = 0; e < 4; ++e) { const bool k = (nib >> e) & 1; const u64 x = (u64)(0x0cu ^ (unsigned)e) << (8 * pos); sel ^= k ? x : 0ull; pos += k; }
                *(LAS u64*)(lds + EP_LUT + wid * 512 + lane * 8) = sel;
            }
#pragma unroll
            for (int i = 0; i < 9; ++i) *(LAS u32x4*)(lds + EP_STRM + wid * STRM + i * 1024 + lane * 16) = (u32x4){tag, tag, tag, tag};
            unsigned start[4], cum[4];
            stream_offsets(keep, start, cum);
            unsigned addr[32]; u32x2 sel[32];
#pragma unroll
            for (int k = 0; k < 4; ++k) {
                unsigned off = start[k];
#pragma unroll
                for (int u = 0; u < 8; ++u) {
                    const unsigned nib = (keep[k] >> (4 * u)) & 15u;
                    sel[k * 8 + u] = *(LAS u32x2*)(lds + EP_LUT + wid * 512 + (((off & 3u) << 4) | nib) * 8);
                    addr[k * 8 + u] = EP_STRM + wid * STRM + (off & ~3u);
                    off += __builtin_popcount(nib);
                }
            }
#pragma unroll
            for (int i = 0; i < 32; ++i) asm volatile("" : "+v"(addr[i]), "+v"(sel[i]));
            WAIT_V(0);
            if (lane == 0) __hip_atomic_store((unsigned*)(pws + IN_MF + slot * 16), call, RLX, SC_SYS);
            const float inv = wait_scale(dead);
            if (!dead) {
                char* const rs = pws + IN_RS + (size_t)slot * STRM + lane * 16;
                const u64 ta = __builtin_amdgcn_s_memrealtime();
                const u64 dt = slot_ticks(idx, cum[3]);
                unsigned sent = 0;
#pragma unroll
                for (int k = 0; k < 4; ++k) {
#pragma unroll
                    for (int u = 0; u < 8; ++u) {
                        const unsigned qd = q4(acc[2 * k + (u >> 2)][u & 3], inv);
                        LAS unsigned* d = (LAS unsigned*)(lds + addr[k * 8 + u]);
                        __hip_atomic_fetch_or(d, __builtin_amdgcn_perm(0u, qd, sel[k * 8 + u][0]), RLX, __HIP_MEMORY_SCOPE_WORKGROUP);
                        __hip_atomic_fetch_or(d + 1, __builtin_amdgcn_perm(0u, qd, sel[k * 8 + u][1]), RLX, __HIP_MEMORY_SCOPE_WORKGROUP);
                    }
                    asm volatile("" ::: "memory");
                    const unsigned lim = k < 3 ? (cum[k] & ~127u) : ((cum[3] + 63u) & ~63u);
                    if (k == 3) while (__builtin_amdgcn_s_memrealtime() - ta < dt) __builtin_amdgcn_s_sleep(1);
                    if (k == 0 || k == 3 || __builtin_amdgcn_s_memrealtime() - ta >= dt) {
                        while (sent < lim) {
                            if (sent + lane * 16 < lim) st16_sys(rs + sent, *(LAS u32x4*)(lds + EP_STRM + wid * STRM + sent + lane * 16));
                            sent = min(sent + 1024u, lim);
                        }
                    }
                }
            }
        }
    }
}

extern "C" void kernel_launch(int rank, int nranks, ncclComm_t comm, hipStream_t stream,
                              void* const* d_in, const int* in_sizes, int n_in,
                              void* d_out, int out_size, const int* in_dtypes, int out_dtype,
                              void* const* d_out_all, void* const* d_ws_all, size_t ws_bytes) {
    if (nranks != NR || (unsigned)rank >= (unsigned)NR || ws_bytes < WS_END) return;
    static bool init[NR];
    if (!init[rank]) {
        HIP_CHECK(hipFuncSetAttribute((const void*)fused_kernel, hipFuncAttributeMaxDynamicSharedMemorySize, LDS_BYTES));
        init[rank] = true;
    }
    Params p;
    p.x = (const u16*)d_in[0]; p.w = (const u16*)d_in[1]; p.out = (char*)d_out;
    for (int r = 0; r < NR; ++r) p.ws[r] = (char*)d_ws_all[r];
    p.rank = rank; p.pad = 0; p.myws = p.ws[rank];
    fused_kernel<<<NWG, 512, LDS_BYTES, stream>>>(p);
    HIP_CHECK(hipGetLastError());
}
```

```cpp
#include <hip/hip_runtime.h>
#include <rccl/rccl.h>
#include <cstdio>

#define HIP_CHECK(expr) do { hipError_t _e = (expr); if (_e != hipSuccess) { \
    fprintf(stderr, "HIP error %s at %s:%d\n", hipGetErrorString(_e), __FILE__, __LINE__); \
    return; } } while (0)

#define LAS __attribute__((address_space(3)))
typedef short s16x4 __attribute__((ext_vector_type(4)));
typedef __bf16 bf16x8 __attribute__((ext_vector_type(8)));
typedef float f32x4 __attribute__((ext_vector_type(4)));
typedef unsigned u32x4 __attribute__((ext_vector_type(4)));
typedef unsigned u32x2 __attribute__((ext_vector_type(2)));
typedef unsigned long long u64;
typedef unsigned short u16;

constexpr int NR = 8;
constexpr int MS = 512, NG = 16384, KD = 4096;
constexpr int NS = NG / NR;
constexpr int BM = 256, BN = 128, BK = 64, NT = KD / BK;
constexpr int NXS = 3, NWS = 4;
constexpr int XSLOT = BM * BK * 2, WSLOT = BK * BN * 2;
constexpr int XBASE = 0, WBASE = NXS * XSLOT;
constexpr int LDS_BYTES = NXS * XSLOT + NWS * WSLOT;
constexpr int NWG = 256;
constexpr int STRM = 8192 + 1024;
constexpr int EP_STRM = 0;
constexpr int RCVB = STRM;
constexpr int EP_RCV = 4 * STRM;
constexpr int EP_LUT = EP_RCV + 4 * RCVB;
constexpr int EP_SCALE = EP_LUT + 4096;

constexpr size_t WS_CTR = 0;
constexpr size_t WS_TMO = 16384;
constexpr size_t WS_LG = 20480;
constexpr int NREP = 16;
constexpr size_t WS_GG = 32768;
constexpr size_t WS_IN = 2u << 20, INBOX = 2u << 20;
constexpr size_t IN_MF = 0;
constexpr size_t IN_MR = 4096;
constexpr size_t IN_RS = IN_MR + 2 * 128 * 1024;
static_assert(IN_RS + 128 * (size_t)STRM <= INBOX, "inbox");
constexpr size_t WS_END = WS_IN + NR * INBOX;
constexpr u64 DEADLINE = 100000000ull;
constexpr int WAUX = 2;
constexpr int NAGG = 2;
constexpr unsigned LINK_BPT = 590;
constexpr unsigned SLOT0 = 2, RWAIT = 60;
constexpr unsigned PACE_T0 = 260, PACE_TAU = 72, PACE_MIN = 132;

struct Params {
    const u16* x; const u16* w;
    char* out; char* myws;
    int rank; int pad;
    char* ws[NR];
};

struct LdsBase { __device__ __forceinline__ LAS char* operator+(unsigned a) const { return (LAS char*)(__UINTPTR_TYPE__)a; } };

#define WAIT_V(n) asm volatile("s_waitcnt vmcnt(" #n ")" ::: "memory")
#define BARRIER() asm volatile("s_barrier" ::: "memory")
#define RLX __ATOMIC_RELAXED
#define SC_AGENT __HIP_MEMORY_SCOPE_AGENT
#define SC_SYS __HIP_MEMORY_SCOPE_SYSTEM

template <int AUX> __device__ __forceinline__ void dma16(__amdgpu_buffer_rsrc_t rsrc, LAS char* dst, unsigned voff, unsigned soff) {
    __builtin_amdgcn_raw_ptr_buffer_load_lds(rsrc, (LAS void*)dst, 16, voff, soff, 0, AUX);
}
__device__ __forceinline__ void st16_sys(char* p, u32x4 v) {
    asm volatile("global_store_dwordx4 %0, %1, off sc0 sc1\n\ts_nop 1" :: "v"(p), "v"(v) : "memory");
}
__device__ __forceinline__ void st16_out(char* p, u32x4 v) {
    asm volatile("global_store_dwordx4 %0, %1, off sc1\n\ts_nop 1" :: "v"(p), "v"(v) : "memory");
}
__device__ __forceinline__ u32x4 ld16_sys(const char* p) {
    u32x4 v; asm volatile("global_load_dwordx4 %0, %1, off sc0 sc1\n\ts_waitcnt vmcnt(0)" : "=&v"(v) : "v"(p) : "memory"); return v;
}
__device__ __forceinline__ unsigned q4(f32x4 v, float inv) {
    unsigned r = 0;
    r = __builtin_amdgcn_cvt_pk_u8_f32(v[0] * inv, 0, r);
    r = __builtin_amdgcn_cvt_pk_u8_f32(v[1] * inv, 1, r);
    r = __builtin_amdgcn_cvt_pk_u8_f32(v[2] * inv, 2, r);
    r = __builtin_amdgcn_cvt_pk_u8_f32(v[3] * inv, 3, r);
    const unsigned h = r & 0x80808080u;
    return (r | (h - (h >> 7))) & 0x7f7f7f7fu;
}
template <int CTRL, int RMASK> __device__ __forceinline__ unsigned dpp0(unsigned x) { return (unsigned)__builtin_amdgcn_update_dpp(0, (int)x, CTRL, RMASK, 0xf, true); }
__device__ __forceinline__ unsigned wave_scan_add(unsigned x) {
    x += dpp0<0x111, 0xf>(x); x += dpp0<0x112, 0xf>(x); x += dpp0<0x114, 0xf>(x); x += dpp0<0x118, 0xf>(x);
    x += dpp0<0x142, 0xa>(x); x += dpp0<0x143, 0xc>(x); return x;
}
__device__ __forceinline__ unsigned wave_max_u(unsigned x) {
    x = max(x, dpp0<0x111, 0xf>(x)); x = max(x, dpp0<0x112, 0xf>(x)); x = max(x, dpp0<0x114, 0xf>(x)); x = max(x, dpp0<0x118, 0xf>(x));
    x = max(x, dpp0<0x142, 0xa>(x)); x = max(x, dpp0<0x143, 0xc>(x)); return (unsigned)__builtin_amdgcn_readlane((int)x, 63);
}
__device__ __forceinline__ void stream_offsets(const u32x4 keep, unsigned (&start)[4], unsigned (&cum)[4]) {
    const unsigned c0 = __builtin_popcount(keep[0]), c1 = __builtin_popcount(keep[1]), c2 = __builtin_popcount(keep[2]), c3 = __builtin_popcount(keep[3]);
    const unsigned a = wave_scan_add(c0 | (c1 << 16)), b = wave_scan_add(c2 | (c3 << 16));
    const unsigned ta = (unsigned)__builtin_amdgcn_readlane((int)a, 63), tb = (unsigned)__builtin_amdgcn_readlane((int)b, 63);
    cum[0] = ta & 0xffffu; cum[1] = cum[0] + (ta >> 16); cum[2] = cum[1] + (tb & 0xffffu); cum[3] = cum[2] + (tb >> 16);
    start[0] = (a & 0xffffu) - c0; start[1] = cum[0] + (a >> 16) - c1; start[2] = cum[1] + (b & 0xffffu) - c2; start[3] = cum[2] + (b >> 16) - c3;
}
__device__ __forceinline__ unsigned stream_tag(unsigned call) { const unsigned n = call % 3u + 1u; return ((n & 1u) << 7) | ((n & 2u) << 14); }
__device__ __forceinline__ u32x4 granule_pack(u32x4 d, unsigned w) {
    const unsigned x = (w | (w << 8)) & 0x00ff00ffu, u = (x | (x << 4)) & 0x0f0f0f0fu;
    return (u32x4){d[0] | ((u << 7) & 0x80808080u), d[1] | ((u << 6) & 0x80808080u), d[2] | ((u << 5) & 0x80808080u), d[3] | ((u << 4) & 0x80808080u)};
}
__device__ __forceinline__ unsigned granule_word(u32x4 d) {
    const unsigned u = ((d[0] >> 7) & 0x01010101u) | ((d[1] >> 6) & 0x02020202u) | ((d[2] >> 5) & 0x04040404u) | ((d[3] >> 4) & 0x08080808u);
    const unsigned x = (u | (u >> 4)) & 0x00ff00ffu;
    return (x | (x >> 8)) & 0xffffu;
}
__device__ __forceinline__ void wire_split(const unsigned (&cum)[4], unsigned& head, unsigned& wire) {
    head = cum[0] & ~127u;
    wire = head + (cum[3] - head + 17u) / 18u * 16u;
}
__device__ __forceinline__ u64 slot_ticks(int pos, unsigned bytes) {
    return (u64)(((unsigned)(3 * (32 - SLOT0) + 8 * pos) * bytes) / (3 * LINK_BPT));
}
__global__ __launch_bounds__(512, 2) void fused_kernel(const Params p) {
    extern __shared__ __attribute__((aligned(16))) char smem[];
    asm volatile("" :: "s"((LAS char*)smem));
    const LdsBase lds;
    asm volatile("" :: "s"(p.x), "s"(p.w), "s"(p.myws), "s"(p.rank));
    const int tid = threadIdx.x, lane = tid & 63;
    const int wid = __builtin_amdgcn_readfirstlane(tid >> 6);
    const int b = blockIdx.x, peer = b & 7, idx = b >> 3;
    auto col_block = [](int i) { return (int)((0xd951fecba8764320ull >> (4 * i)) & 15u); };
    const int cbl = col_block(idx >> 1), mh = idx & 1;
    const int m0 = mh * BM, n0 = (peer * 16 + cbl) * BN;
    const bool local = peer == p.rank;
    char* const myws = p.myws;
    unsigned* const tmo = (unsigned*)(myws + WS_TMO);
    const int fr = lane & 15, g = lane >> 4;
    const int w4 = wid & 3, wm = w4 >> 1, wn = w4 & 1;

    if (__builtin_expect(wid >= 4, 1)) {
        const int lw = w4;
        const u64 tstart = __builtin_amdgcn_s_memrealtime();
        __amdgpu_buffer_rsrc_t rx = __builtin_amdgcn_make_buffer_rsrc((void*)p.x, 0, MS * KD * 2, 0x00020000);
        __amdgpu_buffer_rsrc_t rw = __builtin_amdgcn_make_buffer_rsrc((void*)p.w, 0, (unsigned)((size_t)KD * NG * 2), 0x00020000);
        const unsigned xrow = m0 + lw * 64 + (lane >> 3);
        const unsigned xv0 = xrow * (KD * 2) + (((lane & 7) ^ (lane >> 4)) << 4);
        const unsigned xv1 = xrow * (KD * 2) + (((lane & 7) ^ (4 + (lane >> 4))) << 4);
        const unsigned wv = (lw * 4 + (lane >> 4)) * (NG * 2) + n0 * 2 + (((lane & 15) ^ (((lane >> 4) | ((lw & 2) << 1)) << 1)) << 4);
        auto issueX = [&](int t, int slot) {
            LAS char* d = lds + XBASE + slot * XSLOT + lw * 64 * 128;
#pragma unroll
            for (int i = 0; i < 8; ++i) dma16<0>(rx, d + i * 1024, (i & 1) ? xv1 : xv0, (unsigned)(t * 128 + i * 8 * KD * 2));
        };
        auto issueW = [&](int t) {
#pragma unroll
            for (int i = 0; i < 4; ++i) dma16<WAUX>(rw, lds + WBASE + (t & 3) * WSLOT + i * 4096 + lw * 1024, wv, (unsigned)(t * 64 + i * 16) * (NG * 2));
        };
        auto issueWs = [&](int t) {
#pragma unroll
            for (int i = 0; i < 4; ++i) dma16<0>(rw, lds + WBASE + (t & 3) * WSLOT + i * 4096 + lw * 1024, wv, (unsigned)(t * 64 + i * 16) * (NG * 2));
        };
        unsigned touch = 0;
        const unsigned tv = b * 16384 + (lw >> 1) * 8192 + ((lw & 1) * 8 + (lane & 7)) * 128;
        auto touchX = [&](unsigned kt0) { asm volatile("buffer_load_dword %0, %1, %2, %3 offen" : "+v"(touch) : "v"(tv), "s"(rx), "s"(kt0 * 128u) : "memory"); };
        int xs = 1;
        auto nextX = [&](int t) { issueX(t, xs); xs = (xs == NXS - 1) ? 0 : xs + 1; };
        issueW(0); issueX(0, 0); issueW(1); issueX(1, 1);
        WAIT_V(8);
        BARRIER();
        unsigned cv;
        issueW(2); touchX(3);
        asm volatile("global_load_dword %0, %1, off sc1" : "=v"(cv) : "v"(myws + WS_CTR + b * 64) : "memory");
        issueX(2, 2); issueW(3);
        WAIT_V(14); BARRIER();
        issueX(3, 0); issueW(4); touchX(19); touchX(35); touchX(48);
        asm volatile("s_waitcnt vmcnt(15)" : "+v"(cv) :: "memory");
        BARRIER();
        const unsigned call = __builtin_amdgcn_readfirstlane(cv) + 1u;
        {
            const char* wp;
            if (lw == 0) wp = local ? p.ws[idx & 7] + WS_GG : p.ws[peer] + WS_IN + p.rank * INBOX + (idx & 1) * (INBOX - 64);
            else if (lw == 1) wp = (idx & 2) ? myws + WS_IN + peer * INBOX + (idx & 1) * (INBOX - 64) : p.out + (size_t)(peer * MS + (idx & 1) * (MS - 1)) * NS;
            else wp = (const char*)p.w + (size_t)(idx * 2 + (lw & 1)) * (BK * NG * 2) + 64 * 1024;
            asm volatile("s_load_dword s95, %0, 0x0" :: "s"(wp) : "s95");
        }
        u64 tlast = 0;
        auto pace = [&](int n) {
            if (lw == 1) {
                const u64 a = tstart + PACE_T0 + (unsigned)n * PACE_TAU, c = tlast + PACE_MIN, tgt = a > c ? a : c;
                while (__builtin_amdgcn_s_memrealtime() < tgt) __builtin_amdgcn_s_sleep(1);
            }
        };
        int t = 2;
        const unsigned wl = ((unsigned)(__UINTPTR_TYPE__)p.w + (unsigned)n0 * 2u) >> 7;
        const bool slow = ((wl - 2u) & 7u) < 2u || ((wl - 62u) & 127u) < 2u;
        const bool shift = slow ? mh : (((unsigned)(__UINTPTR_TYPE__)p.x >> 7) & 1u);
        if (shift) { nextX(4); issueW(5); WAIT_V(12); BARRIER(); t = 3; }
        if (slow) {
            for (; t < NT - 4; t += 2) {
                nextX(t + 2); issueWs(t + 3); issueWs(t + 4);
                WAIT_V(16); BARRIER();
                nextX(t + 3);
                pace(t + 2);
                WAIT_V(12); BARRIER();
                tlast = __builtin_amdgcn_s_memrealtime();
            }
        } else {
            for (; t < NT - 4; t += 2) {
                nextX(t + 2); issueW(t + 3); issueW(t + 4);
                WAIT_V(16); BARRIER();
                nextX(t + 3);
                pace(t + 2);
                WAIT_V(12); BARRIER();
                tlast = __builtin_amdgcn_s_memrealtime();
            }
        }
        if (!shift) { nextX(NT - 2); issueW(NT - 1); WAIT_V(12); BARRIER(); }
        nextX(NT - 1);
        char* const inbox = myws + WS_IN + peer * INBOX;
        char* const rs = inbox + IN_RS + (size_t)(idx * 4 + w4) * STRM;
#pragma unroll
        for (int i = 0; i < 4; ++i) st16_sys(rs + (call & 1u) * 4096 + i * 1024 + lane * 16, (u32x4){0u, 0u, 0u, 0u});
        WAIT_V(12); BARRIER();
        WAIT_V(4); BARRIER();
        BARRIER();
        if (wid == 4) asm volatile("ds_write_b128 %0, %1\n\ts_waitcnt lgkmcnt(0)" :: "v"((unsigned)EP_SCALE), "v"((u32x4){0u, 0u, 0u, 0u}) : "memory");
        WAIT_V(0);
        asm volatile("s_waitcnt lgkmcnt(0)" :: "v"(touch) : "s95");
        BARRIER();
        const unsigned tag = stream_tag(call);
        const u64 t0 = __builtin_amdgcn_s_memrealtime();
        if (local) {
            if (idx < NAGG) {
                bool dead = false, ok = false; unsigned m = 0;
                const char* gp = myws + WS_LG + w4 * 2048 + lane * 16;
                char* base = p.ws[0];
#pragma unroll
                for (int r = 1; r < NR; ++r) base = (lane & 7) == r ? p.ws[r] : base;
                base += WS_GG + ((call & 1) * NREP + (lane >> 3)) * 128 + p.rank * 8;
                asm volatile("" : "+v"(base));
                for (;;) {
                    if (!ok) {
                        u32x4 v0, v1;
                        asm volatile("global_load_dwordx4 %0, %2, off sc1\n\tglobal_load_dwordx4 %1, %2, off offset:1024 sc1\n\ts_waitcnt vmcnt(0)" : "=&v"(v0), "=&v"(v1) : "v"(gp) : "memory");
                        m = max(max(v0[0], v0[2]), max(v1[0], v1[2]));
                        ok = v0[1] == call && v0[3] == call && v1[1] == call && v1[3] == call;
                    }
                    if (__all(ok)) break;
                    if (__builtin_amdgcn_s_memrealtime() - t0 > DEADLINE) { dead = true; break; }
                    __builtin_amdgcn_s_sleep(1);
                }
                if (!dead) {
                    m = wave_max_u(m);
                    unsigned cnt = 0;
                    if (lane == 0) {
                        __hip_atomic_fetch_max((LAS unsigned*)(lds + EP_SCALE + 8), m, RLX, __HIP_MEMORY_SCOPE_WORKGROUP);
                        cnt = __hip_atomic_fetch_add((LAS unsigned*)(lds + EP_SCALE + 12), 1u, RLX, __HIP_MEMORY_SCOPE_WORKGROUP);
                    }
                    if (__builtin_amdgcn_readfirstlane(cnt) == 3) {
                        m = __hip_atomic_load((LAS unsigned*)(lds + EP_SCALE + 8), RLX, __HIP_MEMORY_SCOPE_WORKGROUP);
#pragma unroll
                        for (int i = 0; i < NREP / 8; ++i) __hip_atomic_store((u64*)(base + i * 1024), ((u64)call << 32) | m, RLX, SC_SYS);
                    }
                } else if (lane == 0) __hip_atomic_store(tmo, call, RLX, SC_AGENT);
            }
        } else {
            bool dead = false;
            {
                const unsigned nib = lane & 15; unsigned pos = lane >> 4, sel = 0;
#pragma unroll
                for (int e = 0; e < 4; ++e) { const bool k = (nib >> e) & 1; sel |= (k ? pos : 0x0cu) << (8 * e); pos += k; }
                *(LAS unsigned*)(lds + EP_LUT + wid * 512 + lane * 4) = sel;
            }
            unsigned n = 0;
            while (__hip_atomic_load((unsigned*)(inbox + IN_MF + (idx * 4 + w4) * 16), RLX, SC_SYS) < call) {
                if (((++n & 63u) == 0u && __hip_atomic_load(tmo, RLX, SC_AGENT) == call) || __builtin_amdgcn_s_memrealtime() - t0 > DEADLINE) { dead = true; break; }
                __builtin_amdgcn_s_sleep(32);
            }
            const u32x4 keep = ld16_sys(inbox + IN_MR + (size_t)((call & 1) * 128 + idx * 4 + w4) * 1024 + lane * 16);
            unsigned start[4], cum[4];
            stream_offsets(keep, start, cum);
            LAS char* const lb = lds + EP_RCV + w4 * RCVB;
            *(LAS u32x4*)(lb + lane * 32) = keep;
            *(LAS u32x4*)(lb + lane * 32 + 16) = (u32x4){start[0], start[1], start[2], start[3]};
            unsigned addr[32], sel[32];
#pragma unroll
            for (int gg = 0; gg < 4; ++gg) {
                const u32x4 kp = *(LAS u32x4*)(lb + ((lane >> 2) + 16 * gg) * 32), stt = *(LAS u32x4*)(lb + ((lane >> 2) + 16 * gg) * 32 + 16);
#pragma unroll
                for (int i = 0; i < 8; ++i) {
                    const unsigned sh = (i & 1) * 16 + (lane & 3) * 4;
                    const unsigned off = stt[i >> 1] + __builtin_popcount(kp[i >> 1] & ((1u << sh) - 1u)), nib = (kp[i >> 1] >> sh) & 15u;
                    sel[i * 4 + gg] = *(LAS unsigned*)(lds + EP_LUT + wid * 512 + (((off & 3u) << 4) | nib) * 4);
                    addr[i * 4 + gg] = EP_RCV + w4 * RCVB + (off & ~3u);
                }
            }
#pragma unroll
            for (int i = 0; i < 32; ++i) asm volatile("" : "+v"(addr[i]), "+v"(sel[i]));
            unsigned st;
            while ((st = __builtin_amdgcn_readfirstlane(__hip_atomic_load((LAS unsigned*)(lds + EP_SCALE + 4), RLX, __HIP_MEMORY_SCOPE_WORKGROUP))) == 0u) __builtin_amdgcn_s_sleep(8);
            const unsigned total = cum[3];
            const int sidx = (idx - 4 - 8 * w4) & 31;
            {
                const unsigned dt = ((unsigned)slot_ticks(sidx, total) + RWAIT) << 2;
                while (((unsigned)__builtin_amdgcn_s_memrealtime() << 2) - (st & ~3u) < dt) __builtin_amdgcn_s_sleep(2);
            }
            char* o = p.out + (size_t)(peer * MS + m0 + wm * 128 + (lane >> 2)) * NS + col_block(sidx >> 1) * BN + wn * 64 + (lane & 3) * 16;
            unsigned head, wire;
            wire_split(cum, head, wire);
            const __amdgpu_buffer_rsrc_t rr = __builtin_amdgcn_make_buffer_rsrc((void*)rs, 0, wire, 0x00020000);
            unsigned pos = 0;
            while (pos < wire && !dead) {
                u32x4 v[4];
#pragma unroll
                for (int i = 0; i < 4; ++i) v[i] = __builtin_amdgcn_raw_buffer_load_b128(rr, pos + lane * 16 + i * 1024, 0, 17);
                unsigned adv = 4096;
#pragma unroll
                for (int i = 3; i >= 0; --i) {
                    const unsigned off = pos + i * 1024 + lane * 16, w = granule_word(v[i]);
                    const bool in = off < wire;
                    const u64 bad = __ballot(in && (w & 0x8080u) != tag);
                    if (in) {
                        *(LAS u32x4*)(lb + off) = v[i] & 0x7f7f7f7fu;
                        if (off >= head) *(LAS u16*)(lb + wire + ((off - head) >> 3)) = (u16)(w & 0x7f7fu);
                    }
                    if (bad) adv = i * 1024 + (unsigned)__builtin_ctzll(bad) * 16u;
                }
                pos += adv & ~127u;
                if (pos < wire) {
                    if (((++n & 63u) == 0u && __hip_atomic_load(tmo, RLX, SC_AGENT) == call) || __builtin_amdgcn_s_memrealtime() - t0 > DEADLINE) dead = true;
                    __builtin_amdgcn_s_sleep(1);
                }
            }
            unsigned d[32];
#pragma unroll
            for (int i = 0; i < 32; ++i) {
                const LAS unsigned* sp = (const LAS unsigned*)(lds + addr[i]);
                d[i] = __builtin_amdgcn_perm(sp[1], sp[0], sel[i]);
            }
#pragma unroll
            for (int i = 0; i < 8; ++i) {
                u32x4 v = {d[i * 4], d[i * 4 + 1], d[i * 4 + 2], d[i * 4 + 3]};
                if (dead) v = (u32x4){0x80808080u, 0x80808080u, 0x80808080u, 0x80808080u};
                st16_out(o + (size_t)i * 16 * NS, v);
            }
            if (dead && lane == 0) __hip_atomic_store(tmo, call, RLX, SC_AGENT);
        }
    } else {
        const unsigned xa0 = XBASE + wm * 16384 + fr * 128 + ((g ^ (fr >> 1)) << 4);
        const int qq = fr >> 2, pp = lane & 3;
        const int f = qq | ((g & 1) << 2);
        const unsigned wa0 = WBASE + (8 * g + qq) * 256 + ((((wn * 8) + (pp >> 1)) ^ (f << 1)) << 4) + (pp & 1) * 8;
        const unsigned c0 = __hip_atomic_load((unsigned*)(myws + WS_CTR + b * 64), RLX, SC_AGENT);
        f32x4 acc[8][4];
#pragma unroll
        for (int i = 0; i < 8; ++i)
#pragma unroll
            for (int j = 0; j < 4; ++j) acc[i][j] = (f32x4){0.f, 0.f, 0.f, 0.f};
        bf16x8 XP[2][2], XQ[2][2], WA[4][2], WB[4][2];
#define PIN() __builtin_amdgcn_sched_barrier(0)
#define LGKM(n) do { __builtin_amdgcn_s_waitcnt(0xC07F | ((n) << 8)); PIN(); } while (0)
        auto rdX = [&](unsigned a, int off) { bf16x8 v = *(LAS bf16x8*)(lds + a + off); PIN(); return v; };
        auto rdW = [&](unsigned a, int off) { s16x4 v = __builtin_amdgcn_ds_read_tr16_b64_v4i16((LAS s16x4*)(lds + a + off)); PIN(); return v; };
        auto cat = [&](s16x4 lo, s16x4 hi) { return __builtin_bit_cast(bf16x8, __builtin_shufflevector(lo, hi, 0, 1, 2, 3, 4, 5, 6, 7)); };
        auto quarter = [&](int mf, bf16x8 (&X)[2][2], bf16x8 (&W)[4][2], auto fill) {
#pragma unroll
            for (int n = 0; n < 16; ++n) {
                const int i = n >> 3, j = (n >> 1) & 3, ks = n & 1;
                asm volatile("v_mfma_f32_16x16x32_bf16 %0, %1, %2, %0" : "+v"(acc[mf + i][j]) : "v"(W[j][ks]), "v"(X[i][ks])); PIN();
                fill(n);
            }
        };
        int xs = 0, ws = WSLOT;
        unsigned xc0 = xa0, xc1 = xa0 ^ 64, wq[4];
#pragma unroll
        for (int j = 0; j < 4; ++j) wq[j] = (wa0 ^ (j << 5)) + WSLOT;
        auto tile = [&](bf16x8 (&Wc)[4][2], bf16x8 (&Wn)[4][2]) {
            s16x4 lo;
            auto fx = [&](bf16x8 (&D)[2][2], int mf, int n) { if (n < 4) D[n >> 1][n & 1] = rdX((n & 1) ? xc1 : xc0, (mf + (n >> 1)) * 2048); };
            auto fw = [&](int f0, int cnt, int n) {
                const int m = n - 4;
                if (m >= 0 && m < 2 * cnt) {
                    const int f = f0 + (m >> 1), j = f >> 1, ks = f & 1;
                    if (!(m & 1)) lo = rdW(wq[j], ks * 8192); else Wn[j][ks] = cat(lo, rdW(wq[j], ks * 8192 + 1024));
                }
            };
            quarter(0, XP, Wc, [&](int n) { fx(XQ, 2, n); fw(0, 3, n); if (n == 14) LGKM(6); });
            quarter(2, XQ, Wc, [&](int n) { fx(XP, 4, n); fw(3, 3, n); if (n == 14) LGKM(6); });
            quarter(4, XP, Wc, [&](int n) {
                fx(XQ, 6, n); fw(6, 2, n);
                if (n == 9) { xs = (xs == NXS - 1) ? 0 : xs + 1; asm volatile("" : "+s"(xs)); PIN(); }
                if (n == 11) { xc0 = xa0 + xs * XSLOT; asm volatile("" : "+v"(xc0)); PIN(); }
                if (n == 12) { xc1 = (xa0 ^ 64) + xs * XSLOT; asm volatile("" : "+v"(xc1)); PIN(); }
                if (n == 14) LGKM(0);
            });
            BARRIER(); PIN();
            quarter(6, XQ, Wc, [&](int n) {
                fx(XP, 0, n);
                if (n == 5) { ws = (ws + WSLOT) & (NWS * WSLOT - 1); asm volatile("" : "+s"(ws)); PIN(); }
                if (n >= 7 && n < 11) { wq[n - 7] = (wa0 ^ ((n - 7) << 5)) + ws; asm volatile("" : "+v"(wq[n - 7])); PIN(); }
                if (n == 14) LGKM(0);
            });
        };
        const unsigned call = __builtin_amdgcn_readfirstlane(c0) + 1u;
#pragma unroll
        for (int i = 0; i < 8; ++i) asm volatile("" : "+v"(acc[i][0]), "+v"(acc[i][1]), "+v"(acc[i][2]), "+v"(acc[i][3]));
        LGKM(0);
        BARRIER(); PIN();
#pragma unroll
        for (int j = 0; j < 4; ++j)
#pragma unroll
            for (int ks = 0; ks < 2; ++ks) { const s16x4 lo = rdW(wa0 ^ (j << 5), ks * 8192); WA[j][ks] = cat(lo, rdW(wa0 ^ (j << 5), ks * 8192 + 1024)); }
#pragma unroll
        for (int n = 0; n < 4; ++n) XP[n >> 1][n & 1] = rdX((n & 1) ? xc1 : xc0, (n >> 1) * 2048);
        LGKM(0);
#pragma unroll 1
        for (int t = 0; t < NT; t += 2) { tile(WA, WB); tile(WB, WA); }
        asm volatile("s_nop 15" ::: "memory");
#pragma unroll
        for (int i = 0; i < 8; ++i) asm volatile("" : "+v"(acc[i][0]), "+v"(acc[i][1]), "+v"(acc[i][2]), "+v"(acc[i][3]));
        int mx = 0;
#pragma unroll
        for (int i = 0; i < 8; ++i)
#pragma unroll
            for (int j = 0; j < 4; ++j) {
                mx = max(max(mx, __float_as_int(acc[i][j][0])), __float_as_int(acc[i][j][1]));
                mx = max(max(mx, __float_as_int(acc[i][j][2])), __float_as_int(acc[i][j][3]));
            }
        unsigned wmax = wave_max_u((unsigned)mx);
        asm volatile("" : "+s"(wmax));
        BARRIER();
        if (lane == 0) {
            __hip_atomic_store((u64*)(myws + WS_LG) + b * 4 + wid, ((u64)call << 32) | wmax, RLX, SC_AGENT);
            if (wid == 0) __hip_atomic_store((unsigned*)(myws + WS_CTR + b * 64), call, RLX, SC_AGENT);
        }
        const u64 t0 = __builtin_amdgcn_s_memrealtime();
        auto wait_scale = [&](bool& dead) {
            LAS u64* sw = (LAS u64*)(lds + EP_SCALE);
            if (wid == 0) {
                unsigned gm, n = 0;
                for (;;) {
                    const u64 v = __hip_atomic_load((u64*)(myws + WS_GG + ((call & 1) * NREP + (idx & (NREP - 1))) * 128) + (lane & 7), RLX, SC_SYS);
                    gm = (unsigned)v;
                    if (__all((unsigned)(v >> 32) == call)) break;
                    if (((++n & 63u) == 0u && __hip_atomic_load(tmo, RLX, SC_AGENT) == call) || __builtin_amdgcn_s_memrealtime() - t0 > DEADLINE) { dead = true; break; }
                    __builtin_amdgcn_s_sleep(1);
                }
                if (dead && lane == 0) __hip_atomic_store(tmo, call, RLX, SC_AGENT);
                const float am = __uint_as_float(wave_max_u(gm));
                const float inv = am > 0.f ? 127.0f / am : 0.f;
                __hip_atomic_store(sw, ((u64)(((unsigned)__builtin_amdgcn_s_memrealtime() << 2) | (dead ? 2u : 1u)) << 32) | __float_as_uint(inv), RLX, __HIP_MEMORY_SCOPE_WORKGROUP);
                return inv;
            }
            u64 v;
            while (((v = __hip_atomic_load(sw, RLX, __HIP_MEMORY_SCOPE_WORKGROUP)) >> 32) == 0u) __builtin_amdgcn_s_sleep(1);
            dead = ((v >> 32) & 3u) == 2u;
            return __uint_as_float((unsigned)v);
        };
        bool dead = false;
        if (local) {
            const float inv = wait_scale(dead);
            char* o = p.out + (size_t)(p.rank * MS + m0 + wm * 128 + (lane >> 2)) * NS + cbl * BN + wn * 64 + (lane & 3) * 16;
            LAS char* const lb = lds + EP_STRM + wid * STRM;
#pragma unroll
            for (int i = 0; i < 8; ++i)
#pragma unroll
                for (int j = 0; j < 4; ++j) *(LAS unsigned*)(lb + i * 1024 + fr * 64 + ((j ^ (fr >> 2)) & 3) * 16 + g * 4) = q4(acc[i][j], inv);
            u32x4 v[8];
#pragma unroll
            for (int i = 0; i < 8; ++i) v[i] = *(LAS u32x4*)(lb + i * 1024 + (lane >> 2) * 64 + ((lane ^ (lane >> 4)) & 3) * 16);
#pragma unroll
            for (int i = 0; i < 8; ++i) st16_out(o + (size_t)i * 16 * NS, dead ? (u32x4){0x80808080u, 0x80808080u, 0x80808080u, 0x80808080u} : v[i]);
        } else {
            const int slot = ((idx + 4 + 8 * wid) & 31) * 4 + wid;
            char* const pws = p.ws[peer] + WS_IN + p.rank * INBOX;
            const unsigned tag = stream_tag(call);
            u32x4 keep;
#pragma unroll
            for (int k = 0; k < 4; ++k) {
                unsigned m = 0;
#pragma unroll
                for (int u = 7; u >= 0; --u)
#pragma unroll
                    for (int e = 3; e >= 0; --e) m = __builtin_amdgcn_alignbit(m, __float_as_uint(acc[2 * k + (u >> 2)][u & 3][e]), 31);
                keep[k] = ~m;
            }
            WAIT_V(0);
            st16_sys(pws + IN_MR + (size_t)((call & 1) * 128 + slot) * 1024 + lane * 16, keep);
            {
                const unsigned nib = lane & 15; unsigned pos = lane >> 4; u64 sel = 0x0c0c0c0c0c0c0c0cull;
#pragma unroll
                for (int e = 0; e < 4; ++e) { const bool k = (nib >> e) & 1; const u64 x = (u64)(0x0cu ^ (unsigned)e) << (8 * pos); sel ^= k ? x : 0ull; pos += k; }
                *(LAS u64*)(lds + EP_LUT + wid * 512 + lane * 8) = sel;
            }
#pragma unroll
            for (int i = 0; i < 9; ++i) *(LAS u32x4*)(lds + EP_STRM + wid * STRM + i * 1024 + lane * 16) = (u32x4){0u, 0u, 0u, 0u};
            unsigned start[4], cum[4];
            stream_offsets(keep, start, cum);
            unsigned addr[32]; u32x2 sel[32];
#pragma unroll
            for (int k = 0; k < 4; ++k) {
                unsigned off = start[k];
#pragma unroll
                for (int u = 0; u < 8; ++u) {
                    const unsigned nib = (keep[k] >> (4 * u)) & 15u;
                    sel[k * 8 + u] = *(LAS u32x2*)(lds + EP_LUT + wid * 512 + (((off & 3u) << 4) | nib) * 8);
                    addr[k * 8 + u] = EP_STRM + wid * STRM + (off & ~3u);
                    off += __builtin_popcount(nib);
                }
            }
#pragma unroll
            for (int i = 0; i < 32; ++i) asm volatile("" : "+v"(addr[i]), "+v"(sel[i]));
            WAIT_V(0);
            if (lane == 0) __hip_atomic_store((unsigned*)(pws + IN_MF + slot * 16), call, RLX, SC_SYS);
            const float inv = wait_scale(dead);
            if (!dead) {
                char* const rs = pws + IN_RS + (size_t)slot * STRM + lane * 16;
                const u64 ta = __builtin_amdgcn_s_memrealtime();
                const u64 dt = slot_ticks(idx, cum[3]);
                unsigned head, wire;
                wire_split(cum, head, wire);
                LAS char* const img = lds + (EP_STRM + wid * STRM);
#pragma unroll
                for (int k = 0; k < 4; ++k) {
#pragma unroll
                    for (int u = 0; u < 8; ++u) {
                        const unsigned qd = q4(acc[2 * k + (u >> 2)][u & 3], inv);
                        LAS unsigned* d = (LAS unsigned*)(lds + addr[k * 8 + u]);
                        __hip_atomic_fetch_or(d, __builtin_amdgcn_perm(0u, qd, sel[k * 8 + u][0]), RLX, __HIP_MEMORY_SCOPE_WORKGROUP);
                        __hip_atomic_fetch_or(d + 1, __builtin_amdgcn_perm(0u, qd, sel[k * 8 + u][1]), RLX, __HIP_MEMORY_SCOPE_WORKGROUP);
                    }
                    asm volatile("" ::: "memory");
                    if (k == 0)
                        for (unsigned o = 0; o < head; o += 1024)
                            if (o + lane * 16 < head) st16_sys(rs + o, granule_pack(*(LAS u32x4*)(img + o + lane * 16), tag));
                }
                const unsigned lim = (wire + 63u) & ~63u;
                u32x4 pk[6];
#pragma unroll
                for (int i = 0; i < 6; ++i) {
                    pk[i] = granule_pack(*(LAS u32x4*)(img + head + i * 1024 + lane * 16), *(LAS u16*)(img + wire + i * 128 + lane * 2) | tag);
                    asm volatile("" : "+v"(pk[i]));
                }
                while (__builtin_amdgcn_s_memrealtime() - ta < dt) __builtin_amdgcn_s_sleep(1);
#pragma unroll
                for (int i = 0; i < 6; ++i)
                    if (head + i * 1024 + lane * 16 < lim) st16_sys(rs + head + i * 1024, pk[i]);
            }
        }
    }
}

extern "C" void kernel_launch(int rank, int nranks, ncclComm_t comm, hipStream_t stream,
                              void* const* d_in, const int* in_sizes, int n_in,
                              void* d_out, int out_size, const int* in_dtypes, int out_dtype,
                              void* const* d_out_all, void* const* d_ws_all, size_t ws_bytes) {
    if (nranks != NR || (unsigned)rank >= (unsigned)NR || ws_bytes < WS_END) return;
    static bool init[NR];
    if (!init[rank]) {
        HIP_CHECK(hipFuncSetAttribute((const void*)fused_kernel, hipFuncAttributeMaxDynamicSharedMemorySize, LDS_BYTES));
        init[rank] = true;
    }
    Params p;
    p.x = (const u16*)d_in[0]; p.w = (const u16*)d_in[1]; p.out = (char*)d_out;
    for (int r = 0; r < NR; ++r) p.ws[r] = (char*)d_ws_all[r];
    p.rank = rank; p.pad = 0; p.myws = p.ws[rank];
    fused_kernel<<<NWG, 512, LDS_BYTES, stream>>>(p);
    HIP_CHECK(hipGetLastError());
}
```

```cpp
#include <hip/hip_runtime.h>
#include <rccl/rccl.h>
#include <cstdio>

#define HIP_CHECK(expr) do { hipError_t _e = (expr); if (_e != hipSuccess) { \
    fprintf(stderr, "HIP error %s at %s:%d\n", hipGetErrorString(_e), __FILE__, __LINE__); \
    return; } } while (0)

#define LAS __attribute__((address_space(3)))
typedef short s16x4 __attribute__((ext_vector_type(4)));
typedef __bf16 bf16x8 __attribute__((ext_vector_type(8)));
typedef float f32x4 __attribute__((ext_vector_type(4)));
typedef unsigned u32x4 __attribute__((ext_vector_type(4)));
typedef unsigned u32x2 __attribute__((ext_vector_type(2)));
typedef unsigned long long u64;
typedef unsigned short u16;

constexpr int NR = 8;
constexpr int MS = 512, NG = 16384, KD = 4096;
constexpr int NS = NG / NR;
constexpr int BM = 256, BN = 128, BK = 64, NT = KD / BK;
constexpr int NXS = 3, NWS = 4;
constexpr int XSLOT = BM * BK * 2, WSLOT = BK * BN * 2;
constexpr int XBASE = 0, WBASE = NXS * XSLOT;
constexpr int LDS_BYTES = NXS * XSLOT + NWS * WSLOT;
constexpr int NWG = 256;
constexpr int STRM = 8192 + 1024;
constexpr int EP_STRM = 0;
constexpr int RCVB = STRM;
constexpr int EP_RCV = 4 * STRM;
constexpr int EP_LUT = EP_RCV + 4 * RCVB;
constexpr int EP_SCALE = EP_LUT + 4096;

constexpr size_t WS_CTR = 0;
constexpr size_t WS_TMO = 16384;
constexpr size_t WS_LG = 20480;
constexpr int NREP = 16;
constexpr size_t WS_GG = 32768;
constexpr size_t WS_IN = 2u << 20, INBOX = 2u << 20;
constexpr size_t IN_MF = 0;
constexpr size_t IN_MR = 4096;
constexpr size_t IN_RS = IN_MR + 2 * 128 * 1024;
static_assert(IN_RS + 128 * (size_t)STRM <= INBOX, "inbox");
constexpr size_t WS_END = WS_IN + NR * INBOX;
constexpr u64 DEADLINE = 100000000ull;
constexpr int WAUX = 2;
constexpr int NAGG = 2;
constexpr unsigned LINK_BPT = 590;
constexpr unsigned SLOT0 = 2, RWAIT = 60;
constexpr unsigned PACE_T0 = 260, PACE_TAU = 72, PACE_MIN = 132;

struct Params {
    const u16* x; const u16* w;
    char* out; char* myws;
    int rank; int pad;
    char* ws[NR];
};

struct LdsBase { __device__ __forceinline__ LAS char* operator+(unsigned a) const { return (LAS char*)(__UINTPTR_TYPE__)a; } };

#define WAIT_V(n) asm volatile("s_waitcnt vmcnt(" #n ")" ::: "memory")
#define BARRIER() asm volatile("s_barrier" ::: "memory")
#define RLX __ATOMIC_RELAXED
#define SC_AGENT __HIP_MEMORY_SCOPE_AGENT
#define SC_SYS __HIP_MEMORY_SCOPE_SYSTEM

template <int AUX> __device__ __forceinline__ void dma16(__amdgpu_buffer_rsrc_t rsrc, LAS char* dst, unsigned voff, unsigned soff) {
    __builtin_amdgcn_raw_ptr_buffer_load_lds(rsrc, (LAS void*)dst, 16, voff, soff, 0, AUX);
}
__device__ __forceinline__ void st16_sys(char* p, u32x4 v) {
    asm volatile("global_store_dwordx4 %0, %1, off sc0 sc1\n\ts_nop 1" :: "v"(p), "v"(v) : "memory");
}
__device__ __forceinline__ void st16_out(char* p, u32x4 v) {
    asm volatile("global_store_dwordx4 %0, %1, off sc1\n\ts_nop 1" :: "v"(p), "v"(v) : "memory");
}
__device__ __forceinline__ u32x4 ld16_sys(const char* p) {
    u32x4 v; asm volatile("global_load_dwordx4 %0, %1, off sc0 sc1\n\ts_waitcnt vmcnt(0)" : "=&v"(v) : "v"(p) : "memory"); return v;
}
__device__ __forceinline__ unsigned q4(f32x4 v, float inv) {
    unsigned r = 0;
    r = __builtin_amdgcn_cvt_pk_u8_f32(v[0] * inv, 0, r);
    r = __builtin_amdgcn_cvt_pk_u8_f32(v[1] * inv, 1, r);
    r = __builtin_amdgcn_cvt_pk_u8_f32(v[2] * inv, 2, r);
    r = __builtin_amdgcn_cvt_pk_u8_f32(v[3] * inv, 3, r);
    const unsigned h = r & 0x80808080u;
    return (r | (h - (h >> 7))) & 0x7f7f7f7fu;
}
template <int CTRL, int RMASK> __device__ __forceinline__ unsigned dpp0(unsigned x) { return (unsigned)__builtin_amdgcn_update_dpp(0, (int)x, CTRL, RMASK, 0xf, true); }
__device__ __forceinline__ unsigned wave_scan_add(unsigned x) {
    x += dpp0<0x111, 0xf>(x); x += dpp0<0x112, 0xf>(x); x += dpp0<0x114, 0xf>(x); x += dpp0<0x118, 0xf>(x);
    x += dpp0<0x142, 0xa>(x); x += dpp0<0x143, 0xc>(x); return x;
}
__device__ __forceinline__ unsigned wave_max_u(unsigned x) {
    x = max(x, dpp0<0x111, 0xf>(x)); x = max(x, dpp0<0x112, 0xf>(x)); x = max(x, dpp0<0x114, 0xf>(x)); x = max(x, dpp0<0x118, 0xf>(x));
    x = max(x, dpp0<0x142, 0xa>(x)); x = max(x, dpp0<0x143, 0xc>(x)); return (unsigned)__builtin_amdgcn_readlane((int)x, 63);
}
__device__ __forceinline__ void stream_offsets(const u32x4 keep, unsigned (&start)[4], unsigned (&cum)[4]) {
    const unsigned c0 = __builtin_popcount(keep[0]), c1 = __builtin_popcount(keep[1]), c2 = __builtin_popcount(keep[2]), c3 = __builtin_popcount(keep[3]);
    const unsigned a = wave_scan_add(c0 | (c1 << 16)), b = wave_scan_add(c2 | (c3 << 16));
    const unsigned ta = (unsigned)__builtin_amdgcn_readlane((int)a, 63), tb = (unsigned)__builtin_amdgcn_readlane((int)b, 63);
    cum[0] = ta & 0xffffu; cum[1] = cum[0] + (ta >> 16); cum[2] = cum[1] + (tb & 0xffffu); cum[3] = cum[2] + (tb >> 16);
    start[0] = (a & 0xffffu) - c0; start[1] = cum[0] + (a >> 16) - c1; start[2] = cum[1] + (b & 0xffffu) - c2; start[3] = cum[2] + (b >> 16) - c3;
}
__device__ __forceinline__ unsigned stream_tag(unsigned call) { const unsigned n = call % 3u + 1u; return ((n & 1u) << 7) | ((n & 2u) << 14); }
__device__ __forceinline__ u32x4 granule_pack(u32x4 d, unsigned w) {
    const unsigned x = (w | (w << 8)) & 0x00ff00ffu, u = (x | (x << 4)) & 0x0f0f0f0fu;
    return (u32x4){d[0] | ((u << 7) & 0x80808080u), d[1] | ((u << 6) & 0x80808080u), d[2] | ((u << 5) & 0x80808080u), d[3] | ((u << 4) & 0x80808080u)};
}
__device__ __forceinline__ unsigned granule_word(u32x4 d) {
    const unsigned u = ((d[0] >> 7) & 0x01010101u) | ((d[1] >> 6) & 0x02020202u) | ((d[2] >> 5) & 0x04040404u) | ((d[3] >> 4) & 0x08080808u);
    const unsigned x = (u | (u >> 4)) & 0x00ff00ffu;
    return (x | (x >> 8)) & 0xffffu;
}
__device__ __forceinline__ void wire_split(const unsigned (&cum)[4], unsigned& head, unsigned& wire) {
    head = cum[0] & ~127u;
    wire = head + (cum[3] - head + 17u) / 18u * 16u;
}
__device__ __forceinline__ u64 slot_ticks(int pos, unsigned bytes) {
    return (u64)(((unsigned)(3 * (32 - SLOT0) + 8 * pos) * bytes) / (3 * LINK_BPT));
}
__global__ __launch_bounds__(512, 2) void fused_kernel(const Params p) {
    extern __shared__ __attribute__((aligned(16))) char smem[];
    asm volatile("" :: "s"((LAS char*)smem));
    const LdsBase lds;
    asm volatile("" :: "s"(p.x), "s"(p.w), "s"(p.myws), "s"(p.rank));
    const int tid = threadIdx.x, lane = tid & 63;
    const int wid = __builtin_amdgcn_readfirstlane(tid >> 6);
    const int b = blockIdx.x, peer = b & 7, idx = b >> 3;
    auto col_block = [](int i) { return (int)((0xd951fecba8764320ull >> (4 * i)) & 15u); };
    const int cbl = col_block(idx >> 1), mh = idx & 1;
    const int m0 = mh * BM, n0 = (peer * 16 + cbl) * BN;
    const bool local = peer == p.rank;
    char* const myws = p.myws;
    unsigned* const tmo = (unsigned*)(myws + WS_TMO);
    const int fr = lane & 15, g = lane >> 4;
    const int w4 = wid & 3, wm = w4 >> 1, wn = w4 & 1;

    if (__builtin_expect(wid >= 4, 1)) {
        const int lw = w4;
        const unsigned tstart = (unsigned)__builtin_amdgcn_s_memrealtime();
        __amdgpu_buffer_rsrc_t rx = __builtin_amdgcn_make_buffer_rsrc((void*)p.x, 0, MS * KD * 2, 0x00020000);
        __amdgpu_buffer_rsrc_t rw = __builtin_amdgcn_make_buffer_rsrc((void*)p.w, 0, (unsigned)((size_t)KD * NG * 2), 0x00020000);
        const unsigned xrow = m0 + lw * 64 + (lane >> 3);
        const unsigned xv0 = xrow * (KD * 2) + (((lane & 7) ^ (lane >> 4)) << 4);
        const unsigned xv1 = xrow * (KD * 2) + (((lane & 7) ^ (4 + (lane >> 4))) << 4);
        const unsigned wv = (lw * 4 + (lane >> 4)) * (NG * 2) + n0 * 2 + (((lane & 15) ^ (((lane >> 4) | ((lw & 2) << 1)) << 1)) << 4);
        auto issueX = [&](int t, int slot) {
            LAS char* d = lds + XBASE + slot * XSLOT + lw * 64 * 128;
#pragma unroll
            for (int i = 0; i < 8; ++i) dma16<0>(rx, d + i * 1024, (i & 1) ? xv1 : xv0, (unsigned)(t * 128 + i * 8 * KD * 2));
        };
        auto issueW = [&](int t) {
#pragma unroll
            for (int i = 0; i < 4; ++i) dma16<WAUX>(rw, lds + WBASE + (t & 3) * WSLOT + i * 4096 + lw * 1024, wv, (unsigned)(t * 64 + i * 16) * (NG * 2));
        };
        auto issueWs = [&](int t) {
#pragma unroll
            for (int i = 0; i < 4; ++i) dma16<0>(rw, lds + WBASE + (t & 3) * WSLOT + i * 4096 + lw * 1024, wv, (unsigned)(t * 64 + i * 16) * (NG * 2));
        };
        unsigned touch = 0;
        const unsigned tv = b * 16384 + (lw >> 1) * 8192 + ((lw & 1) * 8 + (lane & 7)) * 128;
        auto touchX = [&](unsigned kt0) { asm volatile("buffer_load_dword %0, %1, %2, %3 offen" : "+v"(touch) : "v"(tv), "s"(rx), "s"(kt0 * 128u) : "memory"); };
        int xs = 1;
        auto nextX = [&](int t) { issueX(t, xs); xs = (xs == NXS - 1) ? 0 : xs + 1; };
        issueW(0); issueX(0, 0); issueW(1); issueX(1, 1);
        WAIT_V(8);
        BARRIER();
        unsigned cv;
        issueW(2); touchX(3);
        asm volatile("global_load_dword %0, %1, off sc1" : "=v"(cv) : "v"(myws + WS_CTR + b * 64) : "memory");
        issueX(2, 2); issueW(3);
        WAIT_V(14); BARRIER();
        issueX(3, 0); issueW(4); touchX(19); touchX(35); touchX(48);
        asm volatile("s_waitcnt vmcnt(15)" : "+v"(cv) :: "memory");
        BARRIER();
        const unsigned call = __builtin_amdgcn_readfirstlane(cv) + 1u;
        {
            const char* wp;
            if (lw == 0) wp = local ? p.ws[idx & 7] + WS_GG + 8192 : p.ws[peer] + WS_IN + p.rank * INBOX + 3072 + (idx & 1) * (INBOX - 4096);
            else if (lw == 2) wp = (idx & 2) ? myws + WS_IN + peer * INBOX + 3072 + (idx & 1) * (INBOX - 4096) : p.out + (size_t)(peer * MS + (idx & 1) * (MS - 1)) * NS;
            else wp = (const char*)p.w + (size_t)(idx * 2 + (lw >> 1)) * (BK * NG * 2) + 64 * 1024;
            asm volatile("s_load_dword s95, %0, 0x0" :: "s"(wp) : "s95");
        }
        unsigned tlast = 0;
        auto pace = [&](int n) {
            if (lw == 1) {
                const unsigned tgt = max(PACE_T0 + (unsigned)n * PACE_TAU, tlast + PACE_MIN);
                while ((unsigned)__builtin_amdgcn_s_memrealtime() - tstart < tgt) __builtin_amdgcn_s_sleep(1);
            }
        };
        int t = 2;
        const unsigned wl = ((unsigned)(__UINTPTR_TYPE__)p.w + (unsigned)n0 * 2u) >> 7;
        const bool slow = ((wl - 2u) & 7u) < 2u || ((wl - 62u) & 127u) < 2u;
        const bool shift = slow ? mh : (((unsigned)(__UINTPTR_TYPE__)p.x >> 7) & 1u);
        if (shift) { nextX(4); issueW(5); WAIT_V(12); BARRIER(); t = 3; }
        if (slow) {
            for (; t < NT - 4; t += 2) {
                nextX(t + 2); issueWs(t + 3); issueWs(t + 4);
                WAIT_V(16); BARRIER();
                nextX(t + 3);
                pace(t + 2);
                WAIT_V(12); BARRIER();
                if (lw == 1) tlast = (unsigned)__builtin_amdgcn_s_memrealtime() - tstart;
            }
        } else {
            for (; t < NT - 4; t += 2) {
                nextX(t + 2); issueW(t + 3); issueW(t + 4);
                WAIT_V(16); BARRIER();
                nextX(t + 3);
                pace(t + 2);
                WAIT_V(12); BARRIER();
                if (lw == 1) tlast = (unsigned)__builtin_amdgcn_s_memrealtime() - tstart;
            }
        }
        if (!shift) { nextX(NT - 2); issueW(NT - 1); WAIT_V(12); BARRIER(); }
        nextX(NT - 1);
        char* const inbox = myws + WS_IN + peer * INBOX;
        char* const rs = inbox + IN_RS + (size_t)(idx * 4 + w4) * STRM;
#pragma unroll
        for (int i = 0; i < 4; ++i) st16_sys(rs + (call & 1u) * 4096 + i * 1024 + lane * 16, (u32x4){0u, 0u, 0u, 0u});
        WAIT_V(12); BARRIER();
        WAIT_V(4); BARRIER();
        BARRIER();
        if (wid == 4) asm volatile("ds_write_b128 %0, %1\n\ts_waitcnt lgkmcnt(0)" :: "v"((unsigned)EP_SCALE), "v"((u32x4){0u, 0u, 0u, 0u}) : "memory");
        WAIT_V(0);
        asm volatile("s_waitcnt lgkmcnt(0)" :: "v"(touch) : "s95");
        BARRIER();
        const unsigned tag = stream_tag(call);
        const u64 t0 = __builtin_amdgcn_s_memrealtime();
        if (local) {
            if (idx < NAGG) {
                bool dead = false, ok = false; unsigned m = 0;
                const char* gp = myws + WS_LG + w4 * 2048 + lane * 16;
                char* base = p.ws[0];
#pragma unroll
                for (int r = 1; r < NR; ++r) base = (lane & 7) == r ? p.ws[r] : base;
                base += WS_GG + ((call & 1) * NREP + (lane >> 3)) * 128 + p.rank * 8;
                asm volatile("" : "+v"(base));
                for (;;) {
                    if (!ok) {
                        u32x4 v0, v1;
                        asm volatile("global_load_dwordx4 %0, %2, off sc1\n\tglobal_load_dwordx4 %1, %2, off offset:1024 sc1\n\ts_waitcnt vmcnt(0)" : "=&v"(v0), "=&v"(v1) : "v"(gp) : "memory");
                        m = max(max(v0[0], v0[2]), max(v1[0], v1[2]));
                        ok = v0[1] == call && v0[3] == call && v1[1] == call && v1[3] == call;
                    }
                    if (__all(ok)) break;
                    if (__builtin_amdgcn_s_memrealtime() - t0 > DEADLINE) { dead = true; break; }
                    __builtin_amdgcn_s_sleep(1);
                }
                if (!dead) {
                    m = wave_max_u(m);
                    unsigned cnt = 0;
                    if (lane == 0) {
                        __hip_atomic_fetch_max((LAS unsigned*)(lds + EP_SCALE + 8), m, RLX, __HIP_MEMORY_SCOPE_WORKGROUP);
                        cnt = __hip_atomic_fetch_add((LAS unsigned*)(lds + EP_SCALE + 12), 1u, RLX, __HIP_MEMORY_SCOPE_WORKGROUP);
                    }
                    if (__builtin_amdgcn_readfirstlane(cnt) == 3) {
                        m = __hip_atomic_load((LAS unsigned*)(lds + EP_SCALE + 8), RLX, __HIP_MEMORY_SCOPE_WORKGROUP);
#pragma unroll
                        for (int i = 0; i < NREP / 8; ++i) __hip_atomic_store((u64*)(base + i * 1024), ((u64)call << 32) | m, RLX, SC_SYS);
                    }
                } else if (lane == 0) __hip_atomic_store(tmo, call, RLX, SC_AGENT);
            }
        } else {
            bool dead = false;
            {
                const unsigned nib = lane & 15; unsigned pos = lane >> 4, sel = 0;
#pragma unroll
                for (int e = 0; e < 4; ++e) { const bool k = (nib >> e) & 1; sel |= (k ? pos : 0x0cu) << (8 * e); pos += k; }
                *(LAS unsigned*)(lds + EP_LUT + wid * 512 + lane * 4) = sel;
            }
            unsigned n = 0;
            while (__hip_atomic_load((unsigned*)(inbox + IN_MF + (idx * 4 + w4) * 16), RLX, SC_SYS) < call) {
                if (((++n & 63u) == 0u && __hip_atomic_load(tmo, RLX, SC_AGENT) == call) || __builtin_amdgcn_s_memrealtime() - t0 > DEADLINE) { dead = true; break; }
                __builtin_amdgcn_s_sleep(32);
            }
            const u32x4 keep = ld16_sys(inbox + IN_MR + (size_t)((call & 1) * 128 + idx * 4 + w4) * 1024 + lane * 16);
            unsigned start[4], cum[4];
            stream_offsets(keep, start, cum);
            LAS char* const lb = lds + EP_RCV + w4 * RCVB;
            *(LAS u32x4*)(lb + lane * 32) = keep;
            *(LAS u32x4*)(lb + lane * 32 + 16) = (u32x4){start[0], start[1], start[2], start[3]};
            unsigned addr[32], sel[32];
#pragma unroll
            for (int gg = 0; gg < 4; ++gg) {
                const u32x4 kp = *(LAS u32x4*)(lb + ((lane >> 2) + 16 * gg) * 32), stt = *(LAS u32x4*)(lb + ((lane >> 2) + 16 * gg) * 32 + 16);
#pragma unroll
                for (int i = 0; i < 8; ++i) {
                    const unsigned sh = (i & 1) * 16 + (lane & 3) * 4;
                    const unsigned off = stt[i >> 1] + __builtin_popcount(kp[i >> 1] & ((1u << sh) - 1u)), nib = (kp[i >> 1] >> sh) & 15u;
                    sel[i * 4 + gg] = *(LAS unsigned*)(lds + EP_LUT + wid * 512 + (((off & 3u) << 4) | nib) * 4);
                    addr[i * 4 + gg] = EP_RCV + w4 * RCVB + (off & ~3u);
                }
            }
#pragma unroll
            for (int i = 0; i < 32; ++i) asm volatile("" : "+v"(addr[i]), "+v"(sel[i]));
            unsigned st;
            while ((st = __builtin_amdgcn_readfirstlane(__hip_atomic_load((LAS unsigned*)(lds + EP_SCALE + 4), RLX, __HIP_MEMORY_SCOPE_WORKGROUP))) == 0u) __builtin_amdgcn_s_sleep(8);
            const unsigned total = cum[3];
            const int sidx = (idx - 4 - 8 * w4) & 31;
            {
                const unsigned dt = ((unsigned)slot_ticks(sidx, total) + RWAIT) << 2;
                while (((unsigned)__builtin_amdgcn_s_memrealtime() << 2) - (st & ~3u) < dt) __builtin_amdgcn_s_sleep(2);
            }
            char* o = p.out + (size_t)(peer * MS + m0 + wm * 128 + (lane >> 2)) * NS + col_block(sidx >> 1) * BN + wn * 64 + (lane & 3) * 16;
            unsigned head, wire;
            wire_split(cum, head, wire);
            const __amdgpu_buffer_rsrc_t rr = __builtin_amdgcn_make_buffer_rsrc((void*)rs, 0, wire, 0x00020000);
            unsigned pos = 0;
            while (pos < wire && !dead) {
                u32x4 v[4];
#pragma unroll
                for (int i = 0; i < 4; ++i) v[i] = __builtin_amdgcn_raw_buffer_load_b128(rr, pos + lane * 16 + i * 1024, 0, 17);
                unsigned adv = 4096;
#pragma unroll
                for (int i = 3; i >= 0; --i) {
                    const unsigned off = pos + i * 1024 + lane * 16, w = granule_word(v[i]);
                    const bool in = off < wire;
                    const u64 bad = __ballot(in && (w & 0x8080u) != tag);
                    if (in) {
                        *(LAS u32x4*)(lb + off) = v[i] & 0x7f7f7f7fu;
                        if (off >= head) *(LAS u16*)(lb + wire + ((off - head) >> 3)) = (u16)(w & 0x7f7fu);
                    }
                    if (bad) adv = i * 1024 + (unsigned)__builtin_ctzll(bad) * 16u;
                }
                pos += adv & ~127u;
                if (pos < wire) {
                    if (((++n & 63u) == 0u && __hip_atomic_load(tmo, RLX, SC_AGENT) == call) || __builtin_amdgcn_s_memrealtime() - t0 > DEADLINE) dead = true;
                    __builtin_amdgcn_s_sleep(1);
                }
            }
            unsigned d[32];
#pragma unroll
            for (int i = 0; i < 32; ++i) {
                const LAS unsigned* sp = (const LAS unsigned*)(lds + addr[i]);
                d[i] = __builtin_amdgcn_perm(sp[1], sp[0], sel[i]);
            }
#pragma unroll
            for (int i = 0; i < 8; ++i) {
                u32x4 v = {d[i * 4], d[i * 4 + 1], d[i * 4 + 2], d[i * 4 + 3]};
                if (dead) v = (u32x4){0x80808080u, 0x80808080u, 0x80808080u, 0x80808080u};
                st16_out(o + (size_t)i * 16 * NS, v);
            }
            if (dead && lane == 0) __hip_atomic_store(tmo, call, RLX, SC_AGENT);
        }
    } else {
        const unsigned xa0 = XBASE + wm * 16384 + fr * 128 + ((g ^ (fr >> 1)) << 4);
        const int qq = fr >> 2, pp = lane & 3;
        const int f = qq | ((g & 1) << 2);
        const unsigned wa0 = WBASE + (8 * g + qq) * 256 + ((((wn * 8) + (pp >> 1)) ^ (f << 1)) << 4) + (pp & 1) * 8;
        const unsigned c0 = __hip_atomic_load((unsigned*)(myws + WS_CTR + b * 64), RLX, SC_AGENT);
        f32x4 acc[8][4];
#pragma unroll
        for (int i = 0; i < 8; ++i)
#pragma unroll
            for (int j = 0; j < 4; ++j) acc[i][j] = (f32x4){0.f, 0.f, 0.f, 0.f};
        bf16x8 XP[2][2], XQ[2][2], WA[4][2], WB[4][2];
#define PIN() __builtin_amdgcn_sched_barrier(0)
#define LGKM(n) do { __builtin_amdgcn_s_waitcnt(0xC07F | ((n) << 8)); PIN(); } while (0)
        auto rdX = [&](unsigned a, int off) { bf16x8 v = *(LAS bf16x8*)(lds + a + off); PIN(); return v; };
        auto rdW = [&](unsigned a, int off) { s16x4 v = __builtin_amdgcn_ds_read_tr16_b64_v4i16((LAS s16x4*)(lds + a + off)); PIN(); return v; };
        auto cat = [&](s16x4 lo, s16x4 hi) { return __builtin_bit_cast(bf16x8, __builtin_shufflevector(lo, hi, 0, 1, 2, 3, 4, 5, 6, 7)); };
        auto quarter = [&](int mf, bf16x8 (&X)[2][2], bf16x8 (&W)[4][2], auto fill) {
#pragma unroll
            for (int n = 0; n < 16; ++n) {
                const int i = n >> 3, j = (n >> 1) & 3, ks = n & 1;
                asm volatile("v_mfma_f32_16x16x32_bf16 %0, %1, %2, %0" : "+v"(acc[mf + i][j]) : "v"(W[j][ks]), "v"(X[i][ks])); PIN();
                fill(n);
            }
        };
        int xs = 0, ws = WSLOT;
        unsigned xc0 = xa0, xc1 = xa0 ^ 64, wq[4];
#pragma unroll
        for (int j = 0; j < 4; ++j) wq[j] = (wa0 ^ (j << 5)) + WSLOT;
        auto tile = [&](bf16x8 (&Wc)[4][2], bf16x8 (&Wn)[4][2]) {
            s16x4 lo;
            auto fx = [&](bf16x8 (&D)[2][2], int mf, int n) { if (n < 4) D[n >> 1][n & 1] = rdX((n & 1) ? xc1 : xc0, (mf + (n >> 1)) * 2048); };
            auto fw = [&](int f0, int cnt, int n) {
                const int m = n - 4;
                if (m >= 0 && m < 2 * cnt) {
                    const int f = f0 + (m >> 1), j = f >> 1, ks = f & 1;
                    if (!(m & 1)) lo = rdW(wq[j], ks * 8192); else Wn[j][ks] = cat(lo, rdW(wq[j], ks * 8192 + 1024));
                }
            };
            quarter(0, XP, Wc, [&](int n) { fx(XQ, 2, n); fw(0, 3, n); if (n == 14) LGKM(6); });
            quarter(2, XQ, Wc, [&](int n) { fx(XP, 4, n); fw(3, 3, n); if (n == 14) LGKM(6); });
            quarter(4, XP, Wc, [&](int n) {
                fx(XQ, 6, n); fw(6, 2, n);
                if (n == 9) { xs = (xs == NXS - 1) ? 0 : xs + 1; asm volatile("" : "+s"(xs)); PIN(); }
                if (n == 11) { xc0 = xa0 + xs * XSLOT; asm volatile("" : "+v"(xc0)); PIN(); }
                if (n == 12) { xc1 = (xa0 ^ 64) + xs * XSLOT; asm volatile("" : "+v"(xc1)); PIN(); }
                if (n == 14) LGKM(0);
            });
            BARRIER(); PIN();
            quarter(6, XQ, Wc, [&](int n) {
                fx(XP, 0, n);
                if (n == 5) { ws = (ws + WSLOT) & (NWS * WSLOT - 1); asm volatile("" : "+s"(ws)); PIN(); }
                if (n >= 7 && n < 11) { wq[n - 7] = (wa0 ^ ((n - 7) << 5)) + ws; asm volatile("" : "+v"(wq[n - 7])); PIN(); }
                if (n == 14) LGKM(0);
            });
        };
        const unsigned call = __builtin_amdgcn_readfirstlane(c0) + 1u;
#pragma unroll
        for (int i = 0; i < 8; ++i) asm volatile("" : "+v"(acc[i][0]), "+v"(acc[i][1]), "+v"(acc[i][2]), "+v"(acc[i][3]));
        LGKM(0);
        BARRIER(); PIN();
#pragma unroll
        for (int j = 0; j < 4; ++j)
#pragma unroll
            for (int ks = 0; ks < 2; ++ks) { const s16x4 lo = rdW(wa0 ^ (j << 5), ks * 8192); WA[j][ks] = cat(lo, rdW(wa0 ^ (j << 5), ks * 8192 + 1024)); }
#pragma unroll
        for (int n = 0; n < 4; ++n) XP[n >> 1][n & 1] = rdX((n & 1) ? xc1 : xc0, (n >> 1) * 2048);
        LGKM(0);
#pragma unroll 1
        for (int t = 0; t < NT; t += 2) { tile(WA, WB); tile(WB, WA); }
        asm volatile("s_nop 15" ::: "memory");
#pragma unroll
        for (int i = 0; i < 8; ++i) asm volatile("" : "+v"(acc[i][0]), "+v"(acc[i][1]), "+v"(acc[i][2]), "+v"(acc[i][3]));
        int mx = 0;
#pragma unroll
        for (int i = 0; i < 8; ++i)
#pragma unroll
            for (int j = 0; j < 4; ++j) {
                mx = max(max(mx, __float_as_int(acc[i][j][0])), __float_as_int(acc[i][j][1]));
                mx = max(max(mx, __float_as_int(acc[i][j][2])), __float_as_int(acc[i][j][3]));
            }
        unsigned wmax = wave_max_u((unsigned)mx);
        asm volatile("" : "+s"(wmax));
        BARRIER();
        if (lane == 0) {
            __hip_atomic_store((u64*)(myws + WS_LG) + b * 4 + wid, ((u64)call << 32) | wmax, RLX, SC_AGENT);
            if (wid == 0) __hip_atomic_store((unsigned*)(myws + WS_CTR + b * 64), call, RLX, SC_AGENT);
        }
        const u64 t0 = __builtin_amdgcn_s_memrealtime();
        auto wait_scale = [&](bool& dead) {
            LAS u64* sw = (LAS u64*)(lds + EP_SCALE);
            if (wid == 0) {
                unsigned gm, n = 0;
                for (;;) {
                    const u64 v = __hip_atomic_load((u64*)(myws + WS_GG + ((call & 1) * NREP + (idx & (NREP - 1))) * 128) + (lane & 7), RLX, SC_SYS);
                    gm = (unsigned)v;
                    if (__all((unsigned)(v >> 32) == call)) break;
                    if (((++n & 63u) == 0u && __hip_atomic_load(tmo, RLX, SC_AGENT) == call) || __builtin_amdgcn_s_memrealtime() - t0 > DEADLINE) { dead = true; break; }
                    __builtin_amdgcn_s_sleep(1);
                }
                if (dead && lane == 0) __hip_atomic_store(tmo, call, RLX, SC_AGENT);
                const float am = __uint_as_float(wave_max_u(gm));
                const float inv = am > 0.f ? 127.0f / am : 0.f;
                __hip_atomic_store(sw, ((u64)(((unsigned)__builtin_amdgcn_s_memrealtime() << 2) | (dead ? 2u : 1u)) << 32) | __float_as_uint(inv), RLX, __HIP_MEMORY_SCOPE_WORKGROUP);
                return inv;
            }
            u64 v;
            while (((v = __hip_atomic_load(sw, RLX, __HIP_MEMORY_SCOPE_WORKGROUP)) >> 32) == 0u) __builtin_amdgcn_s_sleep(1);
            dead = ((v >> 32) & 3u) == 2u;
            return __uint_as_float((unsigned)v);
        };
        bool dead = false;
        if (local) {
            const float inv = wait_scale(dead);
            char* o = p.out + (size_t)(p.rank * MS + m0 + wm * 128 + (lane >> 2)) * NS + cbl * BN + wn * 64 + (lane & 3) * 16;
            LAS char* const lb = lds + EP_STRM + wid * STRM;
#pragma unroll
            for (int i = 0; i < 8; ++i)
#pragma unroll
                for (int j = 0; j < 4; ++j) *(LAS unsigned*)(lb + i * 1024 + fr * 64 + ((j ^ (fr >> 2)) & 3) * 16 + g * 4) = q4(acc[i][j], inv);
            u32x4 v[8];
#pragma unroll
            for (int i = 0; i < 8; ++i) v[i] = *(LAS u32x4*)(lb + i * 1024 + (lane >> 2) * 64 + ((lane ^ (lane >> 4)) & 3) * 16);
#pragma unroll
            for (int i = 0; i < 8; ++i) st16_out(o + (size_t)i * 16 * NS, dead ? (u32x4){0x80808080u, 0x80808080u, 0x80808080u, 0x80808080u} : v[i]);
        } else {
            const int slot = ((idx + 4 + 8 * wid) & 31) * 4 + wid;
            char* const pws = p.ws[peer] + WS_IN + p.rank * INBOX;
            const unsigned tag = stream_tag(call);
            u32x4 keep;
#pragma unroll
            for (int k = 0; k < 4; ++k) {
                unsigned m = 0;
#pragma unroll
                for (int u = 7; u >= 0; --u)
#pragma unroll
                    for (int e = 3; e >= 0; --e) m = __builtin_amdgcn_alignbit(m, __float_as_uint(acc[2 * k + (u >> 2)][u & 3][e]), 31);
                keep[k] = ~m;
            }
            WAIT_V(0);
            st16_sys(pws + IN_MR + (size_t)((call & 1) * 128 + slot) * 1024 + lane * 16, keep);
            {
                const unsigned nib = lane & 15; unsigned pos = lane >> 4; u64 sel = 0x0c0c0c0c0c0c0c0cull;
#pragma unroll
                for (int e = 0; e < 4; ++e) { const bool k = (nib >> e) & 1; const u64 x = (u64)(0x0cu ^ (unsigned)e) << (8 * pos); sel ^= k ? x : 0ull; pos += k; }
                *(LAS u64*)(lds + EP_LUT + wid * 512 + lane * 8) = sel;
            }
#pragma unroll
            for (int i = 0; i < 9; ++i) *(LAS u32x4*)(lds + EP_STRM + wid * STRM + i * 1024 + lane * 16) = (u32x4){0u, 0u, 0u, 0u};
            unsigned start[4], cum[4];
            stream_offsets(keep, start, cum);
            unsigned addr[32]; u32x2 sel[32];
#pragma unroll
            for (int k = 0; k < 4; ++k) {
                unsigned off = start[k];
#pragma unroll
                for (int u = 0; u < 8; ++u) {
                    const unsigned nib = (keep[k] >> (4 * u)) & 15u;
                    sel[k * 8 + u] = *(LAS u32x2*)(lds + EP_LUT + wid * 512 + (((off & 3u) << 4) | nib) * 8);
                    addr[k * 8 + u] = EP_STRM + wid * STRM + (off & ~3u);
                    off += __builtin_popcount(nib);
                }
            }
#pragma unroll
            for (int i = 0; i < 32; ++i) asm volatile("" : "+v"(addr[i]), "+v"(sel[i]));
            WAIT_V(0);
            if (lane == 0) __hip_atomic_store((unsigned*)(pws + IN_MF + slot * 16), call, RLX, SC_SYS);
            const float inv = wait_scale(dead);
            if (!dead) {
                char* const rs = pws + IN_RS + (size_t)slot * STRM + lane * 16;
                const u64 ta = __builtin_amdgcn_s_memrealtime();
                const u64 dt = slot_ticks(idx, cum[3]);
                unsigned head, wire;
                wire_split(cum, head, wire);
                LAS char* const img = lds + (EP_STRM + wid * STRM);
#pragma unroll
                for (int k = 0; k < 4; ++k) {
#pragma unroll
                    for (int u = 0; u < 8; ++u) {
                        const unsigned qd = q4(acc[2 * k + (u >> 2)][u & 3], inv);
                        LAS unsigned* d = (LAS unsigned*)(lds + addr[k * 8 + u]);
                        __hip_atomic_fetch_or(d, __builtin_amdgcn_perm(0u, qd, sel[k * 8 + u][0]), RLX, __HIP_MEMORY_SCOPE_WORKGROUP);
                        __hip_atomic_fetch_or(d + 1, __builtin_amdgcn_perm(0u, qd, sel[k * 8 + u][1]), RLX, __HIP_MEMORY_SCOPE_WORKGROUP);
                    }
                    asm volatile("" ::: "memory");
                    if (k == 0)
                        for (unsigned o = 0; o < head; o += 1024)
                            if (o + lane * 16 < head) st16_sys(rs + o, granule_pack(*(LAS u32x4*)(img + o + lane * 16), tag));
                }
                const unsigned lim = (wire + 63u) & ~63u;
                u32x4 pk[6];
#pragma unroll
                for (int i = 0; i < 6; ++i) {
                    pk[i] = granule_pack(*(LAS u32x4*)(img + head + i * 1024 + lane * 16), *(LAS u16*)(img + wire + i * 128 + lane * 2) | tag);
                    asm volatile("" : "+v"(pk[i]));
                }
                while (__builtin_amdgcn_s_memrealtime() - ta < dt) __builtin_amdgcn_s_sleep(1);
#pragma unroll
                for (int i = 0; i < 6; ++i)
                    if (head + i * 1024 + lane * 16 < lim) st16_sys(rs + head + i * 1024, pk[i]);
            }
        }
    }
}

extern "C" void kernel_launch(int rank, int nranks, ncclComm_t comm, hipStream_t stream,
                              void* const* d_in, const int* in_sizes, int n_in,
                              void* d_out, int out_size, const int* in_dtypes, int out_dtype,
                              void* const* d_out_all, void* const* d_ws_all, size_t ws_bytes) {
    if (nranks != NR || (unsigned)rank >= (unsigned)NR || ws_bytes < WS_END) return;
    static bool init[NR];
    if (!init[rank]) {
        HIP_CHECK(hipFuncSetAttribute((const void*)fused_kernel, hipFuncAttributeMaxDynamicSharedMemorySize, LDS_BYTES));
        init[rank] = true;
    }
    Params p;
    p.x = (const u16*)d_in[0]; p.w = (const u16*)d_in[1]; p.out = (char*)d_out;
    for (int r = 0; r < NR; ++r) p.ws[r] = (char*)d_ws_all[r];
    p.rank = rank; p.pad = 0; p.myws = p.ws[rank];
    fused_kernel<<<NWG, 512, LDS_BYTES, stream>>>(p);
    HIP_CHECK(hipGetLastError());
}
```

```cpp
#include <hip/hip_runtime.h>
#include <rccl/rccl.h>
#include <cstdio>

#define HIP_CHECK(expr) do { hipError_t _e = (expr); if (_e != hipSuccess) { \
    fprintf(stderr, "HIP error %s at %s:%d\n", hipGetErrorString(_e), __FILE__, __LINE__); \
    return; } } while (0)

#define LAS __attribute__((address_space(3)))
typedef short s16x4 __attribute__((ext_vector_type(4)));
typedef __bf16 bf16x8 __attribute__((ext_vector_type(8)));
typedef float f32x4 __attribute__((ext_vector_type(4)));
typedef unsigned u32x4 __attribute__((ext_vector_type(4)));
typedef unsigned u32x2 __attribute__((ext_vector_type(2)));
typedef unsigned long long u64;
typedef unsigned short u16;

constexpr int NR = 8;
constexpr int MS = 512, NG = 16384, KD = 4096;
constexpr int NS = NG / NR;
constexpr int BM = 256, BN = 128, BK = 64, NT = KD / BK;
constexpr int NXS = 3, NWS = 4;
constexpr int XSLOT = BM * BK * 2, WSLOT = BK * BN * 2;
constexpr int XBASE = 0, WBASE = NXS * XSLOT;
constexpr int LDS_BYTES = NXS * XSLOT + NWS * WSLOT;
constexpr int NWG = 256;
constexpr int STRM = 8192 + 1024;
constexpr int EP_STRM = 0;
constexpr int RCVB = STRM;
constexpr int EP_RCV = 4 * STRM;
constexpr int EP_LUT = EP_RCV + 4 * RCVB;
constexpr int EP_SCALE = EP_LUT + 4096;

constexpr size_t WS_CTR = 0;
constexpr size_t WS_TMO = 16384;
constexpr size_t WS_LG = 20480;
constexpr int NREP = 16;
constexpr size_t WS_GG = 32768;
constexpr size_t WS_IN = 2u << 20, INBOX = 2u << 20;
constexpr size_t IN_MF = 0;
constexpr size_t IN_MR = 4096;
constexpr size_t IN_RS = IN_MR + 2 * 128 * 1024;
static_assert(IN_RS + 128 * (size_t)STRM <= INBOX, "inbox");
constexpr size_t WS_END = WS_IN + NR * INBOX;
constexpr u64 DEADLINE = 100000000ull;
constexpr int WAUX = 2;
constexpr int NAGG = 2;
constexpr unsigned LINK_BPT = 590;
constexpr unsigned SLOT0 = 2, RWAIT = 60;
constexpr unsigned PACE_T0 = 260, PACE_TAU = 72, PACE_MIN = 132;

struct Params {
    const u16* x; const u16* w;
    char* out; char* myws;
    int rank; int pad;
    char* ws[NR];
};

struct LdsBase { __device__ __forceinline__ LAS char* operator+(unsigned a) const { return (LAS char*)(__UINTPTR_TYPE__)a; } };

#define WAIT_V(n) asm volatile("s_waitcnt vmcnt(" #n ")" ::: "memory")
#define BARRIER() asm volatile("s_barrier" ::: "memory")
#define RLX __ATOMIC_RELAXED
#define SC_AGENT __HIP_MEMORY_SCOPE_AGENT
#define SC_SYS __HIP_MEMORY_SCOPE_SYSTEM

template <int AUX> __device__ __forceinline__ void dma16(__amdgpu_buffer_rsrc_t rsrc, LAS char* dst, unsigned voff, unsigned soff) {
    __builtin_amdgcn_raw_ptr_buffer_load_lds(rsrc, (LAS void*)dst, 16, voff, soff, 0, AUX);
}
__device__ __forceinline__ void st16_sys(char* p, u32x4 v) {
    asm volatile("global_store_dwordx4 %0, %1, off sc0 sc1\n\ts_nop 1" :: "v"(p), "v"(v) : "memory");
}
__device__ __forceinline__ void st16_out(char* p, u32x4 v) {
    asm volatile("global_store_dwordx4 %0, %1, off sc1\n\ts_nop 1" :: "v"(p), "v"(v) : "memory");
}
__device__ __forceinline__ u32x4 ld16_sys(const char* p) {
    u32x4 v; asm volatile("global_load_dwordx4 %0, %1, off sc0 sc1\n\ts_waitcnt vmcnt(0)" : "=&v"(v) : "v"(p) : "memory"); return v;
}
__device__ __forceinline__ unsigned q4(f32x4 v, float inv) {
    unsigned r = 0;
    r = __builtin_amdgcn_cvt_pk_u8_f32(v[0] * inv, 0, r);
    r = __builtin_amdgcn_cvt_pk_u8_f32(v[1] * inv, 1, r);
    r = __builtin_amdgcn_cvt_pk_u8_f32(v[2] * inv, 2, r);
    r = __builtin_amdgcn_cvt_pk_u8_f32(v[3] * inv, 3, r);
    const unsigned h = r & 0x80808080u;
    return (r | (h - (h >> 7))) & 0x7f7f7f7fu;
}
template <int CTRL, int RMASK> __device__ __forceinline__ unsigned dpp0(unsigned x) { return (unsigned)__builtin_amdgcn_update_dpp(0, (int)x, CTRL, RMASK, 0xf, true); }
__device__ __forceinline__ unsigned wave_scan_add(unsigned x) {
    x += dpp0<0x111, 0xf>(x); x += dpp0<0x112, 0xf>(x); x += dpp0<0x114, 0xf>(x); x += dpp0<0x118, 0xf>(x);
    x += dpp0<0x142, 0xa>(x); x += dpp0<0x143, 0xc>(x); return x;
}
__device__ __forceinline__ unsigned wave_max_u(unsigned x) {
    x = max(x, dpp0<0x111, 0xf>(x)); x = max(x, dpp0<0x112, 0xf>(x)); x = max(x, dpp0<0x114, 0xf>(x)); x = max(x, dpp0<0x118, 0xf>(x));
    x = max(x, dpp0<0x142, 0xa>(x)); x = max(x, dpp0<0x143, 0xc>(x)); return (unsigned)__builtin_amdgcn_readlane((int)x, 63);
}
__device__ __forceinline__ void stream_offsets(const u32x4 keep, unsigned (&start)[4], unsigned (&cum)[4]) {
    const unsigned c0 = __builtin_popcount(keep[0]), c1 = __builtin_popcount(keep[1]), c2 = __builtin_popcount(keep[2]), c3 = __builtin_popcount(keep[3]);
    const unsigned a = wave_scan_add(c0 | (c1 << 16)), b = wave_scan_add(c2 | (c3 << 16));
    const unsigned ta = (unsigned)__builtin_amdgcn_readlane((int)a, 63), tb = (unsigned)__builtin_amdgcn_readlane((int)b, 63);
    cum[0] = ta & 0xffffu; cum[1] = cum[0] + (ta >> 16); cum[2] = cum[1] + (tb & 0xffffu); cum[3] = cum[2] + (tb >> 16);
    start[0] = (a & 0xffffu) - c0; start[1] = cum[0] + (a >> 16) - c1; start[2] = cum[1] + (b & 0xffffu) - c2; start[3] = cum[2] + (b >> 16) - c3;
}
__device__ __forceinline__ unsigned stream_tag(unsigned call) { const unsigned n = call % 3u + 1u; return ((n & 1u) << 7) | ((n & 2u) << 14); }
__device__ __forceinline__ u32x4 granule_pack(u32x4 d, unsigned w) {
    const unsigned x = (w | (w << 8)) & 0x00ff00ffu, u = (x | (x << 4)) & 0x0f0f0f0fu;
    return (u32x4){d[0] | ((u << 7) & 0x80808080u), d[1] | ((u << 6) & 0x80808080u), d[2] | ((u << 5) & 0x80808080u), d[3] | ((u << 4) & 0x80808080u)};
}
__device__ __forceinline__ unsigned granule_word(u32x4 d) {
    const unsigned u = ((d[0] >> 7) & 0x01010101u) | ((d[1] >> 6) & 0x02020202u) | ((d[2] >> 5) & 0x04040404u) | ((d[3] >> 4) & 0x08080808u);
    const unsigned x = (u | (u >> 4)) & 0x00ff00ffu;
    return (x | (x >> 8)) & 0xffffu;
}
__device__ __forceinline__ void wire_split(const unsigned (&cum)[4], unsigned& head, unsigned& wire) {
    head = cum[0] / 144u * 128u;
    wire = head + (cum[3] - (head + (head >> 3)) + 17u) / 18u * 16u;
}
__device__ __forceinline__ u64 slot_ticks(int pos, unsigned bytes) {
    return (u64)(((unsigned)(32 - SLOT0 + 3 * pos) * bytes * 8u) / (9u * LINK_BPT));
}
__global__ __launch_bounds__(512, 2) void fused_kernel(const Params p) {
    extern __shared__ __attribute__((aligned(16))) char smem[];
    asm volatile("" :: "s"((LAS char*)smem));
    const LdsBase lds;
    asm volatile("" :: "s"(p.x), "s"(p.w), "s"(p.myws), "s"(p.rank));
    const int tid = threadIdx.x, lane = tid & 63;
    const int wid = __builtin_amdgcn_readfirstlane(tid >> 6);
    const int b = blockIdx.x, peer = b & 7, idx = b >> 3;
    auto col_block = [](int i) { return (int)((0xd951fecba8764320ull >> (4 * i)) & 15u); };
    const int cbl = col_block(idx >> 1), mh = idx & 1;
    const int m0 = mh * BM, n0 = (peer * 16 + cbl) * BN;
    const bool local = peer == p.rank;
    char* const myws = p.myws;
    unsigned* const tmo = (unsigned*)(myws + WS_TMO);
    const int fr = lane & 15, g = lane >> 4;
    const int w4 = wid & 3, wm = w4 >> 1, wn = w4 & 1;

    if (__builtin_expect(wid >= 4, 1)) {
        const int lw = w4;
        const unsigned tstart = (unsigned)__builtin_amdgcn_s_memrealtime();
        __amdgpu_buffer_rsrc_t rx = __builtin_amdgcn_make_buffer_rsrc((void*)p.x, 0, MS * KD * 2, 0x00020000);
        __amdgpu_buffer_rsrc_t rw = __builtin_amdgcn_make_buffer_rsrc((void*)p.w, 0, (unsigned)((size_t)KD * NG * 2), 0x00020000);
        const unsigned xrow = m0 + lw * 64 + (lane >> 3);
        const unsigned xv0 = xrow * (KD * 2) + (((lane & 7) ^ (lane >> 4)) << 4);
        const unsigned xv1 = xrow * (KD * 2) + (((lane & 7) ^ (4 + (lane >> 4))) << 4);
        const unsigned wv = (lw * 4 + (lane >> 4)) * (NG * 2) + n0 * 2 + (((lane & 15) ^ (((lane >> 4) | ((lw & 2) << 1)) << 1)) << 4);
        auto issueX = [&](int t, int slot) {
            LAS char* d = lds + XBASE + slot * XSLOT + lw * 64 * 128;
#pragma unroll
            for (int i = 0; i < 8; ++i) dma16<0>(rx, d + i * 1024, (i & 1) ? xv1 : xv0, (unsigned)(t * 128 + i * 8 * KD * 2));
        };
        auto issueW = [&](int t) {
#pragma unroll
            for (int i = 0; i < 4; ++i) dma16<WAUX>(rw, lds + WBASE + (t & 3) * WSLOT + i * 4096 + lw * 1024, wv, (unsigned)(t * 64 + i * 16) * (NG * 2));
        };
        auto issueWs = [&](int t) {
#pragma unroll
            for (int i = 0; i < 4; ++i) dma16<0>(rw, lds + WBASE + (t & 3) * WSLOT + i * 4096 + lw * 1024, wv, (unsigned)(t * 64 + i * 16) * (NG * 2));
        };
        unsigned touch = 0;
        const unsigned tv = b * 16384 + (lw >> 1) * 8192 + ((lw & 1) * 8 + (lane & 7)) * 128;
        auto touchX = [&](unsigned kt0) { asm volatile("buffer_load_dword %0, %1, %2, %3 offen" : "+v"(touch) : "v"(tv), "s"(rx), "s"(kt0 * 128u) : "memory"); };
        int xs = 1;
        auto nextX = [&](int t) { issueX(t, xs); xs = (xs == NXS - 1) ? 0 : xs + 1; };
        issueW(0); issueX(0, 0); issueW(1); issueX(1, 1);
        WAIT_V(8);
        BARRIER();
        unsigned cv;
        issueW(2); touchX(3);
        asm volatile("global_load_dword %0, %1, off sc1" : "=v"(cv) : "v"(myws + WS_CTR + b * 64) : "memory");
        issueX(2, 2); issueW(3);
        WAIT_V(14); BARRIER();
        issueX(3, 0); issueW(4); touchX(19); touchX(35); touchX(48);
        asm volatile("s_waitcnt vmcnt(15)" : "+v"(cv) :: "memory");
        BARRIER();
        const unsigned call = __builtin_amdgcn_readfirstlane(cv) + 1u;
        {
            const char* wp;
            if (lw == 0) wp = local ? p.ws[idx & 7] + WS_GG + 8192 : p.ws[peer] + WS_IN + p.rank * INBOX + 3072 + (idx & 1) * (INBOX - 4096);
            else if (lw == 2) wp = (idx & 2) ? myws + WS_IN + peer * INBOX + 3072 + (idx & 1) * (INBOX - 4096) : p.out + (size_t)(peer * MS + (idx & 1) * (MS - 1)) * NS;
            else wp = (const char*)p.w + (size_t)(idx * 2 + (lw >> 1)) * (BK * NG * 2) + 64 * 1024;
            asm volatile("s_load_dword s95, %0, 0x0" :: "s"(wp) : "s95");
        }
        unsigned tlast = 0;
        auto pace = [&](int n) {
            if (lw == 1) {
                const unsigned tgt = max(PACE_T0 + (unsigned)n * PACE_TAU, tlast + PACE_MIN);
                while ((unsigned)__builtin_amdgcn_s_memrealtime() - tstart < tgt) __builtin_amdgcn_s_sleep(1);
            }
        };
        int t = 2;
        const unsigned wl = ((unsigned)(__UINTPTR_TYPE__)p.w + (unsigned)n0 * 2u) >> 7;
        const bool slow = ((wl - 2u) & 7u) < 2u || ((wl - 62u) & 127u) < 2u;
        const bool shift = slow ? mh : (((unsigned)(__UINTPTR_TYPE__)p.x >> 7) & 1u);
        if (shift) { nextX(4); issueW(5); WAIT_V(12); BARRIER(); t = 3; }
        if (slow) {
            for (; t < NT - 4; t += 2) {
                nextX(t + 2); issueWs(t + 3); issueWs(t + 4);
                WAIT_V(16); BARRIER();
                nextX(t + 3);
                pace(t + 2);
                WAIT_V(12); BARRIER();
                if (lw == 1) tlast = (unsigned)__builtin_amdgcn_s_memrealtime() - tstart;
            }
        } else {
            for (; t < NT - 4; t += 2) {
                nextX(t + 2); issueW(t + 3); issueW(t + 4);
                WAIT_V(16); BARRIER();
                nextX(t + 3);
                pace(t + 2);
                WAIT_V(12); BARRIER();
                if (lw == 1) tlast = (unsigned)__builtin_amdgcn_s_memrealtime() - tstart;
            }
        }
        if (!shift) { nextX(NT - 2); issueW(NT - 1); WAIT_V(12); BARRIER(); }
        nextX(NT - 1);
        char* const inbox = myws + WS_IN + peer * INBOX;
        char* const rs = inbox + IN_RS + (size_t)(idx * 4 + w4) * STRM;
#pragma unroll
        for (int i = 0; i < 4; ++i) st16_sys(rs + (call & 1u) * 4096 + i * 1024 + lane * 16, (u32x4){0u, 0u, 0u, 0u});
        WAIT_V(12); BARRIER();
        WAIT_V(4); BARRIER();
        BARRIER();
        if (wid == 4) asm volatile("ds_write_b128 %0, %1\n\ts_waitcnt lgkmcnt(0)" :: "v"((unsigned)EP_SCALE), "v"((u32x4){0u, 0u, 0u, 0u}) : "memory");
        WAIT_V(0);
        asm volatile("s_waitcnt lgkmcnt(0)" :: "v"(touch) : "s95");
        BARRIER();
        const unsigned tag = stream_tag(call);
        const u64 t0 = __builtin_amdgcn_s_memrealtime();
        if (local) {
            if (idx < NAGG) {
                bool dead = false, ok = false; unsigned m = 0;
                const char* gp = myws + WS_LG + w4 * 2048 + lane * 16;
                char* base = p.ws[0];
#pragma unroll
                for (int r = 1; r < NR; ++r) base = (lane & 7) == r ? p.ws[r] : base;
                base += WS_GG + ((call & 1) * NREP + (lane >> 3)) * 128 + p.rank * 8;
                asm volatile("" : "+v"(base));
                for (;;) {
                    if (!ok) {
                        u32x4 v0, v1;
                        asm volatile("global_load_dwordx4 %0, %2, off sc1\n\tglobal_load_dwordx4 %1, %2, off offset:1024 sc1\n\ts_waitcnt vmcnt(0)" : "=&v"(v0), "=&v"(v1) : "v"(gp) : "memory");
                        m = max(max(v0[0], v0[2]), max(v1[0], v1[2]));
                        ok = v0[1] == call && v0[3] == call && v1[1] == call && v1[3] == call;
                    }
                    if (__all(ok)) break;
                    if (__builtin_amdgcn_s_memrealtime() - t0 > DEADLINE) { dead = true; break; }
                    __builtin_amdgcn_s_sleep(1);
                }
                if (!dead) {
                    m = wave_max_u(m);
                    unsigned cnt = 0;
                    if (lane == 0) {
                        __hip_atomic_fetch_max((LAS unsigned*)(lds + EP_SCALE + 8), m, RLX, __HIP_MEMORY_SCOPE_WORKGROUP);
                        cnt = __hip_atomic_fetch_add((LAS unsigned*)(lds + EP_SCALE + 12), 1u, RLX, __HIP_MEMORY_SCOPE_WORKGROUP);
                    }
                    if (__builtin_amdgcn_readfirstlane(cnt) == 3) {
                        m = __hip_atomic_load((LAS unsigned*)(lds + EP_SCALE + 8), RLX, __HIP_MEMORY_SCOPE_WORKGROUP);
#pragma unroll
                        for (int i = 0; i < NREP / 8; ++i) __hip_atomic_store((u64*)(base + i * 1024), ((u64)call << 32) | m, RLX, SC_SYS);
                    }
                } else if (lane == 0) __hip_atomic_store(tmo, call, RLX, SC_AGENT);
            }
        } else {
            bool dead = false;
            {
                const unsigned nib = lane & 15; unsigned pos = lane >> 4, sel = 0;
#pragma unroll
                for (int e = 0; e < 4; ++e) { const bool k = (nib >> e) & 1; sel |= (k ? pos : 0x0cu) << (8 * e); pos += k; }
                *(LAS unsigned*)(lds + EP_LUT + wid * 512 + lane * 4) = sel;
            }
            unsigned n = 0;
            while (__hip_atomic_load((unsigned*)(inbox + IN_MF + (idx * 4 + w4) * 16), RLX, SC_SYS) < call) {
                if (((++n & 63u) == 0u && __hip_atomic_load(tmo, RLX, SC_AGENT) == call) || __builtin_amdgcn_s_memrealtime() - t0 > DEADLINE) { dead = true; break; }
                __builtin_amdgcn_s_sleep(32);
            }
            const u32x4 keep = ld16_sys(inbox + IN_MR + (size_t)((call & 1) * 128 + idx * 4 + w4) * 1024 + lane * 16);
            unsigned start[4], cum[4];
            stream_offsets(keep, start, cum);
            LAS char* const lb = lds + EP_RCV + w4 * RCVB;
            *(LAS u32x4*)(lb + lane * 32) = keep;
            *(LAS u32x4*)(lb + lane * 32 + 16) = (u32x4){start[0], start[1], start[2], start[3]};
            unsigned addr[32], sel[32];
#pragma unroll
            for (int gg = 0; gg < 4; ++gg) {
                const u32x4 kp = *(LAS u32x4*)(lb + ((lane >> 2) + 16 * gg) * 32), stt = *(LAS u32x4*)(lb + ((lane >> 2) + 16 * gg) * 32 + 16);
#pragma unroll
                for (int i = 0; i < 8; ++i) {
                    const unsigned sh = (i & 1) * 16 + (lane & 3) * 4;
                    const unsigned off = stt[i >> 1] + __builtin_popcount(kp[i >> 1] & ((1u << sh) - 1u)), nib = (kp[i >> 1] >> sh) & 15u;
                    sel[i * 4 + gg] = *(LAS unsigned*)(lds + EP_LUT + wid * 512 + (((off & 3u) << 4) | nib) * 4);
                    addr[i * 4 + gg] = EP_RCV + w4 * RCVB + (off & ~3u);
                }
            }
#pragma unroll
            for (int i = 0; i < 32; ++i) asm volatile("" : "+v"(addr[i]), "+v"(sel[i]));
            unsigned st;
            while ((st = __builtin_amdgcn_readfirstlane(__hip_atomic_load((LAS unsigned*)(lds + EP_SCALE + 4), RLX, __HIP_MEMORY_SCOPE_WORKGROUP))) == 0u) __builtin_amdgcn_s_sleep(8);
            const unsigned total = cum[3];
            const int sidx = (idx - 4 - 8 * w4) & 31;
            {
                const unsigned dt = ((unsigned)slot_ticks(sidx, total) + RWAIT) << 2;
                while (((unsigned)__builtin_amdgcn_s_memrealtime() << 2) - (st & ~3u) < dt) __builtin_amdgcn_s_sleep(2);
            }
            char* o = p.out + (size_t)(peer * MS + m0 + wm * 128 + (lane >> 2)) * NS + col_block(sidx >> 1) * BN + wn * 64 + (lane & 3) * 16;
            unsigned head, wire;
            wire_split(cum, head, wire);
            const __amdgpu_buffer_rsrc_t rr = __builtin_amdgcn_make_buffer_rsrc((void*)rs, 0, wire, 0x00020000);
            unsigned pos = 0;
            while (pos < wire && !dead) {
                u32x4 v[4];
#pragma unroll
                for (int i = 0; i < 4; ++i) v[i] = __builtin_amdgcn_raw_buffer_load_b128(rr, pos + lane * 16 + i * 1024, 0, 17);
                unsigned adv = 4096;
#pragma unroll
                for (int i = 3; i >= 0; --i) {
                    const unsigned off = pos + i * 1024 + lane * 16, w = granule_word(v[i]);
                    const bool in = off < wire;
                    const u64 bad = __ballot(in && (w & 0x8080u) != tag);
                    if (in) {
                        const bool hd = off < head;
                        *(LAS u32x4*)(lb + off + (hd ? 0u : head >> 3)) = v[i] & 0x7f7f7f7fu;
                        *(LAS u16*)(lb + (hd ? head : wire) + (off >> 3)) = (u16)(w & 0x7f7fu);
                    }
                    if (bad) adv = i * 1024 + (unsigned)__builtin_ctzll(bad) * 16u;
                }
                pos += adv & ~127u;
                if (pos < wire) {
                    if (((++n & 63u) == 0u && __hip_atomic_load(tmo, RLX, SC_AGENT) == call) || __builtin_amdgcn_s_memrealtime() - t0 > DEADLINE) dead = true;
                    __builtin_amdgcn_s_sleep(1);
                }
            }
            unsigned d[32];
#pragma unroll
            for (int i = 0; i < 32; ++i) {
                const LAS unsigned* sp = (const LAS unsigned*)(lds + addr[i]);
                d[i] = __builtin_amdgcn_perm(sp[1], sp[0], sel[i]);
            }
#pragma unroll
            for (int i = 0; i < 8; ++i) {
                u32x4 v = {d[i * 4], d[i * 4 + 1], d[i * 4 + 2], d[i * 4 + 3]};
                if (dead) v = (u32x4){0x80808080u, 0x80808080u, 0x80808080u, 0x80808080u};
                st16_out(o + (size_t)i * 16 * NS, v);
            }
            if (dead && lane == 0) __hip_atomic_store(tmo, call, RLX, SC_AGENT);
        }
    } else {
        const unsigned xa0 = XBASE + wm * 16384 + fr * 128 + ((g ^ (fr >> 1)) << 4);
        const int qq = fr >> 2, pp = lane & 3;
        const int f = qq | ((g & 1) << 2);
        const unsigned wa0 = WBASE + (8 * g + qq) * 256 + ((((wn * 8) + (pp >> 1)) ^ (f << 1)) << 4) + (pp & 1) * 8;
        const unsigned c0 = __hip_atomic_load((unsigned*)(myws + WS_CTR + b * 64), RLX, SC_AGENT);
        f32x4 acc[8][4];
#pragma unroll
        for (int i = 0; i < 8; ++i)
#pragma unroll
            for (int j = 0; j < 4; ++j) acc[i][j] = (f32x4){0.f, 0.f, 0.f, 0.f};
        bf16x8 XP[2][2], XQ[2][2], WA[4][2], WB[4][2];
#define PIN() __builtin_amdgcn_sched_barrier(0)
#define LGKM(n) do { __builtin_amdgcn_s_waitcnt(0xC07F | ((n) << 8)); PIN(); } while (0)
        auto rdX = [&](unsigned a, int off) { bf16x8 v = *(LAS bf16x8*)(lds + a + off); PIN(); return v; };
        auto rdW = [&](unsigned a, int off) { s16x4 v = __builtin_amdgcn_ds_read_tr16_b64_v4i16((LAS s16x4*)(lds + a + off)); PIN(); return v; };
        auto cat = [&](s16x4 lo, s16x4 hi) { return __builtin_bit_cast(bf16x8, __builtin_shufflevector(lo, hi, 0, 1, 2, 3, 4, 5, 6, 7)); };
        auto quarter = [&](int mf, bf16x8 (&X)[2][2], bf16x8 (&W)[4][2], auto fill) {
#pragma unroll
            for (int n = 0; n < 16; ++n) {
                const int i = n >> 3, j = (n >> 1) & 3, ks = n & 1;
                asm volatile("v_mfma_f32_16x16x32_bf16 %0, %1, %2, %0" : "+v"(acc[mf + i][j]) : "v"(W[j][ks]), "v"(X[i][ks])); PIN();
                fill(n);
            }
        };
        int xs = 0, ws = WSLOT;
        unsigned xc0 = xa0, xc1 = xa0 ^ 64, wq[4];
#pragma unroll
        for (int j = 0; j < 4; ++j) wq[j] = (wa0 ^ (j << 5)) + WSLOT;
        auto tile = [&](bf16x8 (&Wc)[4][2], bf16x8 (&Wn)[4][2]) {
            s16x4 lo;
            auto fx = [&](bf16x8 (&D)[2][2], int mf, int n) { if (n < 4) D[n >> 1][n & 1] = rdX((n & 1) ? xc1 : xc0, (mf + (n >> 1)) * 2048); };
            auto fw = [&](int f0, int cnt, int n) {
                const int m = n - 4;
                if (m >= 0 && m < 2 * cnt) {
                    const int f = f0 + (m >> 1), j = f >> 1, ks = f & 1;
                    if (!(m & 1)) lo = rdW(wq[j], ks * 8192); else Wn[j][ks] = cat(lo, rdW(wq[j], ks * 8192 + 1024));
                }
            };
            quarter(0, XP, Wc, [&](int n) { fx(XQ, 2, n); fw(0, 3, n); if (n == 14) LGKM(6); });
            quarter(2, XQ, Wc, [&](int n) { fx(XP, 4, n); fw(3, 3, n); if (n == 14) LGKM(6); });
            quarter(4, XP, Wc, [&](int n) {
                fx(XQ, 6, n); fw(6, 2, n);
                if (n == 9) { xs = (xs == NXS - 1) ? 0 : xs + 1; asm volatile("" : "+s"(xs)); PIN(); }
                if (n == 11) { xc0 = xa0 + xs * XSLOT; asm volatile("" : "+v"(xc0)); PIN(); }
                if (n == 12) { xc1 = (xa0 ^ 64) + xs * XSLOT; asm volatile("" : "+v"(xc1)); PIN(); }
                if (n == 14) LGKM(0);
            });
            BARRIER(); PIN();
            quarter(6, XQ, Wc, [&](int n) {
                fx(XP, 0, n);
                if (n == 5) { ws = (ws + WSLOT) & (NWS * WSLOT - 1); asm volatile("" : "+s"(ws)); PIN(); }
                if (n >= 7 && n < 11) { wq[n - 7] = (wa0 ^ ((n - 7) << 5)) + ws; asm volatile("" : "+v"(wq[n - 7])); PIN(); }
                if (n == 14) LGKM(0);
            });
        };
        const unsigned call = __builtin_amdgcn_readfirstlane(c0) + 1u;
#pragma unroll
        for (int i = 0; i < 8; ++i) asm volatile("" : "+v"(acc[i][0]), "+v"(acc[i][1]), "+v"(acc[i][2]), "+v"(acc[i][3]));
        LGKM(0);
        BARRIER(); PIN();
#pragma unroll
        for (int j = 0; j < 4; ++j)
#pragma unroll
            for (int ks = 0; ks < 2; ++ks) { const s16x4 lo = rdW(wa0 ^ (j << 5), ks * 8192); WA[j][ks] = cat(lo, rdW(wa0 ^ (j << 5), ks * 8192 + 1024)); }
#pragma unroll
        for (int n = 0; n < 4; ++n) XP[n >> 1][n & 1] = rdX((n & 1) ? xc1 : xc0, (n >> 1) * 2048);
        LGKM(0);
#pragma unroll 1
        for (int t = 0; t < NT; t += 2) { tile(WA, WB); tile(WB, WA); }
        asm volatile("s_nop 15" ::: "memory");
#pragma unroll
        for (int i = 0; i < 8; ++i) asm volatile("" : "+v"(acc[i][0]), "+v"(acc[i][1]), "+v"(acc[i][2]), "+v"(acc[i][3]));
        int mx = 0;
#pragma unroll
        for (int i = 0; i < 8; ++i)
#pragma unroll
            for (int j = 0; j < 4; ++j) {
                mx = max(max(mx, __float_as_int(acc[i][j][0])), __float_as_int(acc[i][j][1]));
                mx = max(max(mx, __float_as_int(acc[i][j][2])), __float_as_int(acc[i][j][3]));
            }
        unsigned wmax = wave_max_u((unsigned)mx);
        asm volatile("" : "+s"(wmax));
        BARRIER();
        if (lane == 0) {
            __hip_atomic_store((u64*)(myws + WS_LG) + b * 4 + wid, ((u64)call << 32) | wmax, RLX, SC_AGENT);
            if (wid == 0) __hip_atomic_store((unsigned*)(myws + WS_CTR + b * 64), call, RLX, SC_AGENT);
        }
        const u64 t0 = __builtin_amdgcn_s_memrealtime();
        auto wait_scale = [&](bool& dead) {
            LAS u64* sw = (LAS u64*)(lds + EP_SCALE);
            if (wid == 0) {
                unsigned gm, n = 0;
                for (;;) {
                    const u64 v = __hip_atomic_load((u64*)(myws + WS_GG + ((call & 1) * NREP + (idx & (NREP - 1))) * 128) + (lane & 7), RLX, SC_SYS);
                    gm = (unsigned)v;
                    if (__all((unsigned)(v >> 32) == call)) break;
                    if (((++n & 63u) == 0u && __hip_atomic_load(tmo, RLX, SC_AGENT) == call) || __builtin_amdgcn_s_memrealtime() - t0 > DEADLINE) { dead = true; break; }
                    __builtin_amdgcn_s_sleep(1);
                }
                if (dead && lane == 0) __hip_atomic_store(tmo, call, RLX, SC_AGENT);
                const float am = __uint_as_float(wave_max_u(gm));
                const float inv = am > 0.f ? 127.0f / am : 0.f;
                __hip_atomic_store(sw, ((u64)(((unsigned)__builtin_amdgcn_s_memrealtime() << 2) | (dead ? 2u : 1u)) << 32) | __float_as_uint(inv), RLX, __HIP_MEMORY_SCOPE_WORKGROUP);
                return inv;
            }
            u64 v;
            while (((v = __hip_atomic_load(sw, RLX, __HIP_MEMORY_SCOPE_WORKGROUP)) >> 32) == 0u) __builtin_amdgcn_s_sleep(1);
            dead = ((v >> 32) & 3u) == 2u;
            return __uint_as_float((unsigned)v);
        };
        bool dead = false;
        if (local) {
            const float inv = wait_scale(dead);
            char* o = p.out + (size_t)(p.rank * MS + m0 + wm * 128 + (lane >> 2)) * NS + cbl * BN + wn * 64 + (lane & 3) * 16;
            LAS char* const lb = lds + EP_STRM + wid * STRM;
#pragma unroll
            for (int i = 0; i < 8; ++i)
#pragma unroll
                for (int j = 0; j < 4; ++j) *(LAS unsigned*)(lb + i * 1024 + fr * 64 + ((j ^ (fr >> 2)) & 3) * 16 + g * 4) = q4(acc[i][j], inv);
            u32x4 v[8];
#pragma unroll
            for (int i = 0; i < 8; ++i) v[i] = *(LAS u32x4*)(lb + i * 1024 + (lane >> 2) * 64 + ((lane ^ (lane >> 4)) & 3) * 16);
#pragma unroll
            for (int i = 0; i < 8; ++i) st16_out(o + (size_t)i * 16 * NS, dead ? (u32x4){0x80808080u, 0x80808080u, 0x80808080u, 0x80808080u} : v[i]);
        } else {
            const int slot = ((idx + 4 + 8 * wid) & 31) * 4 + wid;
            char* const pws = p.ws[peer] + WS_IN + p.rank * INBOX;
            const unsigned tag = stream_tag(call);
            u32x4 keep;
#pragma unroll
            for (int k = 0; k < 4; ++k) {
                unsigned m = 0;
#pragma unroll
                for (int u = 7; u >= 0; --u)
#pragma unroll
                    for (int e = 3; e >= 0; --e) m = __builtin_amdgcn_alignbit(m, __float_as_uint(acc[2 * k + (u >> 2)][u & 3][e]), 31);
                keep[k] = ~m;
            }
            WAIT_V(0);
            st16_sys(pws + IN_MR + (size_t)((call & 1) * 128 + slot) * 1024 + lane * 16, keep);
            {
                const unsigned nib = lane & 15; unsigned pos = lane >> 4; u64 sel = 0x0c0c0c0c0c0c0c0cull;
#pragma unroll
                for (int e = 0; e < 4; ++e) { const bool k = (nib >> e) & 1; const u64 x = (u64)(0x0cu ^ (unsigned)e) << (8 * pos); sel ^= k ? x : 0ull; pos += k; }
                *(LAS u64*)(lds + EP_LUT + wid * 512 + lane * 8) = sel;
            }
#pragma unroll
            for (int i = 0; i < 9; ++i) *(LAS u32x4*)(lds + EP_STRM + wid * STRM + i * 1024 + lane * 16) = (u32x4){0u, 0u, 0u, 0u};
            unsigned start[4], cum[4];
            stream_offsets(keep, start, cum);
            unsigned addr[32]; u32x2 sel[32];
#pragma unroll
            for (int k = 0; k < 4; ++k) {
                unsigned off = start[k];
#pragma unroll
                for (int u = 0; u < 8; ++u) {
                    const unsigned nib = (keep[k] >> (4 * u)) & 15u;
                    sel[k * 8 + u] = *(LAS u32x2*)(lds + EP_LUT + wid * 512 + (((off & 3u) << 4) | nib) * 8);
                    addr[k * 8 + u] = EP_STRM + wid * STRM + (off & ~3u);
                    off += __builtin_popcount(nib);
                }
            }
#pragma unroll
            for (int i = 0; i < 32; ++i) asm volatile("" : "+v"(addr[i]), "+v"(sel[i]));
            WAIT_V(0);
            if (lane == 0) __hip_atomic_store((unsigned*)(pws + IN_MF + slot * 16), call, RLX, SC_SYS);
            const float inv = wait_scale(dead);
            if (!dead) {
                char* const rs = pws + IN_RS + (size_t)slot * STRM + lane * 16;
                const u64 ta = __builtin_amdgcn_s_memrealtime();
                const u64 dt = slot_ticks(idx, cum[3]);
                unsigned head, wire;
                wire_split(cum, head, wire);
                LAS char* const img = lds + (EP_STRM + wid * STRM);
#pragma unroll
                for (int k = 0; k < 4; ++k) {
#pragma unroll
                    for (int u = 0; u < 8; ++u) {
                        const unsigned qd = q4(acc[2 * k + (u >> 2)][u & 3], inv);
                        LAS unsigned* d = (LAS unsigned*)(lds + addr[k * 8 + u]);
                        __hip_atomic_fetch_or(d, __builtin_amdgcn_perm(0u, qd, sel[k * 8 + u][0]), RLX, __HIP_MEMORY_SCOPE_WORKGROUP);
                        __hip_atomic_fetch_or(d + 1, __builtin_amdgcn_perm(0u, qd, sel[k * 8 + u][1]), RLX, __HIP_MEMORY_SCOPE_WORKGROUP);
                    }
                    asm volatile("" ::: "memory");
                    if (k == 0)
                        for (unsigned o = 0; o < head; o += 1024)
                            if (o + lane * 16 < head) st16_sys(rs + o, granule_pack(*(LAS u32x4*)(img + o + lane * 16), *(LAS u16*)(img + head + (o >> 3) + lane * 2) | tag));
                }
                const unsigned lim = (wire + 63u) & ~63u;
                u32x4 pk[6];
#pragma unroll
                for (int i = 0; i < 6; ++i) {
                    pk[i] = granule_pack(*(LAS u32x4*)(img + head + (head >> 3) + i * 1024 + lane * 16), *(LAS u16*)(img + wire + (head >> 3) + i * 128 + lane * 2) | tag);
                    asm volatile("" : "+v"(pk[i]));
                }
                while (__builtin_amdgcn_s_memrealtime() - ta < dt) __builtin_amdgcn_s_sleep(1);
#pragma unroll
                for (int i = 0; i < 6; ++i)
                    if (head + i * 1024 + lane * 16 < lim) st16_sys(rs + head + i * 1024, pk[i]);
            }
        }
    }
}

extern "C" void kernel_launch(int rank, int nranks, ncclComm_t comm, hipStream_t stream,
                              void* const* d_in, const int* in_sizes, int n_in,
                              void* d_out, int out_size, const int* in_dtypes, int out_dtype,
                              void* const* d_out_all, void* const* d_ws_all, size_t ws_bytes) {
    if (nranks != NR || (unsigned)rank >= (unsigned)NR || ws_bytes < WS_END) return;
    static bool init[NR];
    if (!init[rank]) {
        HIP_CHECK(hipFuncSetAttribute((const void*)fused_kernel, hipFuncAttributeMaxDynamicSharedMemorySize, LDS_BYTES));
        init[rank] = true;
    }
    Params p;
    p.x = (const u16*)d_in[0]; p.w = (const u16*)d_in[1]; p.out = (char*)d_out;
    for (int r = 0; r < NR; ++r) p.ws[r] = (char*)d_ws_all[r];
    p.rank = rank; p.pad = 0; p.myws = p.ws[rank];
    fused_kernel<<<NWG, 512, LDS_BYTES, stream>>>(p);
    HIP_CHECK(hipGetLastError());
}
```
